```python
import numpy as np

kernel_name = "gpu_gemm_a2a_m8192_k8192_n8192_bf16_none"


ABSMAX_RTOL = 0.008

COMPUTE_DTYPE = "bfloat16"
EPILOGUE = "none"
IN_QUANT = None
OUT_QUANT = None
IN_DTYPES = {"x": "bfloat16", "w_mat": "bfloat16"}
WORLD_SIZE = 8

def _to_bf16(x):
    u = np.ascontiguousarray(x, dtype=np.float32).view(np.uint32)
    return ((u + 0x7FFF + ((u >> 16) & 1)) & 0xFFFF0000).view(np.float32)

def setup_inputs():

    x = _to_bf16(np.random.randn(8192, 8192).astype(np.float32))
    w_mat = _to_bf16((np.random.randn(8192, 8192) / np.sqrt(8192)).astype(np.float32))
    return {"x": x, "w_mat": w_mat}

def _epilogue(y):
    return y

def reference(x, w_mat):
    y = (x @ w_mat).astype(np.float32)
    return _epilogue(y)
```

```cpp
#include <hip/hip_runtime.h>
#include <rccl/rccl.h>
#include <cstdio>

#define HIP_CHECK(expr) do { hipError_t _e = (expr); if (_e != hipSuccess) { \
    fprintf(stderr, "HIP error %s at %s:%d\n", hipGetErrorString(_e), __FILE__, __LINE__); \
    return; } } while (0)

#define LAS __attribute__((address_space(3)))
typedef unsigned short u16;
typedef short bf16x8 __attribute__((ext_vector_type(8)));
typedef short s16x4 __attribute__((ext_vector_type(4)));
typedef float f32x4 __attribute__((ext_vector_type(4)));
typedef float f32x2 __attribute__((ext_vector_type(2)));
typedef unsigned u32x4 __attribute__((ext_vector_type(4)));
typedef __bf16 bf16x2_t __attribute__((ext_vector_type(2)));
constexpr int M_ = 1024, N_ = 8192, K_ = 8192, NR = 8, NO = 1024;
constexpr unsigned long long DEADLINE_TICKS = 100000000ull;
constexpr int WS_TICKET = 0, WS_TMO = 256, WS_FLAGS = 4096;

__device__ __forceinline__ unsigned cvtpk(float lo, float hi) { f32x2 v = {lo, hi}; bf16x2_t b = __builtin_convertvector(v, bf16x2_t); return __builtin_bit_cast(unsigned, b); }
__device__ __forceinline__ s16x4 trrd(LAS const char* p) { return __builtin_amdgcn_ds_read_tr16_b64_v4i16((LAS s16x4*)p); }
#define WAIT_V(n) asm volatile("s_waitcnt vmcnt(" #n ")" ::: "memory")
#define WAIT_L(n) asm volatile("s_waitcnt lgkmcnt(" #n ")" ::: "memory")
#define BAR __builtin_amdgcn_s_barrier()
#define SCHED __builtin_amdgcn_sched_barrier(0)
#define MEMF asm volatile("" ::: "memory")
#define BLDS(srd, voff, soff, GOFF, ldsbase, LOFF) do { unsigned tmp_; asm volatile("s_add_u32 m0, %1, %2\n\ts_add_u32 %0, %3, %4\n\tbuffer_load_dwordx4 %5, %6, %0 offen lds" \
   : "=&s"(tmp_) : "s"(ldsbase), "i"(LOFF), "s"(soff), "i"(GOFF), "v"(voff), "s"(srd) : "memory", "scc"); } while (0)
__device__ __forceinline__ u32x4 make_srd(const void* p, unsigned bytes) {
  unsigned long long a = (unsigned long long)p;
  u32x4 r; r[0] = __builtin_amdgcn_readfirstlane((unsigned)a); r[1] = __builtin_amdgcn_readfirstlane((unsigned)(a >> 32) & 0xffffu); r[2] = bytes; r[3] = 0x00020000u; return r;
}
struct Peers { u16* out[NR]; char* ws[NR]; };

__global__ __launch_bounds__(512, 2) void gemm_a2a(const u16* __restrict__ A, const u16* __restrict__ W, Peers peers, int rank) {
  extern __shared__ __attribute__((aligned(16))) char smem_[];
  LAS char* lds = (LAS char*)smem_;
  constexpr int N = N_, K = K_, M = M_;
  constexpr int HTB = 16384, STG = 3 * HTB;
  const int tid = threadIdx.x, wid = __builtin_amdgcn_readfirstlane(tid >> 6), lane = tid & 63, grp = wid >> 2, w4 = wid & 3, wr = w4 >> 1, wc = w4 & 1, fr = lane & 15, fq = lane >> 4;
  const int b = blockIdx.x, xcd = b & 7, j = b >> 3;
  const int dst = (xcd + rank) & 7;
  const int pm = j & 3, pnl = j >> 2, pn = dst * 8 + pnl;
  const int nt = K / 64;
  char* myws = peers.ws[rank];
  unsigned ticket = 0;
  if (tid == 0) ticket = __hip_atomic_fetch_add((unsigned*)(myws + WS_TICKET), 1u, __ATOMIC_RELAXED, __HIP_MEMORY_SCOPE_AGENT);
  unsigned voffA, voffB, voffA1, voffB1;
  { const int row = (lane >> 3); const int ch = (lane & 7) ^ ((row >> 1) & 7); voffA = (unsigned)(row * K + ch * 8) * 2u; }
  { const int row = 8 + (lane >> 3); const int ch = (lane & 7) ^ ((row >> 1) & 7); voffA1 = (unsigned)((row - 8) * K + ch * 8) * 2u; }
  { const int P = (lane >> 3); const int k = ((P & 1) << 3) | ((P >> 2) & 7), n64 = (P >> 1) & 1;
    const int sg = ((lane & 7) >> 1) ^ (k & 3), ch = sg * 2 + (lane & 1); voffB = (unsigned)(k * N + n64 * 64 + ch * 8) * 2u; }
  { const int P = 8 + (lane >> 3); const int k = ((P & 1) << 3) | ((P >> 2) & 7), n64 = (P >> 1) & 1;
    const int sg = ((lane & 7) >> 1) ^ (k & 3), ch = sg * 2 + (lane & 1); voffB1 = (unsigned)((k - 2) * N + n64 * 64 + ch * 8) * 2u; }
  const int aoff0 = (grp * 128 + wr * 64 + fr) * 128 + ((fq ^ (fr >> 1)) << 4), aoff1 = aoff0 ^ 64;
  const int g = fq, q = fr >> 2, p = fr & 3;
  const int brest = (g >> 1) * 4096 + q * 512 + (g & 1) * 128 + 8 * p + wc * 256 + (q << 5);
  const int bo0 = brest, bo1 = brest ^ 32, bo2 = brest ^ 64, bo3 = brest ^ 96;
  constexpr int kA = 128, kB = 64 * N * 2;
  const u32x4 srdA = make_srd(A, M * K * 2), srdB = make_srd(W, K * N * 2);
  const unsigned wB = (unsigned)(pn * 128 * 2 + (w4 * 16) * N * 2);
  const unsigned wA = (unsigned)(pm * 256 * K * 2 + (grp == 0 ? w4 * 16 : 64 + w4 * 48) * K * 2);
  const unsigned lb0 = __builtin_amdgcn_readfirstlane((unsigned)(size_t)lds);
  const unsigned lB = lb0 + (unsigned)w4 * 4096u, lA = lb0 + (unsigned)HTB + (unsigned)(grp == 0 ? w4 * 2048 : 8192 + w4 * 6144);
#define STAGE(st, kt) do { \
    if (grp == 0) { const unsigned gb = wB + (unsigned)(kt) * kB; const unsigned ga = wA + (unsigned)(kt) * kA; \
      BLDS(srdB, voffB, gb, 0, lB, (st) * STG); BLDS(srdB, voffB1, gb, 2 * N * 2, lB, (st) * STG + 1024); \
      BLDS(srdB, voffB, gb, 4 * N * 2, lB, (st) * STG + 2048); BLDS(srdB, voffB1, gb, 6 * N * 2, lB, (st) * STG + 3072); \
      BLDS(srdA, voffA, ga, 0, lA, (st) * STG); BLDS(srdA, voffA1, ga, 8 * K * 2, lA, (st) * STG + 1024); } \
    else { const unsigned ga = wA + (unsigned)(kt) * kA; \
      BLDS(srdA, voffA, ga, 0, lA, (st) * STG); BLDS(srdA, voffA1, ga, 8 * K * 2, lA, (st) * STG + 1024); \
      BLDS(srdA, voffA, ga, 16 * K * 2, lA, (st) * STG + 2048); BLDS(srdA, voffA1, ga, 24 * K * 2, lA, (st) * STG + 3072); \
      BLDS(srdA, voffA, ga, 32 * K * 2, lA, (st) * STG + 4096); BLDS(srdA, voffA1, ga, 40 * K * 2, lA, (st) * STG + 5120); } } while (0)
#define LDA(st) do { _Pragma("unroll") for (int m = 0; m < 4; ++m) { At[m][0] = *(const LAS bf16x8*)(lds + (st) * STG + HTB + aoff0 + m * 2048); At[m][1] = *(const LAS bf16x8*)(lds + (st) * STG + HTB + aoff1 + m * 2048); } } while (0)
#define LDB1(dst_, st, boff, s) do { s16x4 lo = trrd(lds + (st) * STG + boff + (s) * 8192), hi = trrd(lds + (st) * STG + boff + (s) * 8192 + 2048); dst_ = __builtin_shufflevector(lo, hi, 0, 1, 2, 3, 4, 5, 6, 7); } while (0)
#define LDB(st) do { LDB1(Bt[0][0], st, bo0, 0); LDB1(Bt[1][0], st, bo1, 0); LDB1(Bt[2][0], st, bo2, 0); LDB1(Bt[3][0], st, bo3, 0); \
                     LDB1(Bt[0][1], st, bo0, 1); LDB1(Bt[1][1], st, bo1, 1); LDB1(Bt[2][1], st, bo2, 1); LDB1(Bt[3][1], st, bo3, 1); } while (0)
#define MMA() do { __builtin_amdgcn_s_setprio(1); _Pragma("unroll") for (int k = 0; k < 2; ++k) _Pragma("unroll") for (int m = 0; m < 4; ++m) _Pragma("unroll") for (int n = 0; n < 4; ++n) \
    acc[m][n] = __builtin_amdgcn_mfma_f32_16x16x32_bf16(Bt[n][k], At[m][k], acc[m][n], 0, 0, 0); __builtin_amdgcn_s_setprio(0); } while (0)
  f32x4 acc[4][4];
#pragma unroll
  for (int m = 0; m < 4; ++m)
#pragma unroll
    for (int n = 0; n < 4; ++n) acc[m][n] = (f32x4){0.f, 0.f, 0.f, 0.f};
  bf16x8 At[4][2], Bt[4][2];
  STAGE(0, 0); STAGE(1, 1);
  WAIT_V(6); BAR; MEMF;
  if (grp == 1) BAR;
#define KT(st, stn, t) do { LDB(st); LDA(st); { const int tn = ((t) + 2 < nt) ? (t) + 2 : 0; STAGE(stn, tn); } WAIT_V(6); WAIT_L(0); BAR; SCHED; MMA(); BAR; MEMF; SCHED; } while (0)
  int t = 0;
  for (; t + 3 <= nt; t += 3) { KT(0, 2, t); KT(1, 0, t + 1); KT(2, 1, t + 2); }
  KT(0, 2, t); KT(1, 0, t + 1);
  {
    u16* o = peers.out[dst] + (size_t)(rank * M + pm * 256 + grp * 128 + wr * 64 + fr) * NO + pnl * 128 + wc * 64 + (fq & 1) * 16 + (fq >> 1) * 8;
#pragma unroll
    for (int m = 0; m < 4; ++m)
#pragma unroll
      for (int bj = 0; bj < 2; ++bj) {
        const f32x4 x = acc[m][2 * bj], y = acc[m][2 * bj + 1];
        unsigned x0 = cvtpk(x[0], x[1]), x1 = cvtpk(x[2], x[3]), y0 = cvtpk(y[0], y[1]), y1 = cvtpk(y[2], y[3]);
        auto r0 = __builtin_amdgcn_permlane16_swap(x0, y0, false, false);
        auto r1 = __builtin_amdgcn_permlane16_swap(x1, y1, false, false);
        u32x4 w = {r0[0], r1[0], r0[1], r1[1]};
        *(u32x4*)(o + (size_t)(m * 16) * NO + bj * 32) = w;
      }
  }
  WAIT_V(0);
  if (grp == 0) BAR;
  __syncthreads();
  if (wid == 0) {
    const unsigned call = __builtin_amdgcn_readfirstlane(ticket) / 256u + 1u;
    if (lane == 0 && dst != rank) {
      __builtin_amdgcn_fence(__ATOMIC_RELEASE, "");
      asm volatile("s_waitcnt vmcnt(0)" ::: "memory");
      __hip_atomic_store((unsigned*)(peers.ws[dst] + WS_FLAGS) + rank * 32 + j, call, __ATOMIC_RELAXED, __HIP_MEMORY_SCOPE_SYSTEM);
    }
    if (dst == rank) {
      const unsigned src = (unsigned)lane;
      unsigned* tmo = (unsigned*)(myws + WS_TMO);
      bool bad = false;
      if (src < (unsigned)NR && src != (unsigned)rank) {
        const unsigned* flag = (const unsigned*)(myws + WS_FLAGS) + src * 32 + j;
        unsigned long long t0 = __builtin_amdgcn_s_memrealtime();
        while (__hip_atomic_load(flag, __ATOMIC_RELAXED, __HIP_MEMORY_SCOPE_SYSTEM) < call) {
          if (__hip_atomic_load(tmo, __ATOMIC_RELAXED, __HIP_MEMORY_SCOPE_AGENT) == call || __builtin_amdgcn_s_memrealtime() - t0 > DEADLINE_TICKS) {
            __hip_atomic_store(tmo, call, __ATOMIC_RELAXED, __HIP_MEMORY_SCOPE_AGENT); bad = true; break;
          }
          __builtin_amdgcn_s_sleep(2);
        }
      }
      __builtin_amdgcn_fence(__ATOMIC_ACQUIRE, "");
      if (bad) peers.out[rank][(size_t)(rank * M + pm * 256) * NO + pnl * 128] = (u16)0x7fc0;
    }
  }
}

extern "C" void kernel_launch(int rank, int nranks, ncclComm_t comm, hipStream_t stream,
                              void* const* d_in, const int* in_sizes, int n_in,
                              void* d_out, int out_size, const int* in_dtypes, int out_dtype,
                              void* const* d_out_all, void* const* d_ws_all, size_t ws_bytes) {
  if (nranks != NR || (unsigned)rank >= (unsigned)NR) return;
  static bool init = false;
  if (!init) { HIP_CHECK(hipFuncSetAttribute((const void*)gemm_a2a, hipFuncAttributeMaxDynamicSharedMemorySize, 147456)); init = true; }
  Peers p;
  for (int i = 0; i < NR; ++i) { p.out[i] = (u16*)d_out_all[i]; p.ws[i] = (char*)d_ws_all[i]; }
  gemm_a2a<<<256, 512, 147456, stream>>>((const u16*)d_in[0], (const u16*)d_in[1], p, rank);
  HIP_CHECK(hipGetLastError());
}
```

```cpp
#include <hip/hip_runtime.h>
#include <rccl/rccl.h>
#include <cstdio>

#define HIP_CHECK(expr) do { hipError_t _e = (expr); if (_e != hipSuccess) { \
    fprintf(stderr, "HIP error %s at %s:%d\n", hipGetErrorString(_e), __FILE__, __LINE__); \
    return; } } while (0)

#define LAS __attribute__((address_space(3)))
typedef unsigned short u16;
typedef short bf16x8 __attribute__((ext_vector_type(8)));
typedef short s16x4 __attribute__((ext_vector_type(4)));
typedef float f32x4 __attribute__((ext_vector_type(4)));
typedef float f32x2 __attribute__((ext_vector_type(2)));
typedef unsigned u32x4 __attribute__((ext_vector_type(4)));
typedef __bf16 bf16x2_t __attribute__((ext_vector_type(2)));
constexpr int M = 1024, N = 8192, K = 8192, NR = 8, NO = 1024;
constexpr unsigned long long DEADLINE_TICKS = 100000000ull;
constexpr unsigned WS_TMO = 0;
constexpr unsigned WS_GCNT = 1024;
constexpr unsigned WS_CNT = 4096;
constexpr unsigned WS_DONE = 16384;
constexpr unsigned WS_PFLAG = 65536;
constexpr unsigned WS_CTRL_BYTES = 131072;
constexpr unsigned WS_PART = 1u << 18;
constexpr unsigned UNIT_B = 131072, ROUND_B = 256u * UNIT_B, PART_BYTES = 2u * ROUND_B;
static_assert(PART_BYTES == (1u << 26), "");
constexpr size_t WS_NEED = (size_t)WS_PART + PART_BYTES;
constexpr unsigned OUT_BLK = (unsigned)M * NO * 2;
constexpr int HTB = 16384, LDS_BYTES = 8 * HTB;
constexpr int kA = 128, kB = 64 * N * 2, hA = 128 * K * 2, hB = 256;
#define SLP 4
#define GBT 3000
#define SLP2 8
#define NOGB
#define NSTAG ((wid + j) & 7)
#define W2 5
#define G0OFF 4

__device__ __forceinline__ unsigned cvtpk(float lo, float hi) { f32x2 v = {lo, hi}; bf16x2_t b = __builtin_convertvector(v, bf16x2_t); return __builtin_bit_cast(unsigned, b); }
__device__ __forceinline__ s16x4 trrd(LAS const char* p) { return __builtin_amdgcn_ds_read_tr16_b64_v4i16((LAS s16x4*)p); }
__device__ __forceinline__ unsigned rfl(unsigned x) { return __builtin_amdgcn_readfirstlane(x); }
#define WAIT_V(n) asm volatile("s_waitcnt vmcnt(%0)" :: "n"(n) : "memory")
#define WAIT_L(n) asm volatile("s_waitcnt lgkmcnt(%0)" :: "n"(n) : "memory")
#define WAIT_V2(c, n1, n0) asm volatile("s_cmp_lg_u32 %0, 0\n\ts_waitcnt vmcnt(%1)\n\ts_cbranch_scc1 1f\n\ts_waitcnt vmcnt(%2)\n1:" :: "s"(c), "n"(n1), "n"(n0) : "memory", "scc")
#define WAIT_V3(c2, n2, c1, n1, n0) asm volatile("s_cmp_lg_u32 %0, 0\n\ts_waitcnt vmcnt(%1)\n\ts_cbranch_scc1 1f\n\ts_cmp_lg_u32 %2, 0\n\ts_waitcnt vmcnt(%3)\n\ts_cbranch_scc1 1f\n\ts_waitcnt vmcnt(%4)\n1:" \
    :: "s"(c2), "n"(n2), "s"(c1), "n"(n1), "n"(n0) : "memory", "scc")
#define WAIT_V4(lv, n3, n2, n1, n0) asm volatile("s_cmp_eq_u32 %0, 3\n\ts_waitcnt vmcnt(%1)\n\ts_cbranch_scc1 1f\n\ts_cmp_eq_u32 %0, 2\n\ts_waitcnt vmcnt(%2)\n\ts_cbranch_scc1 1f\n\ts_cmp_eq_u32 %0, 1\n\ts_waitcnt vmcnt(%3)\n\ts_cbranch_scc1 1f\n\ts_waitcnt vmcnt(%4)\n1:" \
    :: "s"(lv), "n"(n3), "n"(n2), "n"(n1), "n"(n0) : "memory", "scc")
#define BAR __builtin_amdgcn_s_barrier()
#define SCHED __builtin_amdgcn_sched_barrier(0)
#define MEMF asm volatile("" ::: "memory")
#define BLDS(srd, voff, soff, GOFF, ldsbase, LOFF) do { unsigned tmp_; asm volatile("s_add_u32 m0, %1, %2\n\ts_add_u32 %0, %3, %4\n\tbuffer_load_dwordx4 %5, %6, %0 offen lds" \
   : "=&s"(tmp_) : "s"(ldsbase), "i"(LOFF), "s"(soff), "i"(GOFF), "v"(voff), "s"(srd) : "memory", "scc"); } while (0)
#define BLD4(dst, srd, voff, soff, GOFF) do { unsigned tmp_; asm volatile("s_add_u32 %1, %2, %3\n\tbuffer_load_dwordx4 %0, %4, %5, %1 offen sc1" \
   : "=v"(dst), "=&s"(tmp_) : "s"(soff), "i"(GOFF), "v"(voff), "s"(srd) : "memory", "scc"); } while (0)
#define BLD1(dst, srd, voff, soff) asm volatile("buffer_load_dword %0, %1, %2, %3 offen sc1" : "=v"(dst) : "v"(voff), "s"(srd), "s"(soff) : "memory")
#define BST4(MOD, data, srd, voff, soff, GOFF) do { unsigned tmp_; asm volatile("s_add_u32 %0, %1, %2\n\tbuffer_store_dwordx4 %3, %4, %5, %0 offen " MOD "\n\ts_nop 1" \
   : "=&s"(tmp_) : "s"(soff), "i"(GOFF), "v"(data), "v"(voff), "s"(srd) : "memory", "scc"); } while (0)
#define BST1(MOD, data, srd, voff, soff) asm volatile("buffer_store_dword %0, %1, %2, %3 offen " MOD :: "v"(data), "v"(voff), "s"(srd), "s"(soff) : "memory")
#define CLD4(c, dst, srd, voff, soff) asm volatile("s_cmp_lg_u32 %1, 0\n\ts_cbranch_scc0 1f\n\tbuffer_load_dwordx4 %0, %2, %3, %4 offen sc1\n1:" : "+v"(dst) : "s"(c), "v"(voff), "s"(srd), "s"(soff) : "memory", "scc")
#define CST4(c, data, srd, voff, soff) asm volatile("s_cmp_lg_u32 %0, 0\n\ts_cbranch_scc0 1f\n\tbuffer_store_dwordx4 %1, %2, %3, %4 offen sc0 sc1\n\ts_nop 1\n1:" :: "s"(c), "v"(data), "v"(voff), "s"(srd), "s"(soff) : "memory", "scc")
#define WAIT_X(n) asm volatile("s_cmp_eq_u32 %0, 4\n\ts_waitcnt vmcnt(12)\n\ts_cbranch_scc1 1f\n\ts_cmp_eq_u32 %0, 2\n\ts_waitcnt vmcnt(10)\n\ts_cbranch_scc1 1f\n\ts_waitcnt vmcnt(8)\n1:" :: "s"(n) : "memory", "scc")
#define WAIT_Y(n) asm volatile("s_cmp_eq_u32 %0, 2\n\ts_waitcnt vmcnt(10)\n\ts_cbranch_scc1 1f\n\ts_waitcnt vmcnt(8)\n1:" :: "s"(n) : "memory", "scc")
__device__ __forceinline__ u32x4 make_srd(const void* p, unsigned bytes) {
  unsigned long long a = (unsigned long long)p;
  u32x4 r; r[0] = rfl((unsigned)a); r[1] = rfl((unsigned)(a >> 32) & 0xffffu); r[2] = bytes; r[3] = 0x00020000u; return r;
}
struct Peers { u16* out[NR]; char* ws[NR]; };
struct Unit { unsigned a, b, obase, olo, ohi; };

__device__ __forceinline__ Unit unit_of(int r, int x, int j, int rank, const Peers& peers) {
  const int h = x >> 2; int d, pnl, pm;
  if (r == 0) {
    if (j < 28) { d = 1 + (j >> 2); pnl = h; pm = j & 3; }
    else { const int e = h * 4 + j - 28; d = e < 7 ? 1 + e : 7; pm = e < 7 ? 0 : 1; pnl = 2; }
  } else if (h == 0) {
    if (j < 28) { d = 1 + (j >> 2); pnl = 3; pm = j & 3; } else { d = 0; pnl = 0; pm = j - 28; }
  } else {
    if (j < 18) { d = 1 + j / 3; pm = 1 + j % 3; pnl = 2; } else if (j < 20) { d = 7; pm = j - 16; pnl = 2; } else { d = 0; pnl = 1 + ((j - 20) >> 2); pm = j & 3; }
  }
  const unsigned dst = (unsigned)((rank + d) & (NR - 1));
  const unsigned long long o = (unsigned long long)peers.out[dst] + (unsigned long long)rank * OUT_BLK;
  Unit u; u.a = (unsigned)(pm * 256 * K * 2 + (x & 3) * 32 * kA); u.b = (unsigned)(((int)dst * 4 + pnl) * 512 + (x & 3) * 32 * kB); u.obase = (unsigned)(pm * 256 * NO * 2 + pnl * 512);
  u.olo = rfl((unsigned)o); u.ohi = rfl((unsigned)(o >> 32) & 0xffffu);
  return u;
}
__device__ __forceinline__ unsigned piece_off(unsigned pc) { return (pc >> 3) * (128u * NO * 2) + (pc & 3u) * (16u * NO * 2) + ((pc >> 2) & 1u) * 256u; }
__device__ __forceinline__ float blo(unsigned w) { return __uint_as_float(w << 16); }
__device__ __forceinline__ float bhi(unsigned w) { return __uint_as_float(w & 0xffff0000u); }
__device__ __forceinline__ f32x4 zero4() { f32x2 a, b; asm volatile("v_mov_b64 %0, 0" : "=v"(a)); asm volatile("v_mov_b64 %0, 0" : "=v"(b)); return __builtin_shufflevector(a, b, 0, 1, 2, 3); }
__device__ __forceinline__ float fand(float v, unsigned m) { return __uint_as_float(__float_as_uint(v) & m); }

__device__ __forceinline__ float fadd1(float a, float b) { float r; asm("v_add_f32_e32 %0, %1, %2" : "=v"(r) : "v"(a), "v"(b)); return r; }
#define STAMP(i) do {} while (0)
#define STAMPV(i, v) do {} while (0)
__global__ __launch_bounds__(512, 2) void gemm_a2a(const u16* __restrict__ A, const u16* __restrict__ W, Peers peers, int rank) {
  extern __shared__ __attribute__((aligned(16))) char smem_[];
  LAS char* lds = (LAS char*)smem_;
  const int tid = threadIdx.x, wid = __builtin_amdgcn_readfirstlane(tid >> 6), lane = tid & 63, wr = wid >> 2, wc = wid & 3, fr = lane & 15, fq = lane >> 4;
  const int b = blockIdx.x, x = b & 7, j = b >> 3;
  char* myws = peers.ws[rank];
  unsigned tsv = 0; STAMP(0);
  const u32x4 srdA = make_srd(A, (unsigned)M * K * 2), srdB = make_srd(W, (unsigned)K * N * 2), srdC = make_srd(myws, WS_CTRL_BYTES), srdP = make_srd(myws, (unsigned)WS_NEED);
  const unsigned vl16 = (unsigned)lane * 16u, vl4 = (unsigned)lane * 4u;
  unsigned cntv; { const unsigned so = WS_CNT + (unsigned)(b * 8 + wid) * 4u; BLD1(cntv, srdC, vl4, so); }
  unsigned voffA, voffB;
  { const int row = wid * 8 + (lane >> 3); const int ch = (lane & 7) ^ ((row >> 1) & 7); voffA = (unsigned)(row * K + ch * 8) * 2u; }
  { const int P = wid * 8 + (lane >> 3); const int k = ((P >> 5) << 4) | ((P & 1) << 3) | ((P >> 2) & 7), n64 = (P >> 1) & 1;
    const int sg = ((lane & 7) >> 1) ^ (k & 3), ch = sg * 2 + (lane & 1); voffB = (unsigned)(k * N + n64 * 64 + ch * 8) * 2u; }
  const unsigned ldsw = rfl((unsigned)(size_t)lds + (unsigned)wid * 1024u);
  const int aoff0 = (wr * 64 + fr) * 128 + ((fq ^ (fr >> 1)) << 4), aoff1 = aoff0 ^ 64;
  const int q = fr >> 2, p = fr & 3;
  const int brest = (fq >> 1) * 4096 + q * 512 + (fq & 1) * 128 + 8 * p + (wc >> 1) * 256;
  const int boff0 = brest + ((((wc & 1) * 2 + 0) ^ q) << 5), boff1 = brest + ((((wc & 1) * 2 + 1) ^ q) << 5);
  const unsigned voffO = (unsigned)((wr * 64 + fr) * NO * 2 + (wc * 32 + (fq & 1) * 16 + (fq >> 1) * 8) * 2);
  const Unit u0 = unit_of(0, x, j, rank, peers);
#define SA(b_, h) (((b_) * 2 + (h)) * HTB)
#define SB(b_, h) ((4 + (b_) * 2 + (h)) * HTB)
#define STAGE_A(bufoff, so, GO) do { BLDS(srdA, voffA, so, (GO), ldsw, (bufoff)); BLDS(srdA, voffA, so, (GO) + 64 * K * 2, ldsw, (bufoff) + 8192); } while (0)
#define STAGE_B(bufoff, so, GO) do { BLDS(srdB, voffB, so, (GO), ldsw, (bufoff)); BLDS(srdB, voffB, so, (GO) + 32 * N * 2, ldsw, (bufoff) + 8192); } while (0)
#define LDA(dst_, b_, h) do { _Pragma("unroll") for (int m = 0; m < 4; ++m) { dst_[m][0] = *(const LAS bf16x8*)(lds + SA(b_, h) + aoff0 + m * 2048); dst_[m][1] = *(const LAS bf16x8*)(lds + SA(b_, h) + aoff1 + m * 2048); } } while (0)
#define LDB1(dst_, b_, h, boff, s) do { s16x4 lo = trrd(lds + SB(b_, h) + boff + (s) * 8192), hi = trrd(lds + SB(b_, h) + boff + (s) * 8192 + 2048); dst_ = __builtin_shufflevector(lo, hi, 0, 1, 2, 3, 4, 5, 6, 7); } while (0)
#define LDB(dst_, b_, h) do { LDB1(dst_[0][0], b_, h, boff0, 0); LDB1(dst_[0][1], b_, h, boff0, 1); LDB1(dst_[1][0], b_, h, boff1, 0); LDB1(dst_[1][1], b_, h, boff1, 1); } while (0)
#define MMA(ai, bj, At, Bt) do { _Pragma("unroll") for (int k = 0; k < 2; ++k) _Pragma("unroll") for (int m = 0; m < 4; ++m) _Pragma("unroll") for (int n = 0; n < 2; ++n) \
    acc[ai][bj][m][n] = __builtin_amdgcn_mfma_f32_16x16x32_bf16(Bt[n][k], At[m][k], acc[ai][bj][m][n], 0, 0, 0); } while (0)
#define PRIO(n) __builtin_amdgcn_s_setprio(n)
#define PACK(w, ai, bj, m) do { const f32x4 x_ = acc[ai][bj][m][0], y_ = acc[ai][bj][m][1]; \
    auto r0_ = __builtin_amdgcn_permlane16_swap(cvtpk(x_[0], x_[1]), cvtpk(y_[0], y_[1]), false, false); \
    auto r1_ = __builtin_amdgcn_permlane16_swap(cvtpk(x_[2], x_[3]), cvtpk(y_[2], y_[3]), false, false); \
    w = (u32x4){r0_[0], r1_[0], r0_[1], r1_[1]}; } while (0)
#define E1(eb, ai, bj, m) do { u32x4 w_; PACK(w_, ai, bj, m); BST4("sc1", w_, srdP, vl16, eb, ((ai) * 8 + (bj) * 4 + (m)) * 8192); \
    acc[ai][bj][m][0] = zero4(); acc[ai][bj][m][1] = zero4(); } while (0)
#define E4(eb, ai, bj) do { E1(eb, ai, bj, 0); E1(eb, ai, bj, 1); E1(eb, ai, bj, 2); E1(eb, ai, bj, 3); } while (0)
#define E16(eb) do { E4(eb, 0, 0); E4(eb, 0, 1); E4(eb, 1, 0); E4(eb, 1, 1); } while (0)
  bf16x8 At[4][2], B0[2][2], B1[2][2];
  unsigned cA = u0.a, cB = u0.b;
  STAGE_B(SB(0, 0), cB, 0); STAGE_B(SB(0, 1), cB, hB); STAGE_A(SA(0, 0), cA, 0); STAGE_A(SA(0, 1), cA, hA);
  const Unit u1 = unit_of(1, x, j, rank, peers);
  f32x4 acc[2][2][4][2];
#pragma unroll
  for (int a = 0; a < 2; ++a)
#pragma unroll
    for (int c = 0; c < 2; ++c)
#pragma unroll
      for (int m = 0; m < 4; ++m)
#pragma unroll
        for (int n = 0; n < 2; ++n) acc[a][c][m][n] = zero4();
  if (wr == 1) BAR;
  WAIT_V(2); BAR; MEMF;
  STAGE_B(SB(1, 0), cB, kB); STAGE_A(SA(1, 0), cA, kA); STAGE_B(SB(1, 1), cB, hB + kB);
  asm volatile("" : "+v"(cntv));
  const unsigned call = rfl(cntv) + 1u;
  const unsigned vcall = call;
  WAIT_V(6); BAR; MEMF;
  STAMP(1);
  const unsigned sp = (unsigned)x & 3u, T = ((unsigned)x >> 2) * 32u + (unsigned)j;
  const unsigned pb = WS_PART + T * 4u * UNIT_B + (unsigned)wid * 1024u;
  const unsigned fb = WS_PFLAG + (T * 8u + (unsigned)wid) * 32u;
  const unsigned n1 = (unsigned)NSTAG;
  float rs[8]; u32x4 Xa, Xb, Ya, Yb, ow; unsigned fv;
  unsigned* tmo = (unsigned*)(myws + WS_TMO);
  unsigned bad = 0;
  unsigned nA = u1.a, nB = u1.b;
  u32x4 srdO = {u0.olo, u0.ohi, 0u, 0x00020000u}, srdZ = srdC; srdZ[2] = 0;
  unsigned lo = pb + sp * 4u * 8192u, oo = u0.obase + piece_off(sp * 4u) - 16u * NO * 2u;
#define LDA2(dst_, b_, h, m0) do { _Pragma("unroll") for (int m = m0; m < m0 + 2; ++m) { dst_[m][0] = *(const LAS bf16x8*)(lds + SA(b_, h) + aoff0 + m * 2048); dst_[m][1] = *(const LAS bf16x8*)(lds + SA(b_, h) + aoff1 + m * 2048); } } while (0)
#define MF(i) acc[1][(i) >> 4][((i) >> 1) & 3][(i) & 1] = __builtin_amdgcn_mfma_f32_16x16x32_bf16(((i) >> 4) ? B1[(i) & 1][((i) >> 3) & 1] : B0[(i) & 1][((i) >> 3) & 1], At[((i) >> 1) & 3][((i) >> 3) & 1], acc[1][(i) >> 4][((i) >> 1) & 3][(i) & 1], 0, 0, 0)
#define UNP(L, e) (((e) & 1) ? bhi(L[(e) >> 1]) : blo(L[(e) >> 1]))
#define SP1LOAD(b_, abuf, aso, XA, XB) do { LDB(B0, b_, 0); BLDS(srdA, voffA, aso, hA, ldsw, abuf); LDB(B1, b_, 1); BLDS(srdA, voffA, aso, hA + 64 * K * 2, ldsw, (abuf) + 8192); \
    LDA2(At, b_, 0, 0); XA; LDA2(At, b_, 0, 2); XB; } while (0)
#define TRIP(KIND, t) do { \
    const bool last = (t) == 30u; \
    const unsigned a1 = cA + ((t) + 1u) * kA; \
    const unsigned a2 = last ? nA : cA + ((t) + 2u) * kA, b2 = last ? nB : cB + ((t) + 2u) * kB; \
    if ((KIND) == 3) SP1LOAD(0, SA(1, 1), a1, BLD4(Xa, srdP, vl16, lo, 0), BLD4(Xb, srdP, vl16, lo, UNIT_B)); \
    else if ((KIND) == 2) SP1LOAD(0, SA(1, 1), a1, BLD1(fv, srdC, vl4, fb), (void)0); \
    else SP1LOAD(0, SA(1, 1), a1, (void)0, (void)0); \
    WAIT_V((KIND) == 1 ? 24 : (KIND) == 2 ? 9 : (KIND) == 3 ? 13 : (KIND) == 4 ? 11 : 8); \
    WAIT_L(0); BAR; PRIO(1); MMA(0, 0, At, B0); MMA(0, 1, At, B1); PRIO(0); BAR; MEMF; SCHED; \
    LDA(At, 0, 1); STAGE_B(SB(0, 0), b2, 0); STAGE_B(SB(0, 1), b2, hB); STAGE_A(SA(0, 0), a2, 0); \
    WAIT_V((KIND) == 1 ? 24 : (KIND) == 2 ? 9 : (KIND) == 3 ? 10 : 8); \
    if ((KIND) >= 3) asm volatile("" : "+v"(Ya), "+v"(Yb)); \
    WAIT_L(0); BAR; SCHED; PRIO(1); \
    if ((KIND) >= 3) {   \
      float tp = 0.f; \
      _Pragma("unroll") for (int e = 0; e < 8; ++e) { float q0, q1; \
        MF(4 * e); q0 = UNP(Ya, e); SCHED; MF(4 * e + 1); q1 = UNP(Yb, e); SCHED; MF(4 * e + 2); q0 = fadd1(q0, q1); SCHED; \
        MF(4 * e + 3); q0 = fadd1(rs[e], q0); if (e & 1) ow[e >> 1] = cvtpk(tp, q0); tp = q0; SCHED; } \
    } else { MMA(1, 0, At, B0); MMA(1, 1, At, B1); } \
    PRIO(0); BAR; MEMF; SCHED; \
    if ((KIND) == 3) { SP1LOAD(1, SA(0, 1), a2, BLD4(Ya, srdP, vl16, lo, 2 * UNIT_B), BLD4(Yb, srdP, vl16, lo, 3 * UNIT_B)); BST4("sc0 sc1", ow, srdO, voffO, oo, 0); } \
    else if ((KIND) == 4) SP1LOAD(1, SA(0, 1), a2, (void)0, BST4("sc0 sc1", ow, srdO, voffO, oo, 0)); \
    else if ((KIND) == 2) { SP1LOAD(1, SA(0, 1), a2, BST1("", vcall, srdZ, vl4, fb), BST1("", vcall, srdZ, vl4, fb)); BST1("", vcall, srdZ, vl4, fb); }     \
    else SP1LOAD(1, SA(0, 1), a2, (void)0, (void)0); \
    WAIT_V((KIND) == 2 ? 12 : (KIND) == 3 ? 13 : (KIND) == 4 ? 9 : 8); \
    if ((KIND) == 1) { const unsigned pubo = fb + sp * 4u; u32x4 sc = srdC; sc[2] = pubo + 4u; BST1("sc1", vcall, sc, vl4, pubo); }     \
    WAIT_L(0); BAR; PRIO(1); MMA(0, 0, At, B0); MMA(0, 1, At, B1); PRIO(0); BAR; MEMF; SCHED; \
    LDA(At, 1, 1); STAGE_B(SB(1, 0), b2, kB); STAGE_B(SB(1, 1), b2, kB + hB); STAGE_A(SA(1, 0), a2, kA); \
    WAIT_V((KIND) == 1 || (KIND) == 4 ? 9 : (KIND) == 2 || (KIND) == 3 ? 11 : 8); \
    if ((KIND) == 3) asm volatile("" : "+v"(Xa), "+v"(Xb)); \
    if ((KIND) == 2) asm volatile("" : "+v"(fv)); \
    WAIT_L(0); BAR; SCHED; PRIO(1); \
    if ((KIND) == 3) {   \
      _Pragma("unroll") for (int e = 0; e < 8; ++e) { float q0, q1; \
        MF(4 * e); q0 = UNP(Xa, e); SCHED; MF(4 * e + 1); q1 = UNP(Xb, e); SCHED; MF(4 * e + 2); rs[e] = fadd1(q0, q1); SCHED; MF(4 * e + 3); SCHED; } \
      lo += 8192u; oo += 16u * NO * 2u; srdO[2] = OUT_BLK; \
    } else { MMA(1, 0, At, B0); MMA(1, 1, At, B1); } \
    PRIO(0); BAR; MEMF; SCHED; \
    STAMP(16 + ((t) >> 2) + ((KIND) ? 8 : sbase)); \
  } while (0)
  auto wait_flags = [&](unsigned f) {
    const unsigned long long t0 = __builtin_amdgcn_s_memrealtime();
    for (;;) {
      unsigned v; BLD1(v, srdC, vl4, fb); WAIT_V(0); asm volatile("" : "+v"(v));
      if (((__builtin_amdgcn_ballot_w64(v >= call) >> (4u * f)) & 15ull) == 15ull) break;
      if (rfl(__hip_atomic_load(tmo, __ATOMIC_RELAXED, __HIP_MEMORY_SCOPE_AGENT)) == call || __builtin_amdgcn_s_memrealtime() - t0 > DEADLINE_TICKS) {
        __hip_atomic_store(tmo, call, __ATOMIC_RELAXED, __HIP_MEMORY_SCOPE_AGENT); bad = 1; break; }
      __builtin_amdgcn_s_sleep(SLP);
    }
    bad = rfl(bad);
  };
#define SUM4(ow_, R) do { asm volatile("" : "+v"(R[0]), "+v"(R[1]), "+v"(R[2]), "+v"(R[3])); _Pragma("unroll") for (int e = 0; e < 4; ++e) { \
        const float lo_ = (blo(R[0][e]) + blo(R[1][e])) + (blo(R[2][e]) + blo(R[3][e])); \
        const float hi_ = (bhi(R[0][e]) + bhi(R[1][e])) + (bhi(R[2][e]) + bhi(R[3][e])); \
        ow_[e] = bad ? 0x7fc07fc0u : cvtpk(lo_, hi_); } } while (0)
#define LD4(R, l, OFF) do { BLD4(R[0], srdP, vl16, l, (OFF) + 0 * UNIT_B); BLD4(R[1], srdP, vl16, l, (OFF) + 1 * UNIT_B); BLD4(R[2], srdP, vl16, l, (OFF) + 2 * UNIT_B); BLD4(R[3], srdP, vl16, l, (OFF) + 3 * UNIT_B); } while (0)
  unsigned t = 0, sbase = 0;
#pragma unroll 1
  for (; t < 32u; t += 2) TRIP(0, t);
  STAMP(2);
  { const unsigned eb = pb + sp * UNIT_B; E16(eb); }
  STAMP(3);
  cA = nA; cB = nB; sbase = 8;
  TRIP(1, 0u);
#pragma unroll 1
  for (t = 2; t < 2u + 2u * n1; t += 2) TRIP(0, t);
  STAMP(4);
  TRIP(2, t); t += 2;
  if ((__builtin_amdgcn_ballot_w64(fv >= call) & 15ull) != 15ull) { wait_flags(0u); BST1("", vcall, srdZ, vl4, fb); BST1("", vcall, srdZ, vl4, fb); BST1("", vcall, srdZ, vl4, fb); }
  STAMP(5);
  asm volatile("" : "=v"(Ya), "=v"(Yb), "=v"(rs[0]), "=v"(rs[1]), "=v"(rs[2]), "=v"(rs[3]), "=v"(rs[4]), "=v"(rs[5]), "=v"(rs[6]), "=v"(rs[7]));
#pragma unroll 1
  for (int i = 0; i < 4; ++i) { TRIP(3, t); t += 2; }
  TRIP(4, t); t += 2;
  STAMP(6);
#pragma unroll 1
  for (; t < 32u; t += 2) TRIP(0, t);
  STAMP(8);
  { const unsigned eb = pb + ROUND_B + sp * UNIT_B; E16(eb); }
  if (wr == 0) BAR;
  STAMP(9);
  WAIT_V(0);
  STAMP(10);
  { u32x4 sc = srdC; const unsigned pubo = fb + 16u + sp * 4u; sc[2] = pubo + 4u; BST1("sc1", vcall, sc, vl4, pubo); }
  { u32x4 sc = srdC; const unsigned so = WS_CNT + (unsigned)(b * 8 + wid) * 4u; sc[2] = so + 4u; BST1("sc1", vcall, sc, vl4, so); }
  STAMP(11);
  wait_flags(1u);
  STAMP(12);
  {
    const unsigned l = pb + ROUND_B + sp * 4u * 8192u, o = u1.obase + piece_off(sp * 4u);
    u32x4 R0[4], R1[4], R2[4], R3[4]; LD4(R0, l, 0); LD4(R1, l, 8192); LD4(R2, l, 16384); LD4(R3, l, 24576);
    const u32x4 so = {u1.olo, u1.ohi, OUT_BLK, 0x00020000u};
    WAIT_V(12); { u32x4 w; SUM4(w, R0); BST4("sc0 sc1", w, so, voffO, o, 0); }
    WAIT_V(9); { u32x4 w; SUM4(w, R1); BST4("sc0 sc1", w, so, voffO, o, 16 * NO * 2); }
    WAIT_V(6); { u32x4 w; SUM4(w, R2); BST4("sc0 sc1", w, so, voffO, o, 32 * NO * 2); }
    WAIT_V(3); { u32x4 w; SUM4(w, R3); BST4("sc0 sc1", w, so, voffO, o, 48 * NO * 2); }
  }
  STAMP(13);
  WAIT_V(0);
  STAMP(14);
  MEMF; BAR; MEMF;
  if (wid == 0) {
    const unsigned pr = (unsigned)lane;
    const bool mine = pr < (unsigned)NR && pr != (unsigned)rank;
    if (mine) __hip_atomic_store((unsigned*)(peers.ws[pr] + WS_DONE) + rank * 256 + b, call, __ATOMIC_RELAXED, __HIP_MEMORY_SCOPE_SYSTEM);
    bool late = false;
    if (mine) {
      const unsigned* flag = (const unsigned*)(myws + WS_DONE) + pr * 256 + b;
      const unsigned long long t0 = __builtin_amdgcn_s_memrealtime();
      while (__hip_atomic_load(flag, __ATOMIC_RELAXED, __HIP_MEMORY_SCOPE_SYSTEM) < call) {
        if (__hip_atomic_load(tmo, __ATOMIC_RELAXED, __HIP_MEMORY_SCOPE_AGENT) == call || __builtin_amdgcn_s_memrealtime() - t0 > DEADLINE_TICKS) {
          __hip_atomic_store(tmo, call, __ATOMIC_RELAXED, __HIP_MEMORY_SCOPE_AGENT); late = true; break; }
        __builtin_amdgcn_s_sleep(SLP);
      }
    }
    __builtin_amdgcn_fence(__ATOMIC_ACQUIRE, "");
    if (late || __hip_atomic_load(tmo, __ATOMIC_RELAXED, __HIP_MEMORY_SCOPE_AGENT) == call) peers.out[rank][(size_t)rank * M * NO + (size_t)b] = (u16)0x7fc0;
  }
}

extern "C" void kernel_launch(int rank, int nranks, ncclComm_t comm, hipStream_t stream,
                              void* const* d_in, const int* in_sizes, int n_in,
                              void* d_out, int out_size, const int* in_dtypes, int out_dtype,
                              void* const* d_out_all, void* const* d_ws_all, size_t ws_bytes) {
  if (nranks != NR || (unsigned)rank >= (unsigned)NR || ws_bytes < WS_NEED) return;
  HIP_CHECK(hipFuncSetAttribute((const void*)gemm_a2a, hipFuncAttributeMaxDynamicSharedMemorySize, LDS_BYTES));
  Peers p;
  for (int i = 0; i < NR; ++i) { p.out[i] = (u16*)d_out_all[i]; p.ws[i] = (char*)d_ws_all[i]; }
  gemm_a2a<<<256, 512, LDS_BYTES, stream>>>((const u16*)d_in[0], (const u16*)d_in[1], p, rank);
  HIP_CHECK(hipGetLastError());
}
```

```cpp
#include <hip/hip_runtime.h>
#include <rccl/rccl.h>
#include <cstdio>

#define HIP_CHECK(expr) do { hipError_t _e = (expr); if (_e != hipSuccess) { \
    fprintf(stderr, "HIP error %s at %s:%d\n", hipGetErrorString(_e), __FILE__, __LINE__); \
    return; } } while (0)

#define LAS __attribute__((address_space(3)))
typedef unsigned short u16;
typedef short bf16x8 __attribute__((ext_vector_type(8)));
typedef short s16x4 __attribute__((ext_vector_type(4)));
typedef float f32x4 __attribute__((ext_vector_type(4)));
typedef float f32x2 __attribute__((ext_vector_type(2)));
typedef unsigned u32x4 __attribute__((ext_vector_type(4)));
typedef __bf16 bf16x2_t __attribute__((ext_vector_type(2)));
constexpr int M = 1024, N = 8192, K = 8192, NR = 8, NO = 1024;
constexpr unsigned long long DEADLINE_TICKS = 100000000ull;
constexpr unsigned WS_TMO = 0;
constexpr unsigned WS_GCNT = 1024;
constexpr unsigned WS_CNT = 4096;
constexpr unsigned WS_DONE = 16384;
constexpr unsigned WS_PFLAG = 65536;
constexpr unsigned WS_CTRL_BYTES = 131072;
constexpr unsigned WS_PART = 1u << 18;
constexpr unsigned UNIT_B = 131072, ROUND_B = 256u * UNIT_B, PART_BYTES = 2u * ROUND_B;
static_assert(PART_BYTES == (1u << 26), "");
constexpr size_t WS_NEED = (size_t)WS_PART + PART_BYTES;
constexpr unsigned OUT_BLK = (unsigned)M * NO * 2;
constexpr int HTB = 16384, LDS_BYTES = 8 * HTB;
constexpr int kA = 128, kB = 64 * N * 2, hA = 128 * K * 2, hB = 256;
#define SLP 4
#define WPOL "nt"
#define LPOL "sc1"
#define EPOL "sc1"
#define OPOL "sc0 sc1"
#define NOGB
#define GBT 3000
#define SLP2 8
#define NSTAG ((wid + j) & 7)
#define W2 5
#define G0OFF 4

__device__ __forceinline__ unsigned cvtpk(float lo, float hi) { f32x2 v = {lo, hi}; bf16x2_t b = __builtin_convertvector(v, bf16x2_t); return __builtin_bit_cast(unsigned, b); }
__device__ __forceinline__ s16x4 trrd(LAS const char* p) { return __builtin_amdgcn_ds_read_tr16_b64_v4i16((LAS s16x4*)p); }
__device__ __forceinline__ unsigned rfl(unsigned x) { return __builtin_amdgcn_readfirstlane(x); }
#define WAIT_V(n) asm volatile("s_waitcnt vmcnt(%0)" :: "n"(n) : "memory")
#define WAIT_L(n) asm volatile("s_waitcnt lgkmcnt(%0)" :: "n"(n) : "memory")
#define WAIT_V2(c, n1, n0) asm volatile("s_cmp_lg_u32 %0, 0\n\ts_waitcnt vmcnt(%1)\n\ts_cbranch_scc1 1f\n\ts_waitcnt vmcnt(%2)\n1:" :: "s"(c), "n"(n1), "n"(n0) : "memory", "scc")
#define WAIT_V3(c2, n2, c1, n1, n0) asm volatile("s_cmp_lg_u32 %0, 0\n\ts_waitcnt vmcnt(%1)\n\ts_cbranch_scc1 1f\n\ts_cmp_lg_u32 %2, 0\n\ts_waitcnt vmcnt(%3)\n\ts_cbranch_scc1 1f\n\ts_waitcnt vmcnt(%4)\n1:" \
    :: "s"(c2), "n"(n2), "s"(c1), "n"(n1), "n"(n0) : "memory", "scc")
#define WAIT_V4(lv, n3, n2, n1, n0) asm volatile("s_cmp_eq_u32 %0, 3\n\ts_waitcnt vmcnt(%1)\n\ts_cbranch_scc1 1f\n\ts_cmp_eq_u32 %0, 2\n\ts_waitcnt vmcnt(%2)\n\ts_cbranch_scc1 1f\n\ts_cmp_eq_u32 %0, 1\n\ts_waitcnt vmcnt(%3)\n\ts_cbranch_scc1 1f\n\ts_waitcnt vmcnt(%4)\n1:" \
    :: "s"(lv), "n"(n3), "n"(n2), "n"(n1), "n"(n0) : "memory", "scc")
#define BAR __builtin_amdgcn_s_barrier()
#define SCHED __builtin_amdgcn_sched_barrier(0)
#define MEMF asm volatile("" ::: "memory")
#define BLDS(srd, voff, soff, GOFF, ldsbase, LOFF) do { unsigned tmp_; asm volatile("s_add_u32 m0, %1, %2\n\ts_add_u32 %0, %3, %4\n\tbuffer_load_dwordx4 %5, %6, %0 offen lds" \
   : "=&s"(tmp_) : "s"(ldsbase), "i"(LOFF), "s"(soff), "i"(GOFF), "v"(voff), "s"(srd) : "memory", "scc"); } while (0)
#define BLDSN(srd, voff, soff, GOFF, ldsbase, LOFF) do { unsigned tmp_; asm volatile("s_add_u32 m0, %1, %2\n\ts_add_u32 %0, %3, %4\n\tbuffer_load_dwordx4 %5, %6, %0 offen " WPOL " lds" \
   : "=&s"(tmp_) : "s"(ldsbase), "i"(LOFF), "s"(soff), "i"(GOFF), "v"(voff), "s"(srd) : "memory", "scc"); } while (0)
#define BLD4(dst, srd, voff, soff, GOFF) do { unsigned tmp_; asm volatile("s_add_u32 %1, %2, %3\n\tbuffer_load_dwordx4 %0, %4, %5, %1 offen " LPOL \
   : "=v"(dst), "=&s"(tmp_) : "s"(soff), "i"(GOFF), "v"(voff), "s"(srd) : "memory", "scc"); } while (0)
#define BLD1(dst, srd, voff, soff) asm volatile("buffer_load_dword %0, %1, %2, %3 offen sc1" : "=v"(dst) : "v"(voff), "s"(srd), "s"(soff) : "memory")
#define BST4(MOD, data, srd, voff, soff, GOFF) do { unsigned tmp_; asm volatile("s_add_u32 %0, %1, %2\n\tbuffer_store_dwordx4 %3, %4, %5, %0 offen " MOD "\n\ts_nop 1" \
   : "=&s"(tmp_) : "s"(soff), "i"(GOFF), "v"(data), "v"(voff), "s"(srd) : "memory", "scc"); } while (0)
#define BST1(MOD, data, srd, voff, soff) asm volatile("buffer_store_dword %0, %1, %2, %3 offen " MOD :: "v"(data), "v"(voff), "s"(srd), "s"(soff) : "memory")
#define CLD4(c, dst, srd, voff, soff) asm volatile("s_cmp_lg_u32 %1, 0\n\ts_cbranch_scc0 1f\n\tbuffer_load_dwordx4 %0, %2, %3, %4 offen sc1\n1:" : "+v"(dst) : "s"(c), "v"(voff), "s"(srd), "s"(soff) : "memory", "scc")
#define CST4(c, data, srd, voff, soff) asm volatile("s_cmp_lg_u32 %0, 0\n\ts_cbranch_scc0 1f\n\tbuffer_store_dwordx4 %1, %2, %3, %4 offen sc0 sc1\n\ts_nop 1\n1:" :: "s"(c), "v"(data), "v"(voff), "s"(srd), "s"(soff) : "memory", "scc")
#define WAIT_X(n) asm volatile("s_cmp_eq_u32 %0, 4\n\ts_waitcnt vmcnt(12)\n\ts_cbranch_scc1 1f\n\ts_cmp_eq_u32 %0, 2\n\ts_waitcnt vmcnt(10)\n\ts_cbranch_scc1 1f\n\ts_waitcnt vmcnt(8)\n1:" :: "s"(n) : "memory", "scc")
#define WAIT_Y(n) asm volatile("s_cmp_eq_u32 %0, 2\n\ts_waitcnt vmcnt(10)\n\ts_cbranch_scc1 1f\n\ts_waitcnt vmcnt(8)\n1:" :: "s"(n) : "memory", "scc")
__device__ __forceinline__ u32x4 make_srd(const void* p, unsigned bytes) {
  unsigned long long a = (unsigned long long)p;
  u32x4 r; r[0] = rfl((unsigned)a); r[1] = rfl((unsigned)(a >> 32) & 0xffffu); r[2] = bytes; r[3] = 0x00020000u; return r;
}
struct Peers { u16* out[NR]; char* ws[NR]; };
struct Unit { unsigned a, b, obase, olo, ohi; };

__device__ __forceinline__ Unit unit_of(int r, int x, int j, int rank, const Peers& peers) {
  const int h = x >> 2; int d, pnl, pm;
  if (r == 0) {
    if (j < 28) { d = 1 + (j >> 2); pnl = h; pm = j & 3; }
    else { const int e = h * 4 + j - 28; d = e < 7 ? 1 + e : 7; pm = e < 7 ? 0 : 1; pnl = 2; }
  } else if (h == 0) {
    if (j < 28) { d = 1 + (j >> 2); pnl = 3; pm = j & 3; } else { d = 0; pnl = 0; pm = j - 28; }
  } else {
    if (j < 18) { d = 1 + j / 3; pm = 1 + j % 3; pnl = 2; } else if (j < 20) { d = 7; pm = j - 16; pnl = 2; } else { d = 0; pnl = 1 + ((j - 20) >> 2); pm = j & 3; }
  }
  const unsigned dst = (unsigned)((rank + d) & (NR - 1));
  const unsigned long long o = (unsigned long long)peers.out[dst] + (unsigned long long)rank * OUT_BLK;
  Unit u; u.a = (unsigned)(pm * 256 * K * 2 + (x & 3) * 32 * kA); u.b = (unsigned)(((int)dst * 4 + pnl) * 512 + (x & 3) * 32 * kB); u.obase = (unsigned)(pm * 256 * NO * 2 + pnl * 512);
  u.olo = rfl((unsigned)o); u.ohi = rfl((unsigned)(o >> 32) & 0xffffu);
  return u;
}
__device__ __forceinline__ unsigned piece_off(unsigned pc) { return (pc >> 3) * (128u * NO * 2) + (pc & 3u) * (16u * NO * 2) + ((pc >> 2) & 1u) * 256u; }
__device__ __forceinline__ float blo(unsigned w) { return __uint_as_float(w << 16); }
__device__ __forceinline__ float bhi(unsigned w) { return __uint_as_float(w & 0xffff0000u); }
__device__ __forceinline__ f32x4 zero4() { f32x2 a, b; asm volatile("v_mov_b64 %0, 0" : "=v"(a)); asm volatile("v_mov_b64 %0, 0" : "=v"(b)); return __builtin_shufflevector(a, b, 0, 1, 2, 3); }
__device__ __forceinline__ float fand(float v, unsigned m) { return __uint_as_float(__float_as_uint(v) & m); }

__device__ __forceinline__ float fadd1(float a, float b) { float r; asm("v_add_f32_e32 %0, %1, %2" : "=v"(r) : "v"(a), "v"(b)); return r; }
#define STAMP(i) do {} while (0)
#define STAMPV(i, v) do {} while (0)
__global__ __launch_bounds__(512, 2) void gemm_a2a(const u16* __restrict__ A, const u16* __restrict__ W, Peers peers, int rank) {
  extern __shared__ __attribute__((aligned(16))) char smem_[];
  LAS char* lds = (LAS char*)smem_;
  const int tid = threadIdx.x, wid = __builtin_amdgcn_readfirstlane(tid >> 6), lane = tid & 63, wr = wid >> 2, wc = wid & 3, fr = lane & 15, fq = lane >> 4;
  const int b = blockIdx.x, x = b & 7, j = b >> 3;
  char* myws = peers.ws[rank];
  unsigned tsv = 0; STAMP(0);
  const u32x4 srdA = make_srd(A, (unsigned)M * K * 2), srdB = make_srd(W, (unsigned)K * N * 2), srdC = make_srd(myws, WS_CTRL_BYTES), srdP = make_srd(myws, (unsigned)WS_NEED);
  const unsigned vl16 = (unsigned)lane * 16u, vl4 = (unsigned)lane * 4u;
  unsigned cntv; { const unsigned so = WS_CNT + (unsigned)(b * 8 + wid) * 4u; BLD1(cntv, srdC, vl4, so); }
  unsigned voffA, voffB;
  { const int row = wid * 8 + (lane >> 3); const int ch = (lane & 7) ^ ((row >> 1) & 7); voffA = (unsigned)(row * K + ch * 8) * 2u; }
  { const int P = wid * 8 + (lane >> 3); const int k = ((P >> 5) << 4) | ((P & 1) << 3) | ((P >> 2) & 7), n64 = (P >> 1) & 1;
    const int sg = ((lane & 7) >> 1) ^ (k & 3), ch = sg * 2 + (lane & 1); voffB = (unsigned)(k * N + n64 * 64 + ch * 8) * 2u; }
  const unsigned ldsw = rfl((unsigned)(size_t)lds + (unsigned)wid * 1024u);
  const int aoff0 = (wr * 64 + fr) * 128 + ((fq ^ (fr >> 1)) << 4), aoff1 = aoff0 ^ 64;
  const int q = fr >> 2, p = fr & 3;
  const int brest = (fq >> 1) * 4096 + q * 512 + (fq & 1) * 128 + 8 * p + (wc >> 1) * 256;
  const int boff0 = brest + ((((wc & 1) * 2 + 0) ^ q) << 5), boff1 = brest + ((((wc & 1) * 2 + 1) ^ q) << 5);
  const unsigned voffO = (unsigned)((wr * 64 + fr) * NO * 2 + (wc * 32 + (fq & 1) * 16 + (fq >> 1) * 8) * 2);
  const Unit u0 = unit_of(0, x, j, rank, peers);
#define SA(b_, h) (((b_) * 2 + (h)) * HTB)
#define SB(b_, h) ((4 + (b_) * 2 + (h)) * HTB)
#define STAGE_A(bufoff, so, GO) do { BLDS(srdA, voffA, so, (GO), ldsw, (bufoff)); BLDS(srdA, voffA, so, (GO) + 64 * K * 2, ldsw, (bufoff) + 8192); } while (0)
#define STAGE_B(bufoff, so, GO) do { BLDSN(srdB, voffB, so, (GO), ldsw, (bufoff)); BLDSN(srdB, voffB, so, (GO) + 32 * N * 2, ldsw, (bufoff) + 8192); } while (0)
#define LDA(dst_, b_, h) do { _Pragma("unroll") for (int m = 0; m < 4; ++m) { dst_[m][0] = *(const LAS bf16x8*)(lds + SA(b_, h) + aoff0 + m * 2048); dst_[m][1] = *(const LAS bf16x8*)(lds + SA(b_, h) + aoff1 + m * 2048); } } while (0)
#define LDB1(dst_, b_, h, boff, s) do { s16x4 lo = trrd(lds + SB(b_, h) + boff + (s) * 8192), hi = trrd(lds + SB(b_, h) + boff + (s) * 8192 + 2048); dst_ = __builtin_shufflevector(lo, hi, 0, 1, 2, 3, 4, 5, 6, 7); } while (0)
#define LDB(dst_, b_, h) do { LDB1(dst_[0][0], b_, h, boff0, 0); LDB1(dst_[0][1], b_, h, boff0, 1); LDB1(dst_[1][0], b_, h, boff1, 0); LDB1(dst_[1][1], b_, h, boff1, 1); } while (0)
#define MMA(ai, bj, At, Bt) do { _Pragma("unroll") for (int k = 0; k < 2; ++k) _Pragma("unroll") for (int m = 0; m < 4; ++m) _Pragma("unroll") for (int n = 0; n < 2; ++n) \
    acc[ai][bj][m][n] = __builtin_amdgcn_mfma_f32_16x16x32_bf16(Bt[n][k], At[m][k], acc[ai][bj][m][n], 0, 0, 0); } while (0)
#define PRIO(n) __builtin_amdgcn_s_setprio(n)
#define PACK(w, ai, bj, m) do { const f32x4 x_ = acc[ai][bj][m][0], y_ = acc[ai][bj][m][1]; \
    auto r0_ = __builtin_amdgcn_permlane16_swap(cvtpk(x_[0], x_[1]), cvtpk(y_[0], y_[1]), false, false); \
    auto r1_ = __builtin_amdgcn_permlane16_swap(cvtpk(x_[2], x_[3]), cvtpk(y_[2], y_[3]), false, false); \
    w = (u32x4){r0_[0], r1_[0], r0_[1], r1_[1]}; } while (0)
#define E1(eb, ai, bj, m) do { u32x4 w_; PACK(w_, ai, bj, m); BST4(EPOL, w_, srdP, vl16, eb, ((ai) * 8 + (bj) * 4 + (m)) * 8192); \
    acc[ai][bj][m][0] = zero4(); acc[ai][bj][m][1] = zero4(); } while (0)
#define E4(eb, ai, bj) do { E1(eb, ai, bj, 0); E1(eb, ai, bj, 1); E1(eb, ai, bj, 2); E1(eb, ai, bj, 3); } while (0)
#define E16(eb) do { E4(eb, 0, 0); E4(eb, 0, 1); E4(eb, 1, 0); E4(eb, 1, 1); } while (0)
  bf16x8 At[4][2], B0[2][2], B1[2][2];
  unsigned cA = u0.a, cB = u0.b;
  STAGE_B(SB(0, 0), cB, 0); STAGE_B(SB(0, 1), cB, hB); STAGE_A(SA(0, 0), cA, 0); STAGE_A(SA(0, 1), cA, hA);
  const Unit u1 = unit_of(1, x, j, rank, peers);
  f32x4 acc[2][2][4][2];
#pragma unroll
  for (int a = 0; a < 2; ++a)
#pragma unroll
    for (int c = 0; c < 2; ++c)
#pragma unroll
      for (int m = 0; m < 4; ++m)
#pragma unroll
        for (int n = 0; n < 2; ++n) acc[a][c][m][n] = zero4();
  if (wr == 1) BAR;
  WAIT_V(2); BAR; MEMF;
  STAGE_B(SB(1, 0), cB, kB); STAGE_A(SA(1, 0), cA, kA); STAGE_B(SB(1, 1), cB, hB + kB);
  asm volatile("" : "+v"(cntv));
  const unsigned call = rfl(cntv) + 1u;
  const unsigned vcall = call;
  WAIT_V(6); BAR; MEMF;
  STAMP(1);
  const unsigned sp = (unsigned)x & 3u, T = ((unsigned)x >> 2) * 32u + (unsigned)j;
  const unsigned pb = WS_PART + T * 4u * UNIT_B + (unsigned)wid * 1024u;
  const unsigned fb = WS_PFLAG + (T * 8u + (unsigned)wid) * 32u;
  const unsigned n1 = (unsigned)NSTAG;
  float rs[8]; u32x4 Xa, Xb, Ya, Yb, ow; unsigned fv;
  unsigned* tmo = (unsigned*)(myws + WS_TMO);
  unsigned bad = 0;
  unsigned nA = u1.a, nB = u1.b;
  u32x4 srdO = {u0.olo, u0.ohi, 0u, 0x00020000u}, srdZ = srdC; srdZ[2] = 0;
  unsigned lo = pb + sp * 4u * 8192u, oo = u0.obase + piece_off(sp * 4u) - 16u * NO * 2u;
#define LDA2(dst_, b_, h, m0) do { _Pragma("unroll") for (int m = m0; m < m0 + 2; ++m) { dst_[m][0] = *(const LAS bf16x8*)(lds + SA(b_, h) + aoff0 + m * 2048); dst_[m][1] = *(const LAS bf16x8*)(lds + SA(b_, h) + aoff1 + m * 2048); } } while (0)
#define MF(i) acc[1][(i) >> 4][((i) >> 1) & 3][(i) & 1] = __builtin_amdgcn_mfma_f32_16x16x32_bf16(((i) >> 4) ? B1[(i) & 1][((i) >> 3) & 1] : B0[(i) & 1][((i) >> 3) & 1], At[((i) >> 1) & 3][((i) >> 3) & 1], acc[1][(i) >> 4][((i) >> 1) & 3][(i) & 1], 0, 0, 0)
#define UNP(L, e) (((e) & 1) ? bhi(L[(e) >> 1]) : blo(L[(e) >> 1]))
#define SP1LOAD(b_, abuf, aso, XA, XB) do { LDB(B0, b_, 0); BLDS(srdA, voffA, aso, hA, ldsw, abuf); LDB(B1, b_, 1); BLDS(srdA, voffA, aso, hA + 64 * K * 2, ldsw, (abuf) + 8192); \
    LDA2(At, b_, 0, 0); XA; LDA2(At, b_, 0, 2); XB; } while (0)
#define TRIP(KIND, t) do { \
    const bool last = (t) == 30u; \
    const unsigned a1 = cA + ((t) + 1u) * kA; \
    const unsigned a2 = last ? nA : cA + ((t) + 2u) * kA, b2 = last ? nB : cB + ((t) + 2u) * kB; \
    if ((KIND) == 3) SP1LOAD(0, SA(1, 1), a1, BLD4(Xa, srdP, vl16, lo, 0), BLD4(Xb, srdP, vl16, lo, UNIT_B)); \
    else if ((KIND) == 2) SP1LOAD(0, SA(1, 1), a1, BLD1(fv, srdC, vl4, fb), (void)0); \
    else SP1LOAD(0, SA(1, 1), a1, (void)0, (void)0); \
    WAIT_V((KIND) == 1 ? 24 : (KIND) == 2 ? 9 : (KIND) == 3 ? 13 : (KIND) == 4 ? 11 : 8); \
    WAIT_L(0); BAR; PRIO(1); MMA(0, 0, At, B0); MMA(0, 1, At, B1); PRIO(0); BAR; MEMF; SCHED; \
    LDA(At, 0, 1); STAGE_B(SB(0, 0), b2, 0); STAGE_B(SB(0, 1), b2, hB); STAGE_A(SA(0, 0), a2, 0); \
    WAIT_V((KIND) == 1 ? 24 : (KIND) == 2 ? 9 : (KIND) == 3 ? 10 : 8); \
    if ((KIND) >= 3) asm volatile("" : "+v"(Ya), "+v"(Yb)); \
    WAIT_L(0); BAR; SCHED; PRIO(1); \
    if ((KIND) >= 3) {   \
      float tp = 0.f; \
      _Pragma("unroll") for (int e = 0; e < 8; ++e) { float q0, q1; \
        MF(4 * e); q0 = UNP(Ya, e); SCHED; MF(4 * e + 1); q1 = UNP(Yb, e); SCHED; MF(4 * e + 2); q0 = fadd1(q0, q1); SCHED; \
        MF(4 * e + 3); q0 = fadd1(rs[e], q0); if (e & 1) ow[e >> 1] = cvtpk(tp, q0); tp = q0; SCHED; } \
    } else { MMA(1, 0, At, B0); MMA(1, 1, At, B1); } \
    PRIO(0); BAR; MEMF; SCHED; \
    if ((KIND) == 3) { SP1LOAD(1, SA(0, 1), a2, BLD4(Ya, srdP, vl16, lo, 2 * UNIT_B), BLD4(Yb, srdP, vl16, lo, 3 * UNIT_B)); BST4(OPOL, ow, srdO, voffO, oo, 0); } \
    else if ((KIND) == 4) SP1LOAD(1, SA(0, 1), a2, (void)0, BST4(OPOL, ow, srdO, voffO, oo, 0)); \
    else if ((KIND) == 2) { SP1LOAD(1, SA(0, 1), a2, BST1("", vcall, srdZ, vl4, fb), BST1("", vcall, srdZ, vl4, fb)); BST1("", vcall, srdZ, vl4, fb); }     \
    else SP1LOAD(1, SA(0, 1), a2, (void)0, (void)0); \
    WAIT_V((KIND) == 2 ? 12 : (KIND) == 3 ? 13 : (KIND) == 4 ? 9 : 8); \
    if ((KIND) == 1) { const unsigned pubo = fb + sp * 4u; u32x4 sc = srdC; sc[2] = pubo + 4u; BST1("sc1", vcall, sc, vl4, pubo); }     \
    WAIT_L(0); BAR; PRIO(1); MMA(0, 0, At, B0); MMA(0, 1, At, B1); PRIO(0); BAR; MEMF; SCHED; \
    LDA(At, 1, 1); STAGE_B(SB(1, 0), b2, kB); STAGE_B(SB(1, 1), b2, kB + hB); STAGE_A(SA(1, 0), a2, kA); \
    WAIT_V((KIND) == 1 || (KIND) == 4 ? 9 : (KIND) == 2 || (KIND) == 3 ? 11 : 8); \
    if ((KIND) == 3) asm volatile("" : "+v"(Xa), "+v"(Xb)); \
    if ((KIND) == 2) asm volatile("" : "+v"(fv)); \
    WAIT_L(0); BAR; SCHED; PRIO(1); \
    if ((KIND) == 3) {   \
      _Pragma("unroll") for (int e = 0; e < 8; ++e) { float q0, q1; \
        MF(4 * e); q0 = UNP(Xa, e); SCHED; MF(4 * e + 1); q1 = UNP(Xb, e); SCHED; MF(4 * e + 2); rs[e] = fadd1(q0, q1); SCHED; MF(4 * e + 3); SCHED; } \
      lo += 8192u; oo += 16u * NO * 2u; srdO[2] = OUT_BLK; \
    } else { MMA(1, 0, At, B0); MMA(1, 1, At, B1); } \
    PRIO(0); BAR; MEMF; SCHED; \
    STAMP(16 + ((t) >> 2) + ((KIND) ? 8 : sbase)); \
  } while (0)
  auto wait_flags = [&](unsigned f) {
    const unsigned long long t0 = __builtin_amdgcn_s_memrealtime();
    for (;;) {
      unsigned v; BLD1(v, srdC, vl4, fb); WAIT_V(0); asm volatile("" : "+v"(v));
      if (((__builtin_amdgcn_ballot_w64(v >= call) >> (4u * f)) & 15ull) == 15ull) break;
      if (rfl(__hip_atomic_load(tmo, __ATOMIC_RELAXED, __HIP_MEMORY_SCOPE_AGENT)) == call || __builtin_amdgcn_s_memrealtime() - t0 > DEADLINE_TICKS) {
        __hip_atomic_store(tmo, call, __ATOMIC_RELAXED, __HIP_MEMORY_SCOPE_AGENT); bad = 1; break; }
      __builtin_amdgcn_s_sleep(SLP);
    }
    bad = rfl(bad);
  };
#define SUM4(ow_, R) do { asm volatile("" : "+v"(R[0]), "+v"(R[1]), "+v"(R[2]), "+v"(R[3])); _Pragma("unroll") for (int e = 0; e < 4; ++e) { \
        const float lo_ = (blo(R[0][e]) + blo(R[1][e])) + (blo(R[2][e]) + blo(R[3][e])); \
        const float hi_ = (bhi(R[0][e]) + bhi(R[1][e])) + (bhi(R[2][e]) + bhi(R[3][e])); \
        ow_[e] = bad ? 0x7fc07fc0u : cvtpk(lo_, hi_); } } while (0)
#define LD4(R, l, OFF) do { BLD4(R[0], srdP, vl16, l, (OFF) + 0 * UNIT_B); BLD4(R[1], srdP, vl16, l, (OFF) + 1 * UNIT_B); BLD4(R[2], srdP, vl16, l, (OFF) + 2 * UNIT_B); BLD4(R[3], srdP, vl16, l, (OFF) + 3 * UNIT_B); } while (0)
  unsigned t = 0, sbase = 0;
#pragma unroll 1
  for (; t < 32u; t += 2) TRIP(0, t);
  STAMP(2);
  { const unsigned eb = pb + sp * UNIT_B; E16(eb); }
  STAMP(3);
  cA = nA; cB = nB; sbase = 8;
  TRIP(1, 0u);
#pragma unroll 1
  for (t = 2; t < 2u + 2u * n1; t += 2) TRIP(0, t);
  STAMP(4);
  TRIP(2, t); t += 2;
  if ((__builtin_amdgcn_ballot_w64(fv >= call) & 15ull) != 15ull) { wait_flags(0u); BST1("", vcall, srdZ, vl4, fb); BST1("", vcall, srdZ, vl4, fb); BST1("", vcall, srdZ, vl4, fb); }
  STAMP(5);
  asm volatile("" : "=v"(Ya), "=v"(Yb), "=v"(rs[0]), "=v"(rs[1]), "=v"(rs[2]), "=v"(rs[3]), "=v"(rs[4]), "=v"(rs[5]), "=v"(rs[6]), "=v"(rs[7]));
#pragma unroll 1
  for (int i = 0; i < 4; ++i) { TRIP(3, t); t += 2; }
  TRIP(4, t); t += 2;
  STAMP(6);
#pragma unroll 1
  for (; t < 32u; t += 2) TRIP(0, t);
  STAMP(8);
  { const unsigned eb = pb + ROUND_B + sp * UNIT_B; E16(eb); }
  if (wr == 0) BAR;
  STAMP(9);
  WAIT_V(0);
  STAMP(10);
  { u32x4 sc = srdC; const unsigned pubo = fb + 16u + sp * 4u; sc[2] = pubo + 4u; BST1("sc1", vcall, sc, vl4, pubo); }
  { u32x4 sc = srdC; const unsigned so = WS_CNT + (unsigned)(b * 8 + wid) * 4u; sc[2] = so + 4u; BST1("sc1", vcall, sc, vl4, so); }
  STAMP(11);
  wait_flags(1u);
  STAMP(12);
  {
    const unsigned l = pb + ROUND_B + sp * 4u * 8192u, o = u1.obase + piece_off(sp * 4u);
    u32x4 R0[4], R1[4], R2[4], R3[4]; LD4(R0, l, 0); LD4(R1, l, 8192); LD4(R2, l, 16384); LD4(R3, l, 24576);
    const u32x4 so = {u1.olo, u1.ohi, OUT_BLK, 0x00020000u};
    WAIT_V(12); { u32x4 w; SUM4(w, R0); BST4(OPOL, w, so, voffO, o, 0); }
    WAIT_V(9); { u32x4 w; SUM4(w, R1); BST4(OPOL, w, so, voffO, o, 16 * NO * 2); }
    WAIT_V(6); { u32x4 w; SUM4(w, R2); BST4(OPOL, w, so, voffO, o, 32 * NO * 2); }
    WAIT_V(3); { u32x4 w; SUM4(w, R3); BST4(OPOL, w, so, voffO, o, 48 * NO * 2); }
  }
  STAMP(13);
  WAIT_V(0);
  STAMP(14);
  MEMF; BAR; MEMF;
  if (wid == 0) {
    const unsigned pr = (unsigned)lane;
    const bool mine = pr < (unsigned)NR && pr != (unsigned)rank;
    if (mine) __hip_atomic_store((unsigned*)(peers.ws[pr] + WS_DONE) + rank * 256 + b, call, __ATOMIC_RELAXED, __HIP_MEMORY_SCOPE_SYSTEM);
    bool late = false;
    if (mine) {
      const unsigned* flag = (const unsigned*)(myws + WS_DONE) + pr * 256 + b;
      const unsigned long long t0 = __builtin_amdgcn_s_memrealtime();
      while (__hip_atomic_load(flag, __ATOMIC_RELAXED, __HIP_MEMORY_SCOPE_SYSTEM) < call) {
        if (__hip_atomic_load(tmo, __ATOMIC_RELAXED, __HIP_MEMORY_SCOPE_AGENT) == call || __builtin_amdgcn_s_memrealtime() - t0 > DEADLINE_TICKS) {
          __hip_atomic_store(tmo, call, __ATOMIC_RELAXED, __HIP_MEMORY_SCOPE_AGENT); late = true; break; }
        __builtin_amdgcn_s_sleep(SLP);
      }
    }
    __builtin_amdgcn_fence(__ATOMIC_ACQUIRE, "");
    if (late || __hip_atomic_load(tmo, __ATOMIC_RELAXED, __HIP_MEMORY_SCOPE_AGENT) == call) peers.out[rank][(size_t)rank * M * NO + (size_t)b] = (u16)0x7fc0;
  }
}

extern "C" void kernel_launch(int rank, int nranks, ncclComm_t comm, hipStream_t stream,
                              void* const* d_in, const int* in_sizes, int n_in,
                              void* d_out, int out_size, const int* in_dtypes, int out_dtype,
                              void* const* d_out_all, void* const* d_ws_all, size_t ws_bytes) {
  if (nranks != NR || (unsigned)rank >= (unsigned)NR || ws_bytes < WS_NEED) return;
  HIP_CHECK(hipFuncSetAttribute((const void*)gemm_a2a, hipFuncAttributeMaxDynamicSharedMemorySize, LDS_BYTES));
  Peers p;
  for (int i = 0; i < NR; ++i) { p.out[i] = (u16*)d_out_all[i]; p.ws[i] = (char*)d_ws_all[i]; }
  gemm_a2a<<<256, 512, LDS_BYTES, stream>>>((const u16*)d_in[0], (const u16*)d_in[1], p, rank);
  HIP_CHECK(hipGetLastError());
}
```

```cpp
#include <hip/hip_runtime.h>
#include <rccl/rccl.h>
#include <cstdio>

#define HIP_CHECK(expr) do { hipError_t _e = (expr); if (_e != hipSuccess) { \
    fprintf(stderr, "HIP error %s at %s:%d\n", hipGetErrorString(_e), __FILE__, __LINE__); \
    return; } } while (0)

#define LAS __attribute__((address_space(3)))
typedef unsigned short u16;
typedef short bf16x8 __attribute__((ext_vector_type(8)));
typedef short s16x4 __attribute__((ext_vector_type(4)));
typedef float f32x4 __attribute__((ext_vector_type(4)));
typedef float f32x2 __attribute__((ext_vector_type(2)));
typedef unsigned u32x4 __attribute__((ext_vector_type(4)));
typedef __bf16 bf16x2_t __attribute__((ext_vector_type(2)));
constexpr int M = 1024, N = 8192, K = 8192, NR = 8, NO = 1024;
constexpr unsigned long long DEADLINE_TICKS = 100000000ull;
constexpr unsigned WS_TMO = 0;
constexpr unsigned WS_CNT = 4096;
constexpr unsigned WS_DONE = 16384;
constexpr unsigned WS_PFLAG = 65536;
constexpr unsigned WS_CTRL_BYTES = 131072;
constexpr unsigned WS_PART = 1u << 18;
constexpr unsigned UNIT_B = 131072, ROUND_B = 256u * UNIT_B, PART_BYTES = 2u * ROUND_B;
constexpr size_t WS_NEED = (size_t)WS_PART + PART_BYTES;
constexpr unsigned OUT_BLK = (unsigned)M * NO * 2;
constexpr int HTB = 16384, STASH = 8 * HTB, LDS_BYTES = 10 * HTB;
constexpr unsigned kA = 128, kB = 64 * N * 2, hA = 128 * K * 2, hB = 256;
constexpr int NK1 = 32;
#define SLP 4
#define NSTAG ((wid + j) & 7)

__device__ __forceinline__ unsigned cvtpk(float lo, float hi) { f32x2 v = {lo, hi}; bf16x2_t b = __builtin_convertvector(v, bf16x2_t); return __builtin_bit_cast(unsigned, b); }
__device__ __forceinline__ s16x4 trrd(LAS const char* p) { return __builtin_amdgcn_ds_read_tr16_b64_v4i16((LAS s16x4*)p); }
__device__ __forceinline__ unsigned rfl(unsigned x) { return __builtin_amdgcn_readfirstlane(x); }
#define WAIT_V(n) asm volatile("s_waitcnt vmcnt(%0)" :: "n"(n) : "memory")
#define WAIT_L(n) asm volatile("s_waitcnt lgkmcnt(%0)" :: "n"(n) : "memory")
#define BAR __builtin_amdgcn_s_barrier()
#define SCHED __builtin_amdgcn_sched_barrier(0)
#define MEMF asm volatile("" ::: "memory")
#define BLDS(srd, voff, soff, GOFF, ldsbase, LOFF) do { unsigned tmp_; asm volatile("s_add_u32 m0, %1, %2\n\ts_add_u32 %0, %3, %4\n\tbuffer_load_dwordx4 %5, %6, %0 offen lds" \
   : "=&s"(tmp_) : "s"(ldsbase), "i"(LOFF), "s"(soff), "i"(GOFF), "v"(voff), "s"(srd) : "memory", "scc"); } while (0)
#define BLD4(dst, srd, voff, soff, GOFF) do { unsigned tmp_; asm volatile("s_add_u32 %1, %2, %3\n\tbuffer_load_dwordx4 %0, %4, %5, %1 offen sc1" \
   : "=v"(dst), "=&s"(tmp_) : "s"(soff), "i"(GOFF), "v"(voff), "s"(srd) : "memory", "scc"); } while (0)
#define BLD1(dst, srd, voff, soff) asm volatile("buffer_load_dword %0, %1, %2, %3 offen sc1" : "=v"(dst) : "v"(voff), "s"(srd), "s"(soff) : "memory")
#define BST4(MOD, data, srd, voff, soff, GOFF) do { unsigned tmp_; asm volatile("s_add_u32 %0, %1, %2\n\tbuffer_store_dwordx4 %3, %4, %5, %0 offen " MOD "\n\ts_nop 1" \
   : "=&s"(tmp_) : "s"(soff), "i"(GOFF), "v"(data), "v"(voff), "s"(srd) : "memory", "scc"); } while (0)
#define BST1(MOD, data, srd, voff, soff) asm volatile("buffer_store_dword %0, %1, %2, %3 offen " MOD :: "v"(data), "v"(voff), "s"(srd), "s"(soff) : "memory")
__device__ __forceinline__ u32x4 make_srd(const void* p, unsigned bytes) {
  unsigned long long a = (unsigned long long)p;
  u32x4 r; r[0] = rfl((unsigned)a); r[1] = rfl((unsigned)(a >> 32) & 0xffffu); r[2] = bytes; r[3] = 0x00020000u; return r;
}
struct Peers { u16* out[NR]; char* ws[NR]; };
struct Unit { unsigned a, b, obase, olo, ohi; };

__device__ __forceinline__ Unit unit_of(int r, int x, int j, int rank, const Peers& peers) {
  const int h = x >> 2; int d, pnl, pm;
  if (r == 0) {
    if (j < 28) { d = 1 + (j >> 2); pnl = h; pm = j & 3; }
    else { const int e = h * 4 + j - 28; d = e < 7 ? 1 + e : 7; pm = e < 7 ? 0 : 1; pnl = 2; }
  } else if (h == 0) {
    if (j < 28) { d = 1 + (j >> 2); pnl = 3; pm = j & 3; } else { d = 0; pnl = 0; pm = j - 28; }
  } else {
    if (j < 18) { d = 1 + j / 3; pm = 1 + j % 3; pnl = 2; } else if (j < 20) { d = 7; pm = j - 16; pnl = 2; } else { d = 0; pnl = 1 + ((j - 20) >> 2); pm = j & 3; }
  }
  const unsigned dst = (unsigned)((rank + d) & (NR - 1));
  const unsigned long long o = (unsigned long long)peers.out[dst] + (unsigned long long)rank * OUT_BLK;
  Unit u; u.a = (unsigned)(pm * 256 * K * 2); u.b = (unsigned)(((int)dst * 4 + pnl) * 512); u.obase = (unsigned)(pm * 256 * NO * 2 + pnl * 512);
  u.olo = rfl((unsigned)o); u.ohi = rfl((unsigned)(o >> 32) & 0xffffu);
  return u;
}
__device__ __forceinline__ float blo(unsigned w) { return __uint_as_float(w << 16); }
__device__ __forceinline__ float bhi(unsigned w) { return __uint_as_float(w & 0xffff0000u); }
__device__ __forceinline__ f32x4 zero4() { f32x2 a, b; asm volatile("v_mov_b64 %0, 0" : "=v"(a)); asm volatile("v_mov_b64 %0, 0" : "=v"(b)); return __builtin_shufflevector(a, b, 0, 1, 2, 3); }
__device__ __forceinline__ float fadd1(float a, float b) { float r; asm("v_add_f32_e32 %0, %1, %2" : "=v"(r) : "v"(a), "v"(b)); return r; }

#define STAMP(i) do {} while (0)
__global__ __launch_bounds__(512, 2) void gemm_a2a(const u16* __restrict__ A, const u16* __restrict__ W, Peers peers, int rank) {
  extern __shared__ __attribute__((aligned(16))) char smem_[];
  LAS char* lds = (LAS char*)smem_;
  const int tid = threadIdx.x, wid = __builtin_amdgcn_readfirstlane(tid >> 6), lane = tid & 63, wr = wid >> 2, wc = wid & 3, fr = lane & 15, fq = lane >> 4;
  const int b = blockIdx.x, x = b & 7, j = b >> 3;
  char* myws = peers.ws[rank];
  unsigned tsv = 0; STAMP(0);
  const u32x4 srdA = make_srd(A, (unsigned)M * K * 2), srdB = make_srd(W, (unsigned)K * N * 2), srdC = make_srd(myws, WS_CTRL_BYTES), srdP = make_srd(myws, (unsigned)WS_NEED);
  const unsigned vl16 = (unsigned)lane * 16u, vl4 = (unsigned)lane * 4u;
  unsigned cntv; { const unsigned so = WS_CNT + (unsigned)(b * 8 + wid) * 4u; BLD1(cntv, srdC, vl4, so); }
  unsigned voffA, voffB;
  { const int row = wid * 8 + (lane >> 3); const int ch = (lane & 7) ^ ((row >> 1) & 7); voffA = (unsigned)(row * K + ch * 8) * 2u; }
  { const int P = wid * 8 + (lane >> 3); const int k = ((P >> 5) << 4) | ((P & 1) << 3) | ((P >> 2) & 7), n64 = (P >> 1) & 1;
    const int sg = ((lane & 7) >> 1) ^ (k & 3), ch = sg * 2 + (lane & 1); voffB = (unsigned)(k * N + n64 * 64 + ch * 8) * 2u; }
  const unsigned ldsw = rfl((unsigned)(size_t)lds + (unsigned)wid * 1024u);
  const int aoff0 = (wr * 64 + fr) * 128 + ((fq ^ (fr >> 1)) << 4), aoff1 = aoff0 ^ 64;
  const int q = fr >> 2, p = fr & 3;
  const int brest = (fq >> 1) * 4096 + q * 512 + (fq & 1) * 128 + 8 * p + (wc >> 1) * 256;
  const int boff0 = brest + ((((wc & 1) * 2 + 0) ^ q) << 5), boff1 = brest + ((((wc & 1) * 2 + 1) ^ q) << 5);
  const unsigned voffO = (unsigned)((wr * 64 + fr) * NO * 2 + (wc * 32 + (fq & 1) * 16 + (fq >> 1) * 8) * 2);
  const unsigned sp = (unsigned)x & 3u, oa = sp >> 1, ob = sp & 1u;
  const unsigned dA = oa ? 0u - hA : hA, dB = ob ? 0u - hB : hB;
  const Unit u0 = unit_of(0, x, j, rank, peers);
#define SA(b_, h) (((b_) * 2 + (h)) * HTB)
#define SB(b_, h) ((4 + (b_) * 2 + (h)) * HTB)
#define STAGE_A(bufoff, so) do { BLDS(srdA2, voffA, so, 0, ldsw, (bufoff)); BLDS(srdA2, voffA, so, 64 * K * 2, ldsw, (bufoff) + 8192); } while (0)
#define STAGE_B(bufoff, so) do { BLDS(srdB2, voffB, so, 0, ldsw, (bufoff)); BLDS(srdB2, voffB, so, 32 * N * 2, ldsw, (bufoff) + 8192); } while (0)
#define LDA2(dst_, b_, h, m0) do { _Pragma("unroll") for (int m = m0; m < m0 + 2; ++m) { dst_[m][0] = *(const LAS bf16x8*)(lds + SA(b_, h) + aoff0 + m * 2048); dst_[m][1] = *(const LAS bf16x8*)(lds + SA(b_, h) + aoff1 + m * 2048); } } while (0)
#define LDA(dst_, b_, h) do { LDA2(dst_, b_, h, 0); LDA2(dst_, b_, h, 2); } while (0)
#define LDB1(dst_, b_, h, boff, s) do { s16x4 lo_ = trrd(lds + SB(b_, h) + boff + (s) * 8192), hi_ = trrd(lds + SB(b_, h) + boff + (s) * 8192 + 2048); dst_ = __builtin_shufflevector(lo_, hi_, 0, 1, 2, 3, 4, 5, 6, 7); } while (0)
#define LDB(dst_, b_, h) do { LDB1(dst_[0][0], b_, h, boff0, 0); LDB1(dst_[0][1], b_, h, boff0, 1); LDB1(dst_[1][0], b_, h, boff1, 0); LDB1(dst_[1][1], b_, h, boff1, 1); } while (0)
#define MMA(ai, bj, At, Bt) do { _Pragma("unroll") for (int k = 0; k < 2; ++k) _Pragma("unroll") for (int m = 0; m < 4; ++m) _Pragma("unroll") for (int n = 0; n < 2; ++n) \
    acc[ai][bj][m][n] = __builtin_amdgcn_mfma_f32_16x16x32_bf16(Bt[n][k], At[m][k], acc[ai][bj][m][n], 0, 0, 0); } while (0)
#define PRIO(n) __builtin_amdgcn_s_setprio(n)
#define PACK(w, ai, bj, m) do { const f32x4 x_ = acc[ai][bj][m][0], y_ = acc[ai][bj][m][1]; \
    auto r0_ = __builtin_amdgcn_permlane16_swap(cvtpk(x_[0], x_[1]), cvtpk(y_[0], y_[1]), false, false); \
    auto r1_ = __builtin_amdgcn_permlane16_swap(cvtpk(x_[2], x_[3]), cvtpk(y_[2], y_[3]), false, false); \
    w = (u32x4){r0_[0], r1_[0], r0_[1], r1_[1]}; } while (0)
#define E1(ZERO, eq, ai, bj, m) do { u32x4 w_; PACK(w_, ai, bj, m); BST4("sc1", w_, srdP, vl16, eq, (m) * 8192); if (ZERO) { acc[ai][bj][m][0] = zero4(); acc[ai][bj][m][1] = zero4(); } } while (0)
#define E4(ZERO, eb, ai, bj) do { const unsigned eq_ = (eb) + ((sp ^ (unsigned)((ai) * 2 + (bj))) << 15); E1(ZERO, eq_, ai, bj, 0); E1(ZERO, eq_, ai, bj, 1); E1(ZERO, eq_, ai, bj, 2); E1(ZERO, eq_, ai, bj, 3); } while (0)
  bf16x8 At[4][2], B0[2][2], B1[2][2];
  const u32x4 srdA2 = srdA, srdB2 = srdB;
  unsigned cA = u0.a + oa * hA + sp * 32u * kA, cB = u0.b + ob * hB + sp * 32u * kB;
  { const unsigned bh = cB + dB, ah = cA + dA; STAGE_B(SB(0, 0), cB); STAGE_B(SB(0, 1), bh); STAGE_A(SA(0, 0), cA); STAGE_A(SA(0, 1), ah); }
  const Unit u1 = unit_of(1, x, j, rank, peers);
  f32x4 acc[2][2][4][2];
#pragma unroll
  for (int a = 0; a < 2; ++a)
#pragma unroll
    for (int c = 0; c < 2; ++c)
#pragma unroll
      for (int m = 0; m < 4; ++m)
#pragma unroll
        for (int n = 0; n < 2; ++n) acc[a][c][m][n] = zero4();
  if (wr == 1) BAR;
  WAIT_V(2); BAR; MEMF;
  { const unsigned b1 = cB + kB, a1 = cA + kA, b1h = b1 + dB; STAGE_B(SB(1, 0), b1); STAGE_A(SA(1, 0), a1); STAGE_B(SB(1, 1), b1h); }
  asm volatile("" : "+v"(cntv));
  const unsigned call = rfl(cntv) + 1u;
  const unsigned vcall = call;
  WAIT_V(6); BAR; MEMF;
  STAMP(1);
  const unsigned T = ((unsigned)x >> 2) * 32u + (unsigned)j;
  const unsigned pb = WS_PART + T * 4u * UNIT_B + (unsigned)wid * 1024u;
  const unsigned fb = WS_PFLAG + (T * 8u + (unsigned)wid) * 32u;
  const unsigned qo = oa * (128u * NO * 2u) + ob * 256u;
  const unsigned n1 = (unsigned)(NSTAG);
  float rs[8]; u32x4 Xa, Xb, Ya, Yb, ow; unsigned fv;
  unsigned* tmo = (unsigned*)(myws + WS_TMO);
  unsigned bad = 0;
  unsigned nA = u1.a + oa * hA + sp * (unsigned)NK1 * kA, nB = u1.b + ob * hB + sp * (unsigned)NK1 * kB, tl = 30u;
  u32x4 srdO = {u0.olo, u0.ohi, 0u, 0x00020000u}, srdZ = srdC; srdZ[2] = 0;
  unsigned l1 = pb + (sp << 15) + ((sp + 1u) & 3u) * UNIT_B, l2 = pb + (sp << 15) + ((sp + 2u) & 3u) * UNIT_B, l3 = pb + (sp << 15) + ((sp + 3u) & 3u) * UNIT_B, oo = u0.obase + qo - 16u * NO * 2u, nrec = ~0u;
  int sto = STASH + wid * 1024 + lane * 16;
#define MF(i) acc[1][(i) >> 4][((i) >> 1) & 3][(i) & 1] = __builtin_amdgcn_mfma_f32_16x16x32_bf16(((i) >> 4) ? B1[(i) & 1][((i) >> 3) & 1] : B0[(i) & 1][((i) >> 3) & 1], At[((i) >> 1) & 3][((i) >> 3) & 1], acc[1][(i) >> 4][((i) >> 1) & 3][(i) & 1], 0, 0, 0)
#define UNP(L, e) (((e) & 1) ? bhi(L[(e) >> 1]) : blo(L[(e) >> 1]))
#define SP1LOAD(b_, abuf, aso, XA, XB) do { const u32x4 sa_ = (b_) ? srdA2 : srdA; LDB(B0, b_, 0); BLDS(sa_, voffA, aso, 0, ldsw, abuf); LDB(B1, b_, 1); BLDS(sa_, voffA, aso, 64 * K * 2, ldsw, (abuf) + 8192); \
    LDA2(At, b_, 0, 0); XA; LDA2(At, b_, 0, 2); XB; } while (0)
#define TRIP(KIND, t) do { \
    const bool last = (t) == tl; \
    const unsigned a1h = cA + ((t) + 1u) * kA + dA; \
    const unsigned a2 = last ? nA : cA + ((t) + 2u) * kA, b2 = last ? nB : cB + ((t) + 2u) * kB; \
    const unsigned a2h = a2 + dA, b2h = b2 + dB, a3 = a2 + kA, b3 = b2 + kB, b3h = b3 + dB; \
    u32x4 srdA2 = srdA, srdB2 = srdB; srdA2[2] &= last ? nrec : ~0u; srdB2[2] &= last ? nrec : ~0u; \
    if ((KIND) == 3) SP1LOAD(0, SA(1, 1), a1h, BLD4(Xa, srdP, vl16, l1, 0), BLD4(Xb, srdP, vl16, l2, 0)); \
    else if ((KIND) == 2) SP1LOAD(0, SA(1, 1), a1h, BLD1(fv, srdC, vl4, fb), (void)0); \
    else SP1LOAD(0, SA(1, 1), a1h, (void)0, (void)0); \
    WAIT_V((KIND) == 1 ? 20 : (KIND) == 2 ? 9 : (KIND) == 3 ? 12 : (KIND) == 4 ? 10 : 8); \
    WAIT_L(0); BAR; PRIO(1); MMA(0, 0, At, B0); MMA(0, 1, At, B1); PRIO(0); BAR; MEMF; SCHED; \
    LDA(At, 0, 1); STAGE_B(SB(0, 0), b2); STAGE_B(SB(0, 1), b2h); STAGE_A(SA(0, 0), a2); \
    WAIT_V((KIND) == 1 ? 20 : (KIND) == 2 ? 9 : (KIND) == 3 ? 10 : 8); \
    if ((KIND) >= 3) asm volatile("" : "+v"(Ya)); \
    WAIT_L(0); BAR; SCHED; PRIO(1); \
    if ((KIND) >= 3) {   \
      float tp = 0.f; \
      _Pragma("unroll") for (int e = 0; e < 8; ++e) { float q0, q1; \
        MF(4 * e); q0 = UNP(Ya, e); SCHED; MF(4 * e + 1); q1 = UNP(Yb, e); SCHED; MF(4 * e + 2); q0 = fadd1(q0, q1); SCHED; \
        MF(4 * e + 3); q0 = fadd1(rs[e], q0); if (e & 1) ow[e >> 1] = cvtpk(tp, q0); tp = q0; SCHED; } \
    } else { MMA(1, 0, At, B0); MMA(1, 1, At, B1); } \
    PRIO(0); BAR; MEMF; SCHED; \
    if ((KIND) == 3) { Yb = *(const LAS u32x4*)(lds + sto); SP1LOAD(1, SA(0, 1), a2h, BLD4(Ya, srdP, vl16, l3, 0), BST4("sc0 sc1", ow, srdO, voffO, oo, 0)); } \
    else if ((KIND) == 4) SP1LOAD(1, SA(0, 1), a2h, (void)0, BST4("sc0 sc1", ow, srdO, voffO, oo, 0)); \
    else if ((KIND) == 2) SP1LOAD(1, SA(0, 1), a2h, BST1("", vcall, srdZ, vl4, fb), BST1("", vcall, srdZ, vl4, fb));     \
    else SP1LOAD(1, SA(0, 1), a2h, (void)0, (void)0); \
    WAIT_V((KIND) == 2 ? 11 : (KIND) == 3 ? 12 : (KIND) == 4 ? 9 : 8); \
    if ((KIND) == 1) { const unsigned pubo = fb + sp * 4u; u32x4 sc = srdC; sc[2] = pubo + 4u; BST1("sc1", vcall, sc, vl4, pubo); }     \
    WAIT_L(0); BAR; PRIO(1); MMA(0, 0, At, B0); MMA(0, 1, At, B1); PRIO(0); BAR; MEMF; SCHED; \
    LDA(At, 1, 1); STAGE_B(SB(1, 0), b3); STAGE_B(SB(1, 1), b3h); STAGE_A(SA(1, 0), a3); \
    WAIT_V((KIND) == 1 || (KIND) == 4 ? 9 : (KIND) == 2 || (KIND) == 3 ? 10 : 8); \
    if ((KIND) == 3) asm volatile("" : "+v"(Xa), "+v"(Xb)); \
    if ((KIND) == 2) asm volatile("" : "+v"(fv)); \
    WAIT_L(0); BAR; SCHED; PRIO(1); \
    if ((KIND) == 3) {   \
      _Pragma("unroll") for (int e = 0; e < 8; ++e) { float q0, q1; \
        MF(4 * e); q0 = UNP(Xa, e); SCHED; MF(4 * e + 1); q1 = UNP(Xb, e); SCHED; MF(4 * e + 2); rs[e] = fadd1(q0, q1); SCHED; MF(4 * e + 3); SCHED; } \
      l1 += 8192u; l2 += 8192u; l3 += 8192u; sto += 8192; oo += 16u * NO * 2u; srdO[2] = OUT_BLK; \
    } else { MMA(1, 0, At, B0); MMA(1, 1, At, B1); } \
    PRIO(0); BAR; MEMF; SCHED; \
  } while (0)
  auto wait_flags = [&](unsigned f) {
    const unsigned long long t0 = __builtin_amdgcn_s_memrealtime();
    for (;;) {
      unsigned v; BLD1(v, srdC, vl4, fb); WAIT_V(0); asm volatile("" : "+v"(v));
      if (((__builtin_amdgcn_ballot_w64(v >= call) >> (4u * f)) & 15ull) == 15ull) break;
      if (rfl(__hip_atomic_load(tmo, __ATOMIC_RELAXED, __HIP_MEMORY_SCOPE_AGENT)) == call || __builtin_amdgcn_s_memrealtime() - t0 > DEADLINE_TICKS) {
        __hip_atomic_store(tmo, call, __ATOMIC_RELAXED, __HIP_MEMORY_SCOPE_AGENT); bad = 1; break; }
      __builtin_amdgcn_s_sleep(SLP);
    }
    bad = rfl(bad);
  };
  unsigned t = 0;
#pragma unroll 1
  for (; t < 32u; t += 2) TRIP(0, t);
  STAMP(2);
  { const unsigned eb = pb + sp * UNIT_B; E4(true, eb, 0, 1); E4(true, eb, 1, 0); E4(true, eb, 1, 1);
#pragma unroll
    for (int m = 0; m < 4; ++m) { u32x4 w_; PACK(w_, 0, 0, m); *(LAS u32x4*)(lds + sto + m * 8192) = w_; acc[0][0][m][0] = zero4(); acc[0][0][m][1] = zero4(); } }
  STAMP(3);
  cA = nA; cB = nB; tl = (unsigned)NK1 - 2u;
  nrec = 0;
  TRIP(1, 0u);
#pragma unroll 1
  for (t = 2; t < 2u + 2u * n1; t += 2) TRIP(0, t);
  STAMP(4);
  TRIP(2, t); t += 2;
  if ((__builtin_amdgcn_ballot_w64(fv >= call) & 15ull) != 15ull) wait_flags(0u);
  STAMP(5);
  asm volatile("" : "=v"(Ya), "=v"(Yb), "=v"(rs[0]), "=v"(rs[1]), "=v"(rs[2]), "=v"(rs[3]), "=v"(rs[4]), "=v"(rs[5]), "=v"(rs[6]), "=v"(rs[7]));
#pragma unroll 1
  for (int i = 0; i < 4; ++i) { TRIP(3, t); t += 2; }
  TRIP(4, t); t += 2;
  STAMP(6);
#pragma unroll 1
  for (; t < (unsigned)NK1; t += 2) TRIP(0, t);
  STAMP(7);
  { const unsigned eb = pb + ROUND_B + sp * UNIT_B; E4(false, eb, 0, 1); E4(false, eb, 1, 0); E4(false, eb, 1, 1); }
  if (wr == 0) BAR;
  STAMP(8);
  WAIT_V(0);
  STAMP(9);
  { u32x4 sc = srdC; const unsigned pubo = fb + 16u + sp * 4u; sc[2] = pubo + 4u; BST1("sc1", vcall, sc, vl4, pubo); }
  { u32x4 sc = srdC; const unsigned so = WS_CNT + (unsigned)(b * 8 + wid) * 4u; sc[2] = so + 4u; BST1("sc1", vcall, sc, vl4, so); }
  wait_flags(1u);
  STAMP(10);
  u32x4 R[4][3];
  { const unsigned lq = pb + ROUND_B + (sp << 15);
    const unsigned m1 = lq + ((sp + 1u) & 3u) * UNIT_B, m2 = lq + ((sp + 2u) & 3u) * UNIT_B, m3 = lq + ((sp + 3u) & 3u) * UNIT_B;
    BLD4(R[0][0], srdP, vl16, m1, 0); BLD4(R[0][1], srdP, vl16, m2, 0); BLD4(R[0][2], srdP, vl16, m3, 0);
    BLD4(R[1][0], srdP, vl16, m1, 8192); BLD4(R[1][1], srdP, vl16, m2, 8192); BLD4(R[1][2], srdP, vl16, m3, 8192);
    BLD4(R[2][0], srdP, vl16, m1, 16384); BLD4(R[2][1], srdP, vl16, m2, 16384); BLD4(R[2][2], srdP, vl16, m3, 16384);
    BLD4(R[3][0], srdP, vl16, m1, 24576); BLD4(R[3][1], srdP, vl16, m2, 24576); BLD4(R[3][2], srdP, vl16, m3, 24576); }
  STAMP(11);
  {
    const u32x4 so = {u1.olo, u1.ohi, OUT_BLK, 0x00020000u}; const unsigned o = u1.obase + qo;
#pragma unroll
    for (int m = 0; m < 4; ++m) {
      if (m == 0) WAIT_V(9); else if (m == 1) WAIT_V(7); else if (m == 2) WAIT_V(5); else WAIT_V(3);
      asm volatile("" : "+v"(R[m][0]), "+v"(R[m][1]), "+v"(R[m][2]));
      float v[8];
#pragma unroll
      for (int e = 0; e < 4; ++e) { auto r_ = __builtin_amdgcn_permlane16_swap(__float_as_uint(acc[0][0][m][0][e]), __float_as_uint(acc[0][0][m][1][e]), false, false); v[e] = __uint_as_float(r_[0]); v[4 + e] = __uint_as_float(r_[1]); }
      u32x4 w;
#pragma unroll
      for (int e = 0; e < 4; ++e) {
        const float lo_ = (v[2 * e] + blo(R[m][0][e])) + (blo(R[m][1][e]) + blo(R[m][2][e])), hi_ = (v[2 * e + 1] + bhi(R[m][0][e])) + (bhi(R[m][1][e]) + bhi(R[m][2][e]));
        w[e] = bad ? 0x7fc07fc0u : cvtpk(lo_, hi_); }
      if (m == 0) BST4("sc0 sc1", w, so, voffO, o, 0); else if (m == 1) BST4("sc0 sc1", w, so, voffO, o, 16 * NO * 2); else if (m == 2) BST4("sc0 sc1", w, so, voffO, o, 32 * NO * 2); else BST4("sc0 sc1", w, so, voffO, o, 48 * NO * 2);
    }
  }
  STAMP(13);
  WAIT_V(0);
  STAMP(14);
  MEMF; BAR; MEMF;
  if (wid == 0) {
    const unsigned pr = (unsigned)lane;
    const bool mine = pr < (unsigned)NR && pr != (unsigned)rank;
    if (mine) __hip_atomic_store((unsigned*)(peers.ws[pr] + WS_DONE) + rank * 256 + b, call, __ATOMIC_RELAXED, __HIP_MEMORY_SCOPE_SYSTEM);
    bool late = false;
    if (mine) {
      const unsigned* flag = (const unsigned*)(myws + WS_DONE) + pr * 256 + b;
      const unsigned long long t0 = __builtin_amdgcn_s_memrealtime();
      while (__hip_atomic_load(flag, __ATOMIC_RELAXED, __HIP_MEMORY_SCOPE_SYSTEM) < call) {
        if (__hip_atomic_load(tmo, __ATOMIC_RELAXED, __HIP_MEMORY_SCOPE_AGENT) == call || __builtin_amdgcn_s_memrealtime() - t0 > DEADLINE_TICKS) {
          __hip_atomic_store(tmo, call, __ATOMIC_RELAXED, __HIP_MEMORY_SCOPE_AGENT); late = true; break; }
        __builtin_amdgcn_s_sleep(SLP);
      }
    }
    __builtin_amdgcn_fence(__ATOMIC_ACQUIRE, "");
    if (late || __hip_atomic_load(tmo, __ATOMIC_RELAXED, __HIP_MEMORY_SCOPE_AGENT) == call) peers.out[rank][(size_t)rank * M * NO + (size_t)b] = (u16)0x7fc0;
  }
}

extern "C" void kernel_launch(int rank, int nranks, ncclComm_t comm, hipStream_t stream,
                              void* const* d_in, const int* in_sizes, int n_in,
                              void* d_out, int out_size, const int* in_dtypes, int out_dtype,
                              void* const* d_out_all, void* const* d_ws_all, size_t ws_bytes) {
  if (nranks != NR || (unsigned)rank >= (unsigned)NR || ws_bytes < WS_NEED) return;
  HIP_CHECK(hipFuncSetAttribute((const void*)gemm_a2a, hipFuncAttributeMaxDynamicSharedMemorySize, LDS_BYTES));
  Peers p;
  for (int i = 0; i < NR; ++i) { p.out[i] = (u16*)d_out_all[i]; p.ws[i] = (char*)d_ws_all[i]; }
  gemm_a2a<<<256, 512, LDS_BYTES, stream>>>((const u16*)d_in[0], (const u16*)d_in[1], p, rank);
  HIP_CHECK(hipGetLastError());
}
```

```cpp
#include <hip/hip_runtime.h>
#include <rccl/rccl.h>
#include <cstdio>

#define HIP_CHECK(expr) do { hipError_t _e = (expr); if (_e != hipSuccess) { \
    fprintf(stderr, "HIP error %s at %s:%d\n", hipGetErrorString(_e), __FILE__, __LINE__); \
    return; } } while (0)

#define LAS __attribute__((address_space(3)))
typedef unsigned short u16;
typedef short bf16x8 __attribute__((ext_vector_type(8)));
typedef short s16x4 __attribute__((ext_vector_type(4)));
typedef float f32x4 __attribute__((ext_vector_type(4)));
typedef float f32x2 __attribute__((ext_vector_type(2)));
typedef unsigned u32x4 __attribute__((ext_vector_type(4)));
typedef __bf16 bf16x2_t __attribute__((ext_vector_type(2)));
constexpr int M = 1024, N = 8192, K = 8192, NR = 8, NO = 1024;
constexpr unsigned long long DEADLINE_TICKS = 100000000ull;
constexpr unsigned WS_TMO = 0;
constexpr unsigned WS_CNT = 4096;
constexpr unsigned WS_DONE = 16384;
constexpr unsigned WS_PFLAG = 65536;
constexpr unsigned WS_CTRL_BYTES = 131072;
constexpr unsigned WS_PART = 1u << 18;
constexpr unsigned UNIT_B = 131072, ROUND_B = 256u * UNIT_B, PART_BYTES = 2u * ROUND_B;
constexpr size_t WS_NEED = (size_t)WS_PART + PART_BYTES;
constexpr unsigned OUT_BLK = (unsigned)M * NO * 2;
static_assert(OUT_BLK / 64 == 16u * NO * 2u, "");
constexpr int HTB = 16384, STASH = 8 * HTB, LDS_BYTES = 10 * HTB;
constexpr unsigned kA = 128, kB = 64 * N * 2, hA = 128 * K * 2, hB = 256;
constexpr int NK1 = 32;
#define SLP 4
#define NSTAG ((wid + j) & 7)

__device__ __forceinline__ unsigned cvtpk(float lo, float hi) { f32x2 v = {lo, hi}; bf16x2_t b = __builtin_convertvector(v, bf16x2_t); return __builtin_bit_cast(unsigned, b); }
__device__ __forceinline__ s16x4 trrd(LAS const char* p) { return __builtin_amdgcn_ds_read_tr16_b64_v4i16((LAS s16x4*)p); }
__device__ __forceinline__ unsigned rfl(unsigned x) { return __builtin_amdgcn_readfirstlane(x); }
#define WAIT_V(n) asm volatile("s_waitcnt vmcnt(%0)" :: "n"(n) : "memory")
#define WAIT_L(n) asm volatile("s_waitcnt lgkmcnt(%0)" :: "n"(n) : "memory")
#define BAR __builtin_amdgcn_s_barrier()
#define SCHED __builtin_amdgcn_sched_barrier(0)
#define MEMF asm volatile("" ::: "memory")
#define BLDS(srd, voff, soff, GOFF, ldsbase, LOFF) do { unsigned tmp_; asm volatile("s_add_u32 m0, %1, %2\n\ts_add_u32 %0, %3, %4\n\tbuffer_load_dwordx4 %5, %6, %0 offen lds" \
   : "=&s"(tmp_) : "s"(ldsbase), "i"(LOFF), "s"(soff), "i"(GOFF), "v"(voff), "s"(srd) : "memory", "scc"); } while (0)
#define BLD4(dst, srd, voff, soff, GOFF) do { unsigned tmp_; asm volatile("s_add_u32 %1, %2, %3\n\tbuffer_load_dwordx4 %0, %4, %5, %1 offen sc1" \
   : "=v"(dst), "=&s"(tmp_) : "s"(soff), "i"(GOFF), "v"(voff), "s"(srd) : "memory", "scc"); } while (0)
#define BLD1(dst, srd, voff, soff) asm volatile("buffer_load_dword %0, %1, %2, %3 offen sc1" : "=v"(dst) : "v"(voff), "s"(srd), "s"(soff) : "memory")
#define BST4(MOD, data, srd, voff, soff, GOFF) do { unsigned tmp_; asm volatile("s_add_u32 %0, %1, %2\n\tbuffer_store_dwordx4 %3, %4, %5, %0 offen " MOD "\n\ts_nop 1" \
   : "=&s"(tmp_) : "s"(soff), "i"(GOFF), "v"(data), "v"(voff), "s"(srd) : "memory", "scc"); } while (0)
#define BST1(MOD, data, srd, voff, soff) asm volatile("buffer_store_dword %0, %1, %2, %3 offen " MOD :: "v"(data), "v"(voff), "s"(srd), "s"(soff) : "memory")
__device__ __forceinline__ u32x4 make_srd(const void* p, unsigned bytes) {
  unsigned long long a = (unsigned long long)p;
  u32x4 r; r[0] = rfl((unsigned)a); r[1] = rfl((unsigned)(a >> 32) & 0xffffu); r[2] = bytes; r[3] = 0x00020000u; return r;
}
struct Peers { u16* out[NR]; char* ws[NR]; };
struct Unit { unsigned a, b, obase, olo, ohi; };

__device__ __forceinline__ Unit unit_of(int r, int x, int j, int rank, const Peers& peers) {
  const int h = x >> 2; int d, pnl, pm;
  if (r == 0) {
    if (j < 28) { d = 1 + (j >> 2); pnl = h; pm = j & 3; }
    else { const int e = h * 4 + j - 28; d = e < 7 ? 1 + e : 7; pm = e < 7 ? 0 : 1; pnl = 2; }
  } else if (h == 0) {
    if (j < 28) { d = 1 + (j >> 2); pnl = 3; pm = j & 3; } else { d = 0; pnl = 0; pm = j - 28; }
  } else {
    if (j < 18) { d = 1 + j / 3; pm = 1 + j % 3; pnl = 2; } else if (j < 20) { d = 7; pm = j - 16; pnl = 2; } else { d = 0; pnl = 1 + ((j - 20) >> 2); pm = j & 3; }
  }
  const unsigned dst = (unsigned)((rank + d) & (NR - 1));
  const unsigned long long o = (unsigned long long)peers.out[dst] + (unsigned long long)rank * OUT_BLK;
  Unit u; u.a = (unsigned)(pm * 256 * K * 2); u.b = (unsigned)(((int)dst * 4 + pnl) * 512); u.obase = (unsigned)(pm * 256 * NO * 2 + pnl * 512);
  u.olo = rfl((unsigned)o); u.ohi = rfl((unsigned)(o >> 32) & 0xffffu);
  return u;
}
__device__ __forceinline__ float blo(unsigned w) { return __uint_as_float(w << 16); }
__device__ __forceinline__ float bhi(unsigned w) { return __uint_as_float(w & 0xffff0000u); }
__device__ __forceinline__ f32x4 zero4() { f32x2 a, b; asm volatile("v_mov_b64 %0, 0" : "=v"(a)); asm volatile("v_mov_b64 %0, 0" : "=v"(b)); return __builtin_shufflevector(a, b, 0, 1, 2, 3); }
__device__ __forceinline__ float fadd1(float a, float b) { float r; asm("v_add_f32_e32 %0, %1, %2" : "=v"(r) : "v"(a), "v"(b)); return r; }

#define STAMP(i) do {} while (0)
__global__ __launch_bounds__(512, 2) void gemm_a2a(const u16* __restrict__ A, const u16* __restrict__ W, Peers peers, int rank) {
  extern __shared__ __attribute__((aligned(16))) char smem_[];
  LAS char* lds = (LAS char*)smem_;
  const int tid = threadIdx.x, wid = __builtin_amdgcn_readfirstlane(tid >> 6), lane = tid & 63, wr = wid >> 2, wc = wid & 3, fr = lane & 15, fq = lane >> 4;
  const int b = blockIdx.x, x = b & 7, j = b >> 3;
  char* myws = peers.ws[rank];
  unsigned tsv = 0; STAMP(0);
  const u32x4 srdA = make_srd(A, (unsigned)M * K * 2), srdB = make_srd(W, (unsigned)K * N * 2), srdC = make_srd(myws, WS_CTRL_BYTES), srdP = make_srd(myws, (unsigned)WS_NEED);
  const unsigned vl16 = (unsigned)lane * 16u, vl4 = (unsigned)lane * 4u;
  unsigned cntv; { const unsigned so = WS_CNT + (unsigned)(b * 8 + wid) * 4u; BLD1(cntv, srdC, vl4, so); }
  unsigned voffA, voffB;
  { const int row = wid * 8 + (lane >> 3); const int ch = (lane & 7) ^ ((row >> 1) & 7); voffA = (unsigned)(row * K + ch * 8) * 2u; }
  { const int P = wid * 8 + (lane >> 3); const int k = ((P >> 5) << 4) | ((P & 1) << 3) | ((P >> 2) & 7), n64 = (P >> 1) & 1;
    const int sg = ((lane & 7) >> 1) ^ (k & 3), ch = sg * 2 + (lane & 1); voffB = (unsigned)(k * N + n64 * 64 + ch * 8) * 2u; }
  const unsigned ldsw = rfl((unsigned)(size_t)lds + (unsigned)wid * 1024u);
  const int aoff0 = (wr * 64 + fr) * 128 + ((fq ^ (fr >> 1)) << 4), aoff1 = aoff0 ^ 64;
  const int q = fr >> 2, p = fr & 3;
  const int brest = (fq >> 1) * 4096 + q * 512 + (fq & 1) * 128 + 8 * p + (wc >> 1) * 256;
  const int boff0 = brest + ((((wc & 1) * 2 + 0) ^ q) << 5), boff1 = brest + ((((wc & 1) * 2 + 1) ^ q) << 5);
  const unsigned voffO = (unsigned)((wr * 64 + fr) * NO * 2 + (wc * 32 + (fq & 1) * 16 + (fq >> 1) * 8) * 2);
  const unsigned sp = (unsigned)x & 3u, oa = sp >> 1, ob = sp & 1u;
  const unsigned dA = oa ? 0u - hA : hA, dB = ob ? 0u - hB : hB;
  const Unit u0 = unit_of(0, x, j, rank, peers);
#define SA(b_, h) (((b_) * 2 + (h)) * HTB)
#define SB(b_, h) ((4 + (b_) * 2 + (h)) * HTB)
#define STAGE_A(bufoff, so) do { BLDS(srdA2, voffA, so, 0, ldsw, (bufoff)); BLDS(srdA2, voffA, so, 64 * K * 2, ldsw, (bufoff) + 8192); } while (0)
#define STAGE_B(bufoff, so) do { BLDS(srdB2, voffB, so, 0, ldsw, (bufoff)); BLDS(srdB2, voffB, so, 32 * N * 2, ldsw, (bufoff) + 8192); } while (0)
#define LDA2(dst_, b_, h, m0) do { _Pragma("unroll") for (int m = m0; m < m0 + 2; ++m) { dst_[m][0] = *(const LAS bf16x8*)(lds + SA(b_, h) + aoff0 + m * 2048); dst_[m][1] = *(const LAS bf16x8*)(lds + SA(b_, h) + aoff1 + m * 2048); } } while (0)
#define LDA(dst_, b_, h) do { LDA2(dst_, b_, h, 0); LDA2(dst_, b_, h, 2); } while (0)
#define LDB1(dst_, b_, h, boff, s) do { s16x4 lo_ = trrd(lds + SB(b_, h) + boff + (s) * 8192), hi_ = trrd(lds + SB(b_, h) + boff + (s) * 8192 + 2048); dst_ = __builtin_shufflevector(lo_, hi_, 0, 1, 2, 3, 4, 5, 6, 7); } while (0)
#define LDB(dst_, b_, h) do { LDB1(dst_[0][0], b_, h, boff0, 0); LDB1(dst_[0][1], b_, h, boff0, 1); LDB1(dst_[1][0], b_, h, boff1, 0); LDB1(dst_[1][1], b_, h, boff1, 1); } while (0)
#define MFM(i) ((i) >> 3)
#define MFN(i) ((MFM(i) & 1) ? 3 - (((i) >> 1) & 3) : (((i) >> 1) & 3))
#define MFK(i) (((i) ^ ((i) >> 1)) & 1)
#define MFA(ai, i) acc[ai][MFN(i) >> 1][MFM(i)][MFN(i) & 1] = __builtin_amdgcn_mfma_f32_16x16x32_bf16((MFN(i) >> 1) ? B1[MFN(i) & 1][MFK(i)] : B0[MFN(i) & 1][MFK(i)], At[MFM(i)][MFK(i)], acc[ai][MFN(i) >> 1][MFM(i)][MFN(i) & 1], 0, 0, 0)
#define MSEG(ai) do { _Pragma("unroll") for (int i_ = 0; i_ < 32; ++i_) { MFA(ai, i_); SCHED; } } while (0)
#define PRIO(n) __builtin_amdgcn_s_setprio(n)
#define PACK(w, ai, bj, m) do { const f32x4 x_ = acc[ai][bj][m][0], y_ = acc[ai][bj][m][1]; \
    auto r0_ = __builtin_amdgcn_permlane16_swap(cvtpk(x_[0], x_[1]), cvtpk(y_[0], y_[1]), false, false); \
    auto r1_ = __builtin_amdgcn_permlane16_swap(cvtpk(x_[2], x_[3]), cvtpk(y_[2], y_[3]), false, false); \
    w = (u32x4){r0_[0], r1_[0], r0_[1], r1_[1]}; } while (0)
#define E1(ZERO, eq, ai, bj, m) do { u32x4 w_; PACK(w_, ai, bj, m); BST4("sc1", w_, srdP, vl16, eq, (m) * 8192); if (ZERO) { acc[ai][bj][m][0] = zero4(); acc[ai][bj][m][1] = zero4(); } } while (0)
#define E4(ZERO, eb, ai, bj) do { const unsigned eq_ = (eb) + ((sp ^ (unsigned)((ai) * 2 + (bj))) << 15); E1(ZERO, eq_, ai, bj, 0); E1(ZERO, eq_, ai, bj, 1); E1(ZERO, eq_, ai, bj, 2); E1(ZERO, eq_, ai, bj, 3); } while (0)
  bf16x8 At[4][2], B0[2][2], B1[2][2];
  const u32x4 srdA2 = srdA, srdB2 = srdB;
  unsigned cA = u0.a + oa * hA + sp * 32u * kA, cB = u0.b + ob * hB + sp * 32u * kB;
  { const unsigned bh = cB + dB, ah = cA + dA; STAGE_B(SB(0, 0), cB); STAGE_B(SB(0, 1), bh); STAGE_A(SA(0, 0), cA); STAGE_A(SA(0, 1), ah); }
  const Unit u1 = unit_of(1, x, j, rank, peers);
  f32x4 acc[2][2][4][2];
#pragma unroll
  for (int a = 0; a < 2; ++a)
#pragma unroll
    for (int c = 0; c < 2; ++c)
#pragma unroll
      for (int m = 0; m < 4; ++m)
#pragma unroll
        for (int n = 0; n < 2; ++n) acc[a][c][m][n] = zero4();
  if (wr == 1) BAR;
  WAIT_V(2); BAR; MEMF;
  { const unsigned b1 = cB + kB, a1 = cA + kA, b1h = b1 + dB; STAGE_B(SB(1, 0), b1); STAGE_A(SA(1, 0), a1); STAGE_B(SB(1, 1), b1h); }
  asm volatile("" : "+v"(cntv));
  const unsigned call = rfl(cntv) + 1u;
  const unsigned vcall = call;
  WAIT_V(6); BAR; MEMF;
  STAMP(1);
  const unsigned T = ((unsigned)x >> 2) * 32u + (unsigned)j;
  const unsigned pb = WS_PART + T * 4u * UNIT_B + (unsigned)wid * 1024u;
  const unsigned fb = WS_PFLAG + (T * 8u + (unsigned)wid) * 32u;
  const unsigned qo = oa * (128u * NO * 2u) + ob * 256u;
  const unsigned n1 = (unsigned)(NSTAG);
  float rs[8]; u32x4 Xa, Xb, Ya, Yb, ow; unsigned fv;
  unsigned* tmo = (unsigned*)(myws + WS_TMO);
  unsigned bad = 0;
  unsigned nA = u1.a + oa * hA + sp * (unsigned)NK1 * kA, nB = u1.b + ob * hB + sp * (unsigned)NK1 * kB, tl = 30u;
  u32x4 srdO = {u0.olo, u0.ohi, 0u, 0x00020000u}, srdZ = srdC; srdZ[2] = 0;
  unsigned l1 = pb + (sp << 15) + ((sp + 1u) & 3u) * UNIT_B, l2 = pb + (sp << 15) + ((sp + 2u) & 3u) * UNIT_B, l3 = pb + (sp << 15) + ((sp + 3u) & 3u) * UNIT_B, oo = u0.obase + qo, nrec = ~0u;
  int sto = STASH + wid * 1024 + lane * 16;
#define MF(i) MFA(1, i)
#define UNP(L, e) (((e) & 1) ? bhi(L[(e) >> 1]) : blo(L[(e) >> 1]))
#define SP1LOAD(b_, abuf, aso, XA, XB) do { const u32x4 sa_ = (b_) ? srdA2 : srdA; LDB(B0, b_, 0); BLDS(sa_, voffA, aso, 0, ldsw, abuf); LDB(B1, b_, 1); BLDS(sa_, voffA, aso, 64 * K * 2, ldsw, (abuf) + 8192); \
    LDA2(At, b_, 0, 0); XA; LDA2(At, b_, 0, 2); XB; } while (0)
#define TRIP(KIND, t) do { \
    const bool last = (t) == tl; \
    const unsigned a1h = cA + ((t) + 1u) * kA + dA; \
    const unsigned a2 = last ? nA : cA + ((t) + 2u) * kA, b2 = last ? nB : cB + ((t) + 2u) * kB; \
    const unsigned a2h = a2 + dA, b2h = b2 + dB, a3 = a2 + kA, b3 = b2 + kB, b3h = b3 + dB; \
    u32x4 srdA2 = srdA, srdB2 = srdB; srdA2[2] &= last ? nrec : ~0u; srdB2[2] &= last ? nrec : ~0u; \
    if ((KIND) == 3) SP1LOAD(0, SA(1, 1), a1h, BLD4(Xa, srdP, vl16, l1, 0), BLD4(Xb, srdP, vl16, l2, 0)); \
    else if ((KIND) == 2) SP1LOAD(0, SA(1, 1), a1h, BLD1(fv, srdC, vl4, fb), (void)0); \
    else SP1LOAD(0, SA(1, 1), a1h, (void)0, (void)0); \
    WAIT_V((KIND) == 1 ? 20 : (KIND) == 2 ? 9 : (KIND) == 3 ? 12 : (KIND) == 4 ? 10 : 8); \
    WAIT_L(0); BAR; PRIO(1); MSEG(0); PRIO(0); BAR; MEMF; SCHED; \
    LDA(At, 0, 1); STAGE_B(SB(0, 0), b2); STAGE_B(SB(0, 1), b2h); STAGE_A(SA(0, 0), a2); \
    WAIT_V((KIND) == 1 ? 20 : (KIND) == 2 ? 9 : (KIND) == 3 ? 10 : 8); \
    if ((KIND) >= 3) asm volatile("" : "+v"(Ya)); \
    WAIT_L(0); BAR; SCHED; PRIO(1); \
    if ((KIND) >= 3) {   \
      float tp = 0.f; \
      _Pragma("unroll") for (int e = 0; e < 8; ++e) { float q0, q1; \
        MF(4 * e); q0 = UNP(Ya, e); SCHED; MF(4 * e + 1); q1 = UNP(Yb, e); SCHED; MF(4 * e + 2); q0 = fadd1(q0, q1); SCHED; \
        MF(4 * e + 3); q0 = fadd1(rs[e], q0); if (e & 1) ow[e >> 1] = cvtpk(tp, q0); tp = q0; SCHED; } \
    } else MSEG(1); \
    PRIO(0); BAR; MEMF; SCHED; \
    if ((KIND) == 3) { Yb = *(const LAS u32x4*)(lds + sto); SP1LOAD(1, SA(0, 1), a2h, BLD4(Ya, srdP, vl16, l3, 0), BST4("sc0 sc1", ow, srdO, voffO, oo, 0)); } \
    else if ((KIND) == 4) SP1LOAD(1, SA(0, 1), a2h, (void)0, BST4("sc0 sc1", ow, srdO, voffO, oo, 0)); \
    else if ((KIND) == 2) SP1LOAD(1, SA(0, 1), a2h, BST1("", vcall, srdZ, vl4, fb), BST1("", vcall, srdZ, vl4, fb));     \
    else SP1LOAD(1, SA(0, 1), a2h, (void)0, (void)0); \
    WAIT_V((KIND) == 2 ? 11 : (KIND) == 3 ? 12 : (KIND) == 4 ? 9 : 8); \
    if ((KIND) == 1) { const unsigned pubo = fb + sp * 4u; u32x4 sc = srdC; sc[2] = pubo + 4u; BST1("sc1", vcall, sc, vl4, pubo); }     \
    WAIT_L(0); BAR; PRIO(1); MSEG(0); PRIO(0); BAR; MEMF; SCHED; \
    LDA(At, 1, 1); STAGE_B(SB(1, 0), b3); STAGE_B(SB(1, 1), b3h); STAGE_A(SA(1, 0), a3); \
    WAIT_V((KIND) == 1 || (KIND) == 4 ? 9 : (KIND) == 2 || (KIND) == 3 ? 10 : 8); \
    if ((KIND) == 3) asm volatile("" : "+v"(Xa), "+v"(Xb)); \
    if ((KIND) == 2) asm volatile("" : "+v"(fv)); \
    WAIT_L(0); BAR; SCHED; PRIO(1); \
    if ((KIND) == 3) {   \
      _Pragma("unroll") for (int e = 0; e < 8; ++e) { float q0, q1; \
        MF(4 * e); q0 = UNP(Xa, e); SCHED; MF(4 * e + 1); q1 = UNP(Xb, e); SCHED; MF(4 * e + 2); rs[e] = fadd1(q0, q1); SCHED; MF(4 * e + 3); SCHED; } \
      l1 += 8192u; l2 += 8192u; l3 += 8192u; sto += 8192; oo += srdO[2] >> 6; srdO[2] = OUT_BLK;     \
    } else MSEG(1); \
    PRIO(0); BAR; MEMF; SCHED; \
  } while (0)
  auto wait_flags = [&](unsigned f) {
    const unsigned long long t0 = __builtin_amdgcn_s_memrealtime();
    for (;;) {
      unsigned v; BLD1(v, srdC, vl4, fb); WAIT_V(0); asm volatile("" : "+v"(v));
      if (((__builtin_amdgcn_ballot_w64(v >= call) >> (4u * f)) & 15ull) == 15ull) break;
      if (rfl(__hip_atomic_load(tmo, __ATOMIC_RELAXED, __HIP_MEMORY_SCOPE_AGENT)) == call || __builtin_amdgcn_s_memrealtime() - t0 > DEADLINE_TICKS) {
        __hip_atomic_store(tmo, call, __ATOMIC_RELAXED, __HIP_MEMORY_SCOPE_AGENT); bad = 1; break; }
      __builtin_amdgcn_s_sleep(SLP);
    }
    bad = rfl(bad);
  };
  unsigned t = 0;
#pragma unroll 1
  for (; t < 32u; t += 2) TRIP(0, t);
  STAMP(2);
  { const unsigned eb = pb + sp * UNIT_B; E4(true, eb, 0, 1); E4(true, eb, 1, 0); E4(true, eb, 1, 1);
#pragma unroll
    for (int m = 0; m < 4; ++m) { u32x4 w_; PACK(w_, 0, 0, m); *(LAS u32x4*)(lds + sto + m * 8192) = w_; acc[0][0][m][0] = zero4(); acc[0][0][m][1] = zero4(); } }
  STAMP(3);
  cA = nA; cB = nB; tl = (unsigned)NK1 - 2u;
  nrec = 0;
  TRIP(1, 0u);
#pragma unroll 1
  for (t = 2; t < 2u + 2u * n1; t += 2) TRIP(0, t);
  STAMP(4);
  TRIP(2, t); t += 2;
  if ((__builtin_amdgcn_ballot_w64(fv >= call) & 15ull) != 15ull) wait_flags(0u);
  STAMP(5);
  asm volatile("" : "=v"(Ya), "=v"(Yb), "=v"(rs[0]), "=v"(rs[1]), "=v"(rs[2]), "=v"(rs[3]), "=v"(rs[4]), "=v"(rs[5]), "=v"(rs[6]), "=v"(rs[7]));
#pragma unroll 1
  for (int i = 0; i < 4; ++i) { TRIP(3, t); t += 2; }
  TRIP(4, t); t += 2;
  STAMP(6);
#pragma unroll 1
  for (; t < (unsigned)NK1; t += 2) TRIP(0, t);
  STAMP(7);
  { const unsigned eb = pb + ROUND_B + sp * UNIT_B; E4(false, eb, 0, 1); E4(false, eb, 1, 0); E4(false, eb, 1, 1); }
  if (wr == 0) BAR;
  STAMP(8);
  WAIT_V(0);
  STAMP(9);
  { u32x4 sc = srdC; const unsigned pubo = fb + 16u + sp * 4u; sc[2] = pubo + 4u; BST1("sc1", vcall, sc, vl4, pubo); }
  { u32x4 sc = srdC; const unsigned so = WS_CNT + (unsigned)(b * 8 + wid) * 4u; sc[2] = so + 4u; BST1("sc1", vcall, sc, vl4, so); }
  wait_flags(1u);
  STAMP(10);
  u32x4 R[4][3];
  { const unsigned lq = pb + ROUND_B + (sp << 15);
    const unsigned m1 = lq + ((sp + 1u) & 3u) * UNIT_B, m2 = lq + ((sp + 2u) & 3u) * UNIT_B, m3 = lq + ((sp + 3u) & 3u) * UNIT_B;
    BLD4(R[0][0], srdP, vl16, m1, 0); BLD4(R[0][1], srdP, vl16, m2, 0); BLD4(R[0][2], srdP, vl16, m3, 0);
    BLD4(R[1][0], srdP, vl16, m1, 8192); BLD4(R[1][1], srdP, vl16, m2, 8192); BLD4(R[1][2], srdP, vl16, m3, 8192);
    BLD4(R[2][0], srdP, vl16, m1, 16384); BLD4(R[2][1], srdP, vl16, m2, 16384); BLD4(R[2][2], srdP, vl16, m3, 16384);
    BLD4(R[3][0], srdP, vl16, m1, 24576); BLD4(R[3][1], srdP, vl16, m2, 24576); BLD4(R[3][2], srdP, vl16, m3, 24576); }
  STAMP(11);
  {
    const u32x4 so = {u1.olo, u1.ohi, OUT_BLK, 0x00020000u}; const unsigned o = u1.obase + qo;
#pragma unroll
    for (int m = 0; m < 4; ++m) {
      if (m == 0) WAIT_V(9); else if (m == 1) WAIT_V(7); else if (m == 2) WAIT_V(5); else WAIT_V(3);
      asm volatile("" : "+v"(R[m][0]), "+v"(R[m][1]), "+v"(R[m][2]));
      float v[8];
#pragma unroll
      for (int e = 0; e < 4; ++e) { auto r_ = __builtin_amdgcn_permlane16_swap(__float_as_uint(acc[0][0][m][0][e]), __float_as_uint(acc[0][0][m][1][e]), false, false); v[e] = __uint_as_float(r_[0]); v[4 + e] = __uint_as_float(r_[1]); }
      u32x4 w;
#pragma unroll
      for (int e = 0; e < 4; ++e) {
        const float lo_ = (v[2 * e] + blo(R[m][0][e])) + (blo(R[m][1][e]) + blo(R[m][2][e])), hi_ = (v[2 * e + 1] + bhi(R[m][0][e])) + (bhi(R[m][1][e]) + bhi(R[m][2][e]));
        w[e] = bad ? 0x7fc07fc0u : cvtpk(lo_, hi_); }
      if (m == 0) BST4("sc0 sc1", w, so, voffO, o, 0); else if (m == 1) BST4("sc0 sc1", w, so, voffO, o, 16 * NO * 2); else if (m == 2) BST4("sc0 sc1", w, so, voffO, o, 32 * NO * 2); else BST4("sc0 sc1", w, so, voffO, o, 48 * NO * 2);
    }
  }
  STAMP(13);
  WAIT_V(0);
  STAMP(14);
  MEMF; BAR; MEMF;
  if (wid == 0) {
    const unsigned pr = (unsigned)lane;
    const bool mine = pr < (unsigned)NR && pr != (unsigned)rank;
    if (mine) __hip_atomic_store((unsigned*)(peers.ws[pr] + WS_DONE) + rank * 256 + b, call, __ATOMIC_RELAXED, __HIP_MEMORY_SCOPE_SYSTEM);
    bool late = false;
    if (mine) {
      const unsigned* flag = (const unsigned*)(myws + WS_DONE) + pr * 256 + b;
      const unsigned long long t0 = __builtin_amdgcn_s_memrealtime();
      while (__hip_atomic_load(flag, __ATOMIC_RELAXED, __HIP_MEMORY_SCOPE_SYSTEM) < call) {
        if (__hip_atomic_load(tmo, __ATOMIC_RELAXED, __HIP_MEMORY_SCOPE_AGENT) == call || __builtin_amdgcn_s_memrealtime() - t0 > DEADLINE_TICKS) {
          __hip_atomic_store(tmo, call, __ATOMIC_RELAXED, __HIP_MEMORY_SCOPE_AGENT); late = true; break; }
        __builtin_amdgcn_s_sleep(SLP);
      }
    }
    __builtin_amdgcn_fence(__ATOMIC_ACQUIRE, "");
    if (late || __hip_atomic_load(tmo, __ATOMIC_RELAXED, __HIP_MEMORY_SCOPE_AGENT) == call) peers.out[rank][(size_t)rank * M * NO + (size_t)b] = (u16)0x7fc0;
  }
}

extern "C" void kernel_launch(int rank, int nranks, ncclComm_t comm, hipStream_t stream,
                              void* const* d_in, const int* in_sizes, int n_in,
                              void* d_out, int out_size, const int* in_dtypes, int out_dtype,
                              void* const* d_out_all, void* const* d_ws_all, size_t ws_bytes) {
  if (nranks != NR || (unsigned)rank >= (unsigned)NR || ws_bytes < WS_NEED) return;
  HIP_CHECK(hipFuncSetAttribute((const void*)gemm_a2a, hipFuncAttributeMaxDynamicSharedMemorySize, LDS_BYTES));
  Peers p;
  for (int i = 0; i < NR; ++i) { p.out[i] = (u16*)d_out_all[i]; p.ws[i] = (char*)d_ws_all[i]; }
  gemm_a2a<<<256, 512, LDS_BYTES, stream>>>((const u16*)d_in[0], (const u16*)d_in[1], p, rank);
  HIP_CHECK(hipGetLastError());
}
```

```cpp
#include <hip/hip_runtime.h>
#include <rccl/rccl.h>
#include <cstdio>

#define HIP_CHECK(expr) do { hipError_t _e = (expr); if (_e != hipSuccess) { \
    fprintf(stderr, "HIP error %s at %s:%d\n", hipGetErrorString(_e), __FILE__, __LINE__); \
    return; } } while (0)

#define LAS __attribute__((address_space(3)))
typedef unsigned short u16;
typedef short bf16x8 __attribute__((ext_vector_type(8)));
typedef short s16x4 __attribute__((ext_vector_type(4)));
typedef float f32x4 __attribute__((ext_vector_type(4)));
typedef float f32x2 __attribute__((ext_vector_type(2)));
typedef unsigned u32x4 __attribute__((ext_vector_type(4)));
typedef __bf16 bf16x2_t __attribute__((ext_vector_type(2)));
constexpr int M = 1024, N = 8192, K = 8192, NR = 8, NO = 1024;
constexpr unsigned long long DEADLINE_TICKS = 100000000ull;
constexpr unsigned WS_TMO = 0;
constexpr unsigned WS_CNT = 4096;
constexpr unsigned WS_DONE = 16384;
constexpr unsigned WS_PFLAG = 65536;
constexpr unsigned WS_CTRL_BYTES = 131072;
constexpr unsigned WS_PART = 1u << 18;
constexpr unsigned UNIT_B = 131072, ROUND_B = 256u * UNIT_B, PART_BYTES = 2u * ROUND_B;
constexpr size_t WS_NEED = (size_t)WS_PART + PART_BYTES;
constexpr unsigned OUT_BLK = (unsigned)M * NO * 2;
static_assert(OUT_BLK / 64 == 16u * NO * 2u, "");
constexpr int HTB = 16384, STASH = 8 * HTB, LDS_BYTES = 10 * HTB;
constexpr unsigned kA = 128, kB = 64 * N * 2, hA = 128 * K * 2, hB = 256;
constexpr int NK1 = 32;
#define SLP 2
#define NSTAG ((wid + j) & 7)

__device__ __forceinline__ unsigned cvtpk(float lo, float hi) { f32x2 v = {lo, hi}; bf16x2_t b = __builtin_convertvector(v, bf16x2_t); return __builtin_bit_cast(unsigned, b); }
__device__ __forceinline__ s16x4 trrd(LAS const char* p) { return __builtin_amdgcn_ds_read_tr16_b64_v4i16((LAS s16x4*)p); }
__device__ __forceinline__ unsigned rfl(unsigned x) { return __builtin_amdgcn_readfirstlane(x); }
#define WAIT_V(n) asm volatile("s_waitcnt vmcnt(%0)" :: "n"(n) : "memory")
#define WAIT_L(n) asm volatile("s_waitcnt lgkmcnt(%0)" :: "n"(n) : "memory")
#define BAR __builtin_amdgcn_s_barrier()
#define SCHED __builtin_amdgcn_sched_barrier(0)
#define MEMF asm volatile("" ::: "memory")
#define BLDS(srd, voff, soff, GOFF, ldsbase, LOFF) do { unsigned tmp_; asm volatile("s_add_u32 m0, %1, %2\n\ts_add_u32 %0, %3, %4\n\tbuffer_load_dwordx4 %5, %6, %0 offen lds" \
   : "=&s"(tmp_) : "s"(ldsbase), "i"(LOFF), "s"(soff), "i"(GOFF), "v"(voff), "s"(srd) : "memory", "scc"); } while (0)
#define BLD4(dst, srd, voff, soff, GOFF) do { unsigned tmp_; asm volatile("s_add_u32 %1, %2, %3\n\tbuffer_load_dwordx4 %0, %4, %5, %1 offen sc1" \
   : "=v"(dst), "=&s"(tmp_) : "s"(soff), "i"(GOFF), "v"(voff), "s"(srd) : "memory", "scc"); } while (0)
#define BLD1(dst, srd, voff, soff) asm volatile("buffer_load_dword %0, %1, %2, %3 offen sc1" : "=v"(dst) : "v"(voff), "s"(srd), "s"(soff) : "memory")
#define BST4(MOD, data, srd, voff, soff, GOFF) do { unsigned tmp_; asm volatile("s_add_u32 %0, %1, %2\n\tbuffer_store_dwordx4 %3, %4, %5, %0 offen " MOD "\n\ts_nop 1" \
   : "=&s"(tmp_) : "s"(soff), "i"(GOFF), "v"(data), "v"(voff), "s"(srd) : "memory", "scc"); } while (0)
#define BST1(MOD, data, srd, voff, soff) asm volatile("buffer_store_dword %0, %1, %2, %3 offen " MOD :: "v"(data), "v"(voff), "s"(srd), "s"(soff) : "memory")
__device__ __forceinline__ u32x4 make_srd(const void* p, unsigned bytes) {
  unsigned long long a = (unsigned long long)p;
  u32x4 r; r[0] = rfl((unsigned)a); r[1] = rfl((unsigned)(a >> 32) & 0xffffu); r[2] = bytes; r[3] = 0x00020000u; return r;
}
struct Peers { u16* out[NR]; char* ws[NR]; };
struct Unit { unsigned a, b, obase, olo, ohi; };

__device__ __forceinline__ Unit unit_of(int r, int x, int j, int rank, const Peers& peers) {
  const int h = x >> 2; int d, pnl, pm;
  if (r == 0) {
    if (j < 28) { d = 1 + (j >> 2); pnl = h; pm = j & 3; }
    else { const int e = h * 4 + j - 28; d = e < 7 ? 1 + e : 7; pm = e < 7 ? 0 : 1; pnl = 2; }
  } else if (h == 0) {
    if (j < 28) { d = 1 + (j >> 2); pnl = 3; pm = j & 3; } else { d = 0; pnl = 0; pm = j - 28; }
  } else {
    if (j < 18) { d = 1 + j / 3; pm = 1 + j % 3; pnl = 2; } else if (j < 20) { d = 7; pm = j - 16; pnl = 2; } else { d = 0; pnl = 1 + ((j - 20) >> 2); pm = j & 3; }
  }
  const unsigned dst = (unsigned)((rank + d) & (NR - 1));
  const unsigned long long o = (unsigned long long)peers.out[dst] + (unsigned long long)rank * OUT_BLK;
  Unit u; u.a = (unsigned)(pm * 256 * K * 2); u.b = (unsigned)(((int)dst * 4 + pnl) * 512); u.obase = (unsigned)(pm * 256 * NO * 2 + pnl * 512);
  u.olo = rfl((unsigned)o); u.ohi = rfl((unsigned)(o >> 32) & 0xffffu);
  return u;
}
__device__ __forceinline__ float blo(unsigned w) { return __uint_as_float(w << 16); }
__device__ __forceinline__ float bhi(unsigned w) { return __uint_as_float(w & 0xffff0000u); }
__device__ __forceinline__ f32x4 zero4() { f32x2 a, b; asm volatile("v_mov_b64 %0, 0" : "=v"(a)); asm volatile("v_mov_b64 %0, 0" : "=v"(b)); return __builtin_shufflevector(a, b, 0, 1, 2, 3); }
__device__ __forceinline__ float fadd1(float a, float b) { float r; asm("v_add_f32_e32 %0, %1, %2" : "=v"(r) : "v"(a), "v"(b)); return r; }

#define STAMP(i) do {} while (0)
__global__ __launch_bounds__(512, 2) void gemm_a2a(const u16* __restrict__ A, const u16* __restrict__ W, Peers peers, int rank) {
  extern __shared__ __attribute__((aligned(16))) char smem_[];
  LAS char* lds = (LAS char*)smem_;
  const int tid = threadIdx.x, wid = __builtin_amdgcn_readfirstlane(tid >> 6), lane = tid & 63, wr = wid >> 2, wc = wid & 3, fr = lane & 15, fq = lane >> 4;
  const int b = blockIdx.x, x = b & 7, j = b >> 3;
  char* myws = peers.ws[rank];
  unsigned tsv = 0; STAMP(0);
  const u32x4 srdA = make_srd(A, (unsigned)M * K * 2), srdB = make_srd(W, (unsigned)K * N * 2), srdC = make_srd(myws, WS_CTRL_BYTES), srdP = make_srd(myws, (unsigned)WS_NEED);
  const unsigned vl16 = (unsigned)lane * 16u, vl4 = (unsigned)lane * 4u;
  unsigned cntv; { const unsigned so = WS_CNT + (unsigned)(b * 8 + wid) * 4u; BLD1(cntv, srdC, vl4, so); }
  unsigned voffA, voffB;
  { const int row = wid * 8 + (lane >> 3); const int ch = (lane & 7) ^ ((row >> 1) & 7); voffA = (unsigned)(row * K + ch * 8) * 2u; }
  { const int P = wid * 8 + (lane >> 3); const int k = ((P >> 5) << 4) | ((P & 1) << 3) | ((P >> 2) & 7), n64 = (P >> 1) & 1;
    const int sg = ((lane & 7) >> 1) ^ (k & 3), ch = sg * 2 + (lane & 1); voffB = (unsigned)(k * N + n64 * 64 + ch * 8) * 2u; }
  const unsigned ldsw = rfl((unsigned)(size_t)lds + (unsigned)wid * 1024u);
  const int aoff0 = (wr * 64 + fr) * 128 + ((fq ^ (fr >> 1)) << 4), aoff1 = aoff0 ^ 64;
  const int q = fr >> 2, p = fr & 3;
  const int brest = (fq >> 1) * 4096 + q * 512 + (fq & 1) * 128 + 8 * p + (wc >> 1) * 256;
  const int boff0 = brest + ((((wc & 1) * 2 + 0) ^ q) << 5), boff1 = brest + ((((wc & 1) * 2 + 1) ^ q) << 5);
  const unsigned voffO = (unsigned)((wr * 64 + fr) * NO * 2 + (wc * 32 + (fq & 1) * 16 + (fq >> 1) * 8) * 2);
  const unsigned sp = (unsigned)x & 3u, oa = sp >> 1, ob = sp & 1u;
  const unsigned dA = oa ? 0u - hA : hA, dB = ob ? 0u - hB : hB;
  const Unit u0 = unit_of(0, x, j, rank, peers);
#define SA(b_, h) (((b_) * 2 + (h)) * HTB)
#define SB(b_, h) ((4 + (b_) * 2 + (h)) * HTB)
#define STAGE_A(bufoff, so) do { BLDS(srdA2, voffA, so, 0, ldsw, (bufoff)); BLDS(srdA2, voffA, so, 64 * K * 2, ldsw, (bufoff) + 8192); } while (0)
#define STAGE_B(bufoff, so) do { BLDS(srdB2, voffB, so, 0, ldsw, (bufoff)); BLDS(srdB2, voffB, so, 32 * N * 2, ldsw, (bufoff) + 8192); } while (0)
#define LDA2(dst_, b_, h, m0) do { _Pragma("unroll") for (int m = m0; m < m0 + 2; ++m) { dst_[m][0] = *(const LAS bf16x8*)(lds + SA(b_, h) + aoff0 + m * 2048); dst_[m][1] = *(const LAS bf16x8*)(lds + SA(b_, h) + aoff1 + m * 2048); } } while (0)
#define LDA(dst_, b_, h) do { LDA2(dst_, b_, h, 0); LDA2(dst_, b_, h, 2); } while (0)
#define LDB1(dst_, b_, h, boff, s) do { s16x4 lo_ = trrd(lds + SB(b_, h) + boff + (s) * 8192), hi_ = trrd(lds + SB(b_, h) + boff + (s) * 8192 + 2048); dst_ = __builtin_shufflevector(lo_, hi_, 0, 1, 2, 3, 4, 5, 6, 7); } while (0)
#define LDB(dst_, b_, h) do { LDB1(dst_[0][0], b_, h, boff0, 0); LDB1(dst_[0][1], b_, h, boff0, 1); LDB1(dst_[1][0], b_, h, boff1, 0); LDB1(dst_[1][1], b_, h, boff1, 1); } while (0)
#define MFM(i) ((i) >> 3)
#define MFN(i) ((MFM(i) & 1) ? 3 - (((i) >> 1) & 3) : (((i) >> 1) & 3))
#define MFK(i) (((i) ^ ((i) >> 1)) & 1)
#define MFA(ai, i) acc[ai][MFN(i) >> 1][MFM(i)][MFN(i) & 1] = __builtin_amdgcn_mfma_f32_16x16x32_bf16((MFN(i) >> 1) ? B1[MFN(i) & 1][MFK(i)] : B0[MFN(i) & 1][MFK(i)], At[MFM(i)][MFK(i)], acc[ai][MFN(i) >> 1][MFM(i)][MFN(i) & 1], 0, 0, 0)
#define MSEG(ai) do { _Pragma("unroll") for (int i_ = 0; i_ < 32; ++i_) { MFA(ai, i_); SCHED; } } while (0)
#define PRIO(n) __builtin_amdgcn_s_setprio(n)
#define PACK(w, ai, bj, m) do { const f32x4 x_ = acc[ai][bj][m][0], y_ = acc[ai][bj][m][1]; \
    auto r0_ = __builtin_amdgcn_permlane16_swap(cvtpk(x_[0], x_[1]), cvtpk(y_[0], y_[1]), false, false); \
    auto r1_ = __builtin_amdgcn_permlane16_swap(cvtpk(x_[2], x_[3]), cvtpk(y_[2], y_[3]), false, false); \
    w = (u32x4){r0_[0], r1_[0], r0_[1], r1_[1]}; } while (0)
#define E1(ZERO, eq, ai, bj, m) do { u32x4 w_; PACK(w_, ai, bj, m); BST4("sc1", w_, srdP, vl16, eq, (m) * 8192); if (ZERO) { acc[ai][bj][m][0] = zero4(); acc[ai][bj][m][1] = zero4(); } } while (0)
#define E4(ZERO, eb, ai, bj) do { const unsigned eq_ = (eb) + ((sp ^ (unsigned)((ai) * 2 + (bj))) << 15); E1(ZERO, eq_, ai, bj, 0); E1(ZERO, eq_, ai, bj, 1); E1(ZERO, eq_, ai, bj, 2); E1(ZERO, eq_, ai, bj, 3); } while (0)
  bf16x8 At[4][2], B0[2][2], B1[2][2];
  const u32x4 srdA2 = srdA, srdB2 = srdB;
  unsigned cA = u0.a + oa * hA + sp * 32u * kA, cB = u0.b + ob * hB + sp * 32u * kB;
  { const unsigned bh = cB + dB, ah = cA + dA; STAGE_B(SB(0, 0), cB); STAGE_B(SB(0, 1), bh); STAGE_A(SA(0, 0), cA); STAGE_A(SA(0, 1), ah); }
  const Unit u1 = unit_of(1, x, j, rank, peers);
  f32x4 acc[2][2][4][2];
#pragma unroll
  for (int a = 0; a < 2; ++a)
#pragma unroll
    for (int c = 0; c < 2; ++c)
#pragma unroll
      for (int m = 0; m < 4; ++m)
#pragma unroll
        for (int n = 0; n < 2; ++n) acc[a][c][m][n] = zero4();
  if (wr == 1) BAR;
  WAIT_V(2); BAR; MEMF;
  { const unsigned b1 = cB + kB, a1 = cA + kA, b1h = b1 + dB; STAGE_B(SB(1, 0), b1); STAGE_A(SA(1, 0), a1); STAGE_B(SB(1, 1), b1h); }
  asm volatile("" : "+v"(cntv));
  const unsigned call = rfl(cntv) + 1u;
  const unsigned vcall = call;
  WAIT_V(6); BAR; MEMF;
  STAMP(1);
  const unsigned T = ((unsigned)x >> 2) * 32u + (unsigned)j;
  const unsigned pb = WS_PART + T * 4u * UNIT_B + (unsigned)wid * 1024u;
  const unsigned fb = WS_PFLAG + (T * 8u + (unsigned)wid) * 32u;
  const unsigned qo = oa * (128u * NO * 2u) + ob * 256u;
  const unsigned n1 = (unsigned)(NSTAG);
  float rs[8]; u32x4 Xa, Xb, Ya, Yb, ow; unsigned fv;
  unsigned* tmo = (unsigned*)(myws + WS_TMO);
  unsigned bad = 0;
  unsigned nA = u1.a + oa * hA + sp * (unsigned)NK1 * kA, nB = u1.b + ob * hB + sp * (unsigned)NK1 * kB, tl = 30u;
  u32x4 srdO = {u0.olo, u0.ohi, 0u, 0x00020000u}, srdZ = srdC; srdZ[2] = 0;
  unsigned l1 = pb + (sp << 15) + ((sp + 1u) & 3u) * UNIT_B, l2 = pb + (sp << 15) + ((sp + 2u) & 3u) * UNIT_B, l3 = pb + (sp << 15) + ((sp + 3u) & 3u) * UNIT_B, oo = u0.obase + qo, nrec = ~0u;
  int sto = STASH + wid * 1024 + lane * 16;
#define MF(i) MFA(1, i)
#define UNP(L, e) (((e) & 1) ? bhi(L[(e) >> 1]) : blo(L[(e) >> 1]))
#define SP1LOAD(b_, abuf, aso, XA, XB) do { const u32x4 sa_ = (b_) ? srdA2 : srdA; LDB(B0, b_, 0); BLDS(sa_, voffA, aso, 0, ldsw, abuf); LDB(B1, b_, 1); BLDS(sa_, voffA, aso, 64 * K * 2, ldsw, (abuf) + 8192); \
    LDA2(At, b_, 0, 0); XA; LDA2(At, b_, 0, 2); XB; } while (0)
#define TRIP(KIND, t) do { \
    const bool last = (t) == tl; \
    const unsigned a1h = cA + ((t) + 1u) * kA + dA; \
    const unsigned a2 = last ? nA : cA + ((t) + 2u) * kA, b2 = last ? nB : cB + ((t) + 2u) * kB; \
    const unsigned a2h = a2 + dA, b2h = b2 + dB, a3 = a2 + kA, b3 = b2 + kB, b3h = b3 + dB; \
    u32x4 srdA2 = srdA, srdB2 = srdB; srdA2[2] &= last ? nrec : ~0u; srdB2[2] &= last ? nrec : ~0u; \
    if ((KIND) == 3) SP1LOAD(0, SA(1, 1), a1h, BLD4(Xa, srdP, vl16, l1, 0), BLD4(Xb, srdP, vl16, l2, 0)); \
    else if ((KIND) == 2) SP1LOAD(0, SA(1, 1), a1h, BLD1(fv, srdC, vl4, fb), (void)0); \
    else SP1LOAD(0, SA(1, 1), a1h, (void)0, (void)0); \
    WAIT_V((KIND) == 1 ? 20 : (KIND) == 2 ? 9 : (KIND) == 3 ? 12 : (KIND) == 4 ? 10 : 8); \
    WAIT_L(0); BAR; PRIO(1); MSEG(0); PRIO(0); BAR; MEMF; SCHED; \
    LDA(At, 0, 1); STAGE_B(SB(0, 0), b2); STAGE_B(SB(0, 1), b2h); STAGE_A(SA(0, 0), a2); \
    WAIT_V((KIND) == 1 ? 20 : (KIND) == 2 ? 9 : (KIND) == 3 ? 10 : 8); \
    if ((KIND) >= 3) asm volatile("" : "+v"(Ya)); \
    WAIT_L(0); BAR; SCHED; PRIO(1); \
    if ((KIND) >= 3) {   \
      float tp = 0.f; \
      _Pragma("unroll") for (int e = 0; e < 8; ++e) { float q0, q1; \
        MF(4 * e); q0 = UNP(Ya, e); SCHED; MF(4 * e + 1); q1 = UNP(Yb, e); SCHED; MF(4 * e + 2); q0 = fadd1(q0, q1); SCHED; \
        MF(4 * e + 3); q0 = fadd1(rs[e], q0); if (e & 1) ow[e >> 1] = cvtpk(tp, q0); tp = q0; SCHED; } \
    } else MSEG(1); \
    PRIO(0); BAR; MEMF; SCHED; \
    if ((KIND) == 3) { Yb = *(const LAS u32x4*)(lds + sto); SP1LOAD(1, SA(0, 1), a2h, BLD4(Ya, srdP, vl16, l3, 0), BST4("sc0 sc1", ow, srdO, voffO, oo, 0)); } \
    else if ((KIND) == 4) SP1LOAD(1, SA(0, 1), a2h, (void)0, BST4("sc0 sc1", ow, srdO, voffO, oo, 0)); \
    else if ((KIND) == 2) SP1LOAD(1, SA(0, 1), a2h, BST1("", vcall, srdZ, vl4, fb), BST1("", vcall, srdZ, vl4, fb));     \
    else SP1LOAD(1, SA(0, 1), a2h, (void)0, (void)0); \
    WAIT_V((KIND) == 2 ? 11 : (KIND) == 3 ? 12 : (KIND) == 4 ? 9 : 8); \
    if ((KIND) == 1) { const unsigned pubo = fb + sp * 4u; u32x4 sc = srdC; sc[2] = pubo + 4u; BST1("sc1", vcall, sc, vl4, pubo); }     \
    WAIT_L(0); BAR; PRIO(1); MSEG(0); PRIO(0); BAR; MEMF; SCHED; \
    LDA(At, 1, 1); STAGE_B(SB(1, 0), b3); STAGE_B(SB(1, 1), b3h); STAGE_A(SA(1, 0), a3); \
    WAIT_V((KIND) == 1 || (KIND) == 4 ? 9 : (KIND) == 2 || (KIND) == 3 ? 10 : 8); \
    if ((KIND) == 3) asm volatile("" : "+v"(Xa), "+v"(Xb)); \
    if ((KIND) == 2) asm volatile("" : "+v"(fv)); \
    WAIT_L(0); BAR; SCHED; PRIO(1); \
    if ((KIND) == 3) {   \
      _Pragma("unroll") for (int e = 0; e < 8; ++e) { float q0, q1; \
        MF(4 * e); q0 = UNP(Xa, e); SCHED; MF(4 * e + 1); q1 = UNP(Xb, e); SCHED; MF(4 * e + 2); rs[e] = fadd1(q0, q1); SCHED; MF(4 * e + 3); SCHED; } \
      l1 += 8192u; l2 += 8192u; l3 += 8192u; sto += 8192; oo += srdO[2] >> 6; srdO[2] = OUT_BLK;     \
    } else MSEG(1); \
    PRIO(0); BAR; MEMF; SCHED; \
  } while (0)
  auto wait_flags = [&](unsigned f) {
    const unsigned long long t0 = __builtin_amdgcn_s_memrealtime();
    const unsigned pv = lane < 8 ? fb + vl4 : WS_TMO, z = 0;
    for (;;) {
      unsigned v; BLD1(v, srdC, pv, z); WAIT_V(0); asm volatile("" : "+v"(v));
      if (((__builtin_amdgcn_ballot_w64(v >= call) >> (4u * f)) & 15ull) == 15ull) break;
      if ((__builtin_amdgcn_ballot_w64(v == call) >> 63) != 0ull || __builtin_amdgcn_s_memrealtime() - t0 > DEADLINE_TICKS) {
        __hip_atomic_store(tmo, call, __ATOMIC_RELAXED, __HIP_MEMORY_SCOPE_AGENT); bad = 1; break; }
      __builtin_amdgcn_s_sleep(SLP);
    }
    bad = rfl(bad);
  };
  unsigned t = 0;
#pragma unroll 1
  for (; t < 32u; t += 2) TRIP(0, t);
  STAMP(2);
  { const unsigned eb = pb + sp * UNIT_B; E4(true, eb, 0, 1); E4(true, eb, 1, 0); E4(true, eb, 1, 1);
#pragma unroll
    for (int m = 0; m < 4; ++m) { u32x4 w_; PACK(w_, 0, 0, m); *(LAS u32x4*)(lds + sto + m * 8192) = w_; acc[0][0][m][0] = zero4(); acc[0][0][m][1] = zero4(); } }
  STAMP(3);
  cA = nA; cB = nB; tl = (unsigned)NK1 - 2u;
  nrec = 0;
  TRIP(1, 0u);
#pragma unroll 1
  for (t = 2; t < 2u + 2u * n1; t += 2) TRIP(0, t);
  STAMP(4);
  TRIP(2, t); t += 2;
  if ((__builtin_amdgcn_ballot_w64(fv >= call) & 15ull) != 15ull) wait_flags(0u);
  STAMP(5);
  asm volatile("" : "=v"(Ya), "=v"(Yb), "=v"(rs[0]), "=v"(rs[1]), "=v"(rs[2]), "=v"(rs[3]), "=v"(rs[4]), "=v"(rs[5]), "=v"(rs[6]), "=v"(rs[7]));
#pragma unroll 1
  for (int i = 0; i < 4; ++i) { TRIP(3, t); t += 2; }
  TRIP(4, t); t += 2;
  STAMP(6);
#pragma unroll 1
  for (; t < (unsigned)NK1; t += 2) TRIP(0, t);
  STAMP(7);
  { const unsigned eb = pb + ROUND_B + sp * UNIT_B; E4(false, eb, 0, 1); E4(false, eb, 1, 0); E4(false, eb, 1, 1); }
  if (wr == 0) BAR;
  STAMP(8);
  WAIT_V(0);
  STAMP(9);
  { u32x4 sc = srdC; const unsigned pubo = fb + 16u + sp * 4u; sc[2] = pubo + 4u; BST1("sc1", vcall, sc, vl4, pubo); }
  { u32x4 sc = srdC; const unsigned so = WS_CNT + (unsigned)(b * 8 + wid) * 4u; sc[2] = so + 4u; BST1("sc1", vcall, sc, vl4, so); }
  wait_flags(1u);
  STAMP(10);
  u32x4 R[4][3];
  { const unsigned lq = pb + ROUND_B + (sp << 15);
    const unsigned m1 = lq + ((sp + 1u) & 3u) * UNIT_B, m2 = lq + ((sp + 2u) & 3u) * UNIT_B, m3 = lq + ((sp + 3u) & 3u) * UNIT_B;
    BLD4(R[0][0], srdP, vl16, m1, 0); BLD4(R[0][1], srdP, vl16, m2, 0); BLD4(R[0][2], srdP, vl16, m3, 0);
    BLD4(R[1][0], srdP, vl16, m1, 8192); BLD4(R[1][1], srdP, vl16, m2, 8192); BLD4(R[1][2], srdP, vl16, m3, 8192);
    BLD4(R[2][0], srdP, vl16, m1, 16384); BLD4(R[2][1], srdP, vl16, m2, 16384); BLD4(R[2][2], srdP, vl16, m3, 16384);
    BLD4(R[3][0], srdP, vl16, m1, 24576); BLD4(R[3][1], srdP, vl16, m2, 24576); BLD4(R[3][2], srdP, vl16, m3, 24576); }
  STAMP(11);
  {
    const u32x4 so = {u1.olo, u1.ohi, OUT_BLK, 0x00020000u}; const unsigned o = u1.obase + qo;
#pragma unroll
    for (int m = 0; m < 4; ++m) {
      if (m == 0) WAIT_V(9); else if (m == 1) WAIT_V(7); else if (m == 2) WAIT_V(5); else WAIT_V(3);
      asm volatile("" : "+v"(R[m][0]), "+v"(R[m][1]), "+v"(R[m][2]));
      float v[8];
#pragma unroll
      for (int e = 0; e < 4; ++e) { auto r_ = __builtin_amdgcn_permlane16_swap(__float_as_uint(acc[0][0][m][0][e]), __float_as_uint(acc[0][0][m][1][e]), false, false); v[e] = __uint_as_float(r_[0]); v[4 + e] = __uint_as_float(r_[1]); }
      u32x4 w;
#pragma unroll
      for (int e = 0; e < 4; ++e) {
        const float lo_ = (v[2 * e] + blo(R[m][0][e])) + (blo(R[m][1][e]) + blo(R[m][2][e])), hi_ = (v[2 * e + 1] + bhi(R[m][0][e])) + (bhi(R[m][1][e]) + bhi(R[m][2][e]));
        w[e] = bad ? 0x7fc07fc0u : cvtpk(lo_, hi_); }
      if (m == 0) BST4("sc0 sc1", w, so, voffO, o, 0); else if (m == 1) BST4("sc0 sc1", w, so, voffO, o, 16 * NO * 2); else if (m == 2) BST4("sc0 sc1", w, so, voffO, o, 32 * NO * 2); else BST4("sc0 sc1", w, so, voffO, o, 48 * NO * 2);
    }
  }
  STAMP(13);
  WAIT_V(0);
  STAMP(14);
  MEMF; BAR; MEMF;
  if (wid == 0) {
    const unsigned pr = (unsigned)lane;
    const bool mine = pr < (unsigned)NR && pr != (unsigned)rank;
    if (mine) __hip_atomic_store((unsigned*)(peers.ws[pr] + WS_DONE) + rank * 256 + b, call, __ATOMIC_RELAXED, __HIP_MEMORY_SCOPE_SYSTEM);
    bool late = false;
    {
      const unsigned* flag = mine ? (const unsigned*)(myws + WS_DONE) + pr * 256 + b : tmo;
      const unsigned long long t0 = __builtin_amdgcn_s_memrealtime();
      for (;;) {
        const unsigned v = __hip_atomic_load(flag, __ATOMIC_RELAXED, __HIP_MEMORY_SCOPE_SYSTEM);
        if (__builtin_amdgcn_ballot_w64(mine && v < call) == 0ull) break;
        if (__builtin_amdgcn_ballot_w64(!mine && v == call) != 0ull || __builtin_amdgcn_s_memrealtime() - t0 > DEADLINE_TICKS) {
          __hip_atomic_store(tmo, call, __ATOMIC_RELAXED, __HIP_MEMORY_SCOPE_AGENT); late = true; break; }
        __builtin_amdgcn_s_sleep(SLP);
      }
    }
    __builtin_amdgcn_fence(__ATOMIC_ACQUIRE, "");
    if (lane == 0 && (late || __hip_atomic_load(tmo, __ATOMIC_RELAXED, __HIP_MEMORY_SCOPE_AGENT) == call)) peers.out[rank][(size_t)rank * M * NO + (size_t)b] = (u16)0x7fc0;
  }
}

extern "C" void kernel_launch(int rank, int nranks, ncclComm_t comm, hipStream_t stream,
                              void* const* d_in, const int* in_sizes, int n_in,
                              void* d_out, int out_size, const int* in_dtypes, int out_dtype,
                              void* const* d_out_all, void* const* d_ws_all, size_t ws_bytes) {
  if (nranks != NR || (unsigned)rank >= (unsigned)NR || ws_bytes < WS_NEED) return;
  HIP_CHECK(hipFuncSetAttribute((const void*)gemm_a2a, hipFuncAttributeMaxDynamicSharedMemorySize, LDS_BYTES));
  Peers p;
  for (int i = 0; i < NR; ++i) { p.out[i] = (u16*)d_out_all[i]; p.ws[i] = (char*)d_ws_all[i]; }
  gemm_a2a<<<256, 512, LDS_BYTES, stream>>>((const u16*)d_in[0], (const u16*)d_in[1], p, rank);
  HIP_CHECK(hipGetLastError());
}
```

```cpp
#include <hip/hip_runtime.h>
#include <rccl/rccl.h>
#include <cstdio>

#define HIP_CHECK(expr) do { hipError_t _e = (expr); if (_e != hipSuccess) { \
    fprintf(stderr, "HIP error %s at %s:%d\n", hipGetErrorString(_e), __FILE__, __LINE__); \
    return; } } while (0)

#define LAS __attribute__((address_space(3)))
typedef unsigned short u16;
typedef short bf16x8 __attribute__((ext_vector_type(8)));
typedef short s16x4 __attribute__((ext_vector_type(4)));
typedef float f32x4 __attribute__((ext_vector_type(4)));
typedef float f32x2 __attribute__((ext_vector_type(2)));
typedef unsigned u32x4 __attribute__((ext_vector_type(4)));
typedef __bf16 bf16x2_t __attribute__((ext_vector_type(2)));
constexpr int M = 1024, N = 8192, K = 8192, NR = 8, NO = 1024;
constexpr unsigned long long DEADLINE_TICKS = 100000000ull;
constexpr unsigned WS_TMO = 0;
constexpr unsigned WS_CNT = 4096;
constexpr unsigned WS_DONE = 16384;
constexpr unsigned WS_PFLAG = 65536;
constexpr unsigned WS_CTRL_BYTES = 131072;
constexpr unsigned WS_PART = 1u << 18;
constexpr unsigned UNIT_B = 131072, ROUND_B = 256u * UNIT_B, PART_BYTES = 2u * ROUND_B;
constexpr size_t WS_NEED = (size_t)WS_PART + PART_BYTES;
constexpr unsigned OUT_BLK = (unsigned)M * NO * 2;
static_assert(OUT_BLK / 64 == 16u * NO * 2u, "");
constexpr int HTB = 16384, STASH = 8 * HTB, LDS_BYTES = 10 * HTB;
constexpr unsigned kA = 128, kB = 64 * N * 2, hA = 128 * K * 2, hB = 256;
constexpr int NK1 = 32;
#define SLP 2
#define NSTAG ((wid + j) & 7)

__device__ __forceinline__ unsigned cvtpk(float lo, float hi) { f32x2 v = {lo, hi}; bf16x2_t b = __builtin_convertvector(v, bf16x2_t); return __builtin_bit_cast(unsigned, b); }
__device__ __forceinline__ s16x4 trrd(LAS const char* p) { return __builtin_amdgcn_ds_read_tr16_b64_v4i16((LAS s16x4*)p); }
__device__ __forceinline__ unsigned rfl(unsigned x) { return __builtin_amdgcn_readfirstlane(x); }
#define WAIT_V(n) asm volatile("s_waitcnt vmcnt(%0)" :: "n"(n) : "memory")
#define WAIT_L(n) asm volatile("s_waitcnt lgkmcnt(%0)" :: "n"(n) : "memory")
#define BAR __builtin_amdgcn_s_barrier()
#define SCHED __builtin_amdgcn_sched_barrier(0)
#define MEMF asm volatile("" ::: "memory")
#define BLDS(srd, voff, soff, GOFF, ldsbase, LOFF) do { unsigned tmp_; asm volatile("s_add_u32 m0, %1, %2\n\ts_add_u32 %0, %3, %4\n\tbuffer_load_dwordx4 %5, %6, %0 offen lds" \
   : "=&s"(tmp_) : "s"(ldsbase), "i"(LOFF), "s"(soff), "i"(GOFF), "v"(voff), "s"(srd) : "memory", "scc"); } while (0)
#define BLD4(dst, srd, voff, soff, GOFF) do { unsigned tmp_; asm volatile("s_add_u32 %1, %2, %3\n\tbuffer_load_dwordx4 %0, %4, %5, %1 offen sc1" \
   : "=v"(dst), "=&s"(tmp_) : "s"(soff), "i"(GOFF), "v"(voff), "s"(srd) : "memory", "scc"); } while (0)
#define BLD1(dst, srd, voff, soff) asm volatile("buffer_load_dword %0, %1, %2, %3 offen sc1" : "=v"(dst) : "v"(voff), "s"(srd), "s"(soff) : "memory")
#define BST4(MOD, data, srd, voff, soff, GOFF) do { unsigned tmp_; asm volatile("s_add_u32 %0, %1, %2\n\tbuffer_store_dwordx4 %3, %4, %5, %0 offen " MOD "\n\ts_nop 1" \
   : "=&s"(tmp_) : "s"(soff), "i"(GOFF), "v"(data), "v"(voff), "s"(srd) : "memory", "scc"); } while (0)
#define BST1(MOD, data, srd, voff, soff) asm volatile("buffer_store_dword %0, %1, %2, %3 offen " MOD :: "v"(data), "v"(voff), "s"(srd), "s"(soff) : "memory")
__device__ __forceinline__ u32x4 make_srd(const void* p, unsigned bytes) {
  unsigned long long a = (unsigned long long)p;
  u32x4 r; r[0] = rfl((unsigned)a); r[1] = rfl((unsigned)(a >> 32) & 0xffffu); r[2] = bytes; r[3] = 0x00020000u; return r;
}
struct Peers { u16* out[NR]; char* ws[NR]; };
struct Unit { unsigned a, b, obase, olo, ohi; };

__device__ __forceinline__ Unit unit_of(int r, int x, int j, int rank, const Peers& peers) {
  const int h = x >> 2; int d, pnl, pm;
  if (r == 0) {
    if (j < 28) { d = 1 + (j >> 2); pnl = h; pm = j & 3; }
    else { const int e = h * 4 + j - 28; d = e < 7 ? 1 + e : 7; pm = e < 7 ? 0 : 1; pnl = 2; }
  } else if (h == 0) {
    if (j < 28) { d = 1 + (j >> 2); pnl = 3; pm = j & 3; } else { d = 0; pnl = 0; pm = j - 28; }
  } else {
    if (j < 18) { d = 1 + j / 3; pm = 1 + j % 3; pnl = 2; } else if (j < 20) { d = 7; pm = j - 16; pnl = 2; } else { d = 0; pnl = 1 + ((j - 20) >> 2); pm = j & 3; }
  }
  const unsigned dst = (unsigned)((rank + d) & (NR - 1));
  const unsigned long long o = (unsigned long long)peers.out[dst] + (unsigned long long)rank * OUT_BLK;
  Unit u; u.a = (unsigned)(pm * 256 * K * 2); u.b = (unsigned)(((int)dst * 4 + pnl) * 512); u.obase = (unsigned)(pm * 256 * NO * 2 + pnl * 512);
  u.olo = rfl((unsigned)o); u.ohi = rfl((unsigned)(o >> 32) & 0xffffu);
  return u;
}
__device__ __forceinline__ float blo(unsigned w) { return __uint_as_float(w << 16); }
__device__ __forceinline__ float bhi(unsigned w) { return __uint_as_float(w & 0xffff0000u); }
__device__ __forceinline__ f32x4 zero4() { f32x2 a, b; asm volatile("v_mov_b64 %0, 0" : "=v"(a)); asm volatile("v_mov_b64 %0, 0" : "=v"(b)); return __builtin_shufflevector(a, b, 0, 1, 2, 3); }
__device__ __forceinline__ float fadd1(float a, float b) { float r = a + b; asm("" : "+v"(r)); return r; }

__global__ __launch_bounds__(512, 2) void gemm_a2a(const u16* __restrict__ A, const u16* __restrict__ W, Peers peers, int rank) {
  extern __shared__ __attribute__((aligned(16))) char smem_[];
  LAS char* lds = (LAS char*)smem_;
  const int tid = threadIdx.x, wid = __builtin_amdgcn_readfirstlane(tid >> 6), lane = tid & 63, wr = wid >> 2, wc = wid & 3, fr = lane & 15, fq = lane >> 4;
  const int b = blockIdx.x, x = b & 7, j = b >> 3;
  char* myws = peers.ws[rank];
  const u32x4 srdA = make_srd(A, (unsigned)M * K * 2), srdB = make_srd(W, (unsigned)K * N * 2), srdC = make_srd(myws, WS_CTRL_BYTES), srdP = make_srd(myws, (unsigned)WS_NEED);
  const unsigned vl16 = (unsigned)lane * 16u, vl4 = (unsigned)lane * 4u;
  unsigned cntv; { const unsigned so = WS_CNT + (unsigned)(b * 8 + wid) * 4u; BLD1(cntv, srdC, vl4, so); }
  unsigned voffA, voffB;
  { const int row = wid * 8 + (lane >> 3); const int ch = (lane & 7) ^ ((row >> 1) & 7); voffA = (unsigned)(row * K + ch * 8) * 2u; }
  { const int P = wid * 8 + (lane >> 3); const int k = ((P >> 5) << 4) | ((P & 1) << 3) | ((P >> 2) & 7), n64 = (P >> 1) & 1;
    const int sg = ((lane & 7) >> 1) ^ (k & 3), ch = sg * 2 + (lane & 1); voffB = (unsigned)(k * N + n64 * 64 + ch * 8) * 2u; }
  const unsigned ldsw = rfl((unsigned)(size_t)lds + (unsigned)wid * 1024u);
  const int aoff0 = (wr * 64 + fr) * 128 + ((fq ^ (fr >> 1)) << 4), aoff1 = aoff0 ^ 64;
  const int q = fr >> 2, p = fr & 3;
  const int brest = (fq >> 1) * 4096 + q * 512 + (fq & 1) * 128 + 8 * p + (wc >> 1) * 256;
  const int boff0 = brest + ((((wc & 1) * 2 + 0) ^ q) << 5), boff1 = brest + ((((wc & 1) * 2 + 1) ^ q) << 5);
  const unsigned voffO = (unsigned)((wr * 64 + fr) * NO * 2 + (wc * 32 + (fq & 1) * 16 + (fq >> 1) * 8) * 2);
  const unsigned sp = (unsigned)x & 3u, oa = sp >> 1, ob = sp & 1u;
  const unsigned dA = oa ? 0u - hA : hA, dB = ob ? 0u - hB : hB;
  const Unit u0 = unit_of(0, x, j, rank, peers);
#define SA(b_, h) (((b_) * 2 + (h)) * HTB)
#define SB(b_, h) ((4 + (b_) * 2 + (h)) * HTB)
#define STAGE_A(bufoff, so) do { BLDS(srdA2, voffA, so, 0, ldsw, (bufoff)); BLDS(srdA2, voffA, so, 64 * K * 2, ldsw, (bufoff) + 8192); } while (0)
#define STAGE_B(bufoff, so) do { BLDS(srdB2, voffB, so, 0, ldsw, (bufoff)); BLDS(srdB2, voffB, so, 32 * N * 2, ldsw, (bufoff) + 8192); } while (0)
#define LDA2(dst_, b_, h, m0) do { _Pragma("unroll") for (int m = m0; m < m0 + 2; ++m) { dst_[m][0] = *(const LAS bf16x8*)(lds + SA(b_, h) + aoff0 + m * 2048); dst_[m][1] = *(const LAS bf16x8*)(lds + SA(b_, h) + aoff1 + m * 2048); } } while (0)
#define LDA(dst_, b_, h) do { LDA2(dst_, b_, h, 0); LDA2(dst_, b_, h, 2); } while (0)
#define LDB1(dst_, b_, h, boff, s) do { s16x4 lo_ = trrd(lds + SB(b_, h) + boff + (s) * 8192), hi_ = trrd(lds + SB(b_, h) + boff + (s) * 8192 + 2048); dst_ = __builtin_shufflevector(lo_, hi_, 0, 1, 2, 3, 4, 5, 6, 7); } while (0)
#define LDB(dst_, b_, h) do { LDB1(dst_[0][0], b_, h, boff0, 0); LDB1(dst_[0][1], b_, h, boff0, 1); LDB1(dst_[1][0], b_, h, boff1, 0); LDB1(dst_[1][1], b_, h, boff1, 1); } while (0)
#define MFM(i) ((i) >> 3)
#define MFN(i) ((MFM(i) & 1) ? 3 - (((i) >> 1) & 3) : (((i) >> 1) & 3))
#define MFK(i) (((i) ^ ((i) >> 1)) & 1)
#define MFA(ai, i) acc[ai][MFN(i) >> 1][MFM(i)][MFN(i) & 1] = __builtin_amdgcn_mfma_f32_16x16x32_bf16((MFN(i) >> 1) ? B1[MFN(i) & 1][MFK(i)] : B0[MFN(i) & 1][MFK(i)], At[MFM(i)][MFK(i)], acc[ai][MFN(i) >> 1][MFM(i)][MFN(i) & 1], 0, 0, 0)
#define MSEG(ai) do { _Pragma("unroll") for (int i_ = 0; i_ < 32; ++i_) { MFA(ai, i_); SCHED; } } while (0)
#define PRIO(n) __builtin_amdgcn_s_setprio(n)
#define PACK(w, ai, bj, m) do { const f32x4 x_ = acc[ai][bj][m][0], y_ = acc[ai][bj][m][1]; \
    auto r0_ = __builtin_amdgcn_permlane16_swap(cvtpk(x_[0], x_[1]), cvtpk(y_[0], y_[1]), false, false); \
    auto r1_ = __builtin_amdgcn_permlane16_swap(cvtpk(x_[2], x_[3]), cvtpk(y_[2], y_[3]), false, false); \
    w = (u32x4){r0_[0], r1_[0], r0_[1], r1_[1]}; } while (0)
#define E1(ZERO, eq, ai, bj, m) do { u32x4 w_; PACK(w_, ai, bj, m); BST4("sc1", w_, srdP, vl16, eq, (m) * 8192); if (ZERO) { acc[ai][bj][m][0] = zero4(); acc[ai][bj][m][1] = zero4(); } } while (0)
#define E4(ZERO, eb, ai, bj) do { const unsigned eq_ = (eb) + ((sp ^ (unsigned)((ai) * 2 + (bj))) << 15); E1(ZERO, eq_, ai, bj, 0); E1(ZERO, eq_, ai, bj, 1); E1(ZERO, eq_, ai, bj, 2); E1(ZERO, eq_, ai, bj, 3); } while (0)
  bf16x8 At[4][2], B0[2][2], B1[2][2];
  const u32x4 srdA2 = srdA, srdB2 = srdB;
  unsigned cA = u0.a + oa * hA + sp * 32u * kA, cB = u0.b + ob * hB + sp * 32u * kB;
  { const unsigned bh = cB + dB, ah = cA + dA; STAGE_B(SB(0, 0), cB); STAGE_B(SB(0, 1), bh); STAGE_A(SA(0, 0), cA); STAGE_A(SA(0, 1), ah); }
  const Unit u1 = unit_of(1, x, j, rank, peers);
  f32x4 acc[2][2][4][2];
#pragma unroll
  for (int a = 0; a < 2; ++a)
#pragma unroll
    for (int c = 0; c < 2; ++c)
#pragma unroll
      for (int m = 0; m < 4; ++m)
#pragma unroll
        for (int n = 0; n < 2; ++n) acc[a][c][m][n] = zero4();
  if (wr == 1) BAR;
  WAIT_V(2); BAR; MEMF;
  { const unsigned b1 = cB + kB, a1 = cA + kA, b1h = b1 + dB; STAGE_B(SB(1, 0), b1); STAGE_A(SA(1, 0), a1); STAGE_B(SB(1, 1), b1h); }
  asm volatile("" : "+v"(cntv));
  const unsigned call = rfl(cntv) + 1u;
  const unsigned vcall = call;
  WAIT_V(6); BAR; MEMF;
  const unsigned T = ((unsigned)x >> 2) * 32u + (unsigned)j;
  const unsigned pb = WS_PART + T * 4u * UNIT_B + (unsigned)wid * 1024u;
  const unsigned fb = WS_PFLAG + (T * 8u + (unsigned)wid) * 32u;
  const unsigned qo = oa * (128u * NO * 2u) + ob * 256u;
  const unsigned n1 = (unsigned)(NSTAG);
  float rs[8]; u32x4 Xa, Xb, Ya, Yb, ow; unsigned fv;
  unsigned* tmo = (unsigned*)(myws + WS_TMO);
  unsigned bad = 0;
  unsigned nA = u1.a + oa * hA + sp * (unsigned)NK1 * kA, nB = u1.b + ob * hB + sp * (unsigned)NK1 * kB, tl = 30u;
  u32x4 srdO = {u0.olo, u0.ohi, 0u, 0x00020000u}, srdZ = srdC; srdZ[2] = 0;
  unsigned l1 = pb + (sp << 15) + ((sp + 1u) & 3u) * UNIT_B, l2 = pb + (sp << 15) + ((sp + 2u) & 3u) * UNIT_B, l3 = pb + (sp << 15) + ((sp + 3u) & 3u) * UNIT_B, oo = u0.obase + qo, nrec = ~0u;
  int sto = STASH + wid * 1024 + lane * 16;
#define MF(i) MFA(1, i)
#define UNP(L, e) (((e) & 1) ? bhi(L[(e) >> 1]) : blo(L[(e) >> 1]))
#define SP1LOAD(b_, abuf, aso, XA, XB) do { const u32x4 sa_ = (b_) ? srdA2 : srdA; LDB(B0, b_, 0); BLDS(sa_, voffA, aso, 0, ldsw, abuf); LDB(B1, b_, 1); BLDS(sa_, voffA, aso, 64 * K * 2, ldsw, (abuf) + 8192); \
    LDA2(At, b_, 0, 0); XA; LDA2(At, b_, 0, 2); XB; } while (0)
#define TRIP(KIND, t) do { \
    const bool last = (t) == tl; \
    const unsigned a1h = cA + ((t) + 1u) * kA + dA; \
    const unsigned a2 = last ? nA : cA + ((t) + 2u) * kA, b2 = last ? nB : cB + ((t) + 2u) * kB; \
    const unsigned a2h = a2 + dA, b2h = b2 + dB, a3 = a2 + kA, b3 = b2 + kB, b3h = b3 + dB; \
    u32x4 srdA2 = srdA, srdB2 = srdB; srdA2[2] &= last ? nrec : ~0u; srdB2[2] &= last ? nrec : ~0u; \
    if ((KIND) == 3) SP1LOAD(0, SA(1, 1), a1h, BLD4(Xa, srdP, vl16, l1, 0), BLD4(Xb, srdP, vl16, l2, 0)); \
    else if ((KIND) == 2) SP1LOAD(0, SA(1, 1), a1h, BLD1(fv, srdC, vl4, fb), (void)0); \
    else SP1LOAD(0, SA(1, 1), a1h, (void)0, (void)0); \
    WAIT_V((KIND) == 1 ? 20 : (KIND) == 2 ? 9 : (KIND) == 3 ? 12 : (KIND) == 4 ? 10 : 8); \
    WAIT_L(0); BAR; PRIO(1); MSEG(0); PRIO(0); BAR; MEMF; SCHED; \
    LDA(At, 0, 1); STAGE_B(SB(0, 0), b2); STAGE_B(SB(0, 1), b2h); STAGE_A(SA(0, 0), a2); \
    WAIT_V((KIND) == 1 ? 20 : (KIND) == 2 ? 9 : (KIND) == 3 ? 10 : 8); \
    if ((KIND) >= 3) asm volatile("" : "+v"(Ya)); \
    WAIT_L(0); BAR; SCHED; PRIO(1); \
    if ((KIND) >= 3) {   \
      float tp = 0.f; \
      _Pragma("unroll") for (int e = 0; e < 8; ++e) { float q0, q1; \
        MF(4 * e); q0 = UNP(Ya, e); SCHED; MF(4 * e + 1); q1 = UNP(Yb, e); SCHED; MF(4 * e + 2); q0 = fadd1(q0, q1); SCHED; \
        MF(4 * e + 3); q0 = fadd1(rs[e], q0); if (e & 1) ow[e >> 1] = cvtpk(tp, q0); tp = q0; SCHED; } \
    } else MSEG(1); \
    PRIO(0); BAR; MEMF; SCHED; \
    if ((KIND) == 3) { Yb = *(const LAS u32x4*)(lds + sto); SP1LOAD(1, SA(0, 1), a2h, BLD4(Ya, srdP, vl16, l3, 0), BST4("sc0 sc1", ow, srdO, voffO, oo, 0)); } \
    else if ((KIND) == 4) SP1LOAD(1, SA(0, 1), a2h, (void)0, BST4("sc0 sc1", ow, srdO, voffO, oo, 0)); \
    else if ((KIND) == 2) SP1LOAD(1, SA(0, 1), a2h, BST1("", vcall, srdZ, vl4, fb), BST1("", vcall, srdZ, vl4, fb));     \
    else SP1LOAD(1, SA(0, 1), a2h, (void)0, (void)0); \
    WAIT_V((KIND) == 2 ? 11 : (KIND) == 3 ? 12 : (KIND) == 4 ? 9 : 8); \
    if ((KIND) == 1) { const unsigned pubo = fb + sp * 4u; u32x4 sc = srdC; sc[2] = pubo + 4u; BST1("sc1", vcall, sc, vl4, pubo); }     \
    WAIT_L(0); BAR; PRIO(1); MSEG(0); PRIO(0); BAR; MEMF; SCHED; \
    LDA(At, 1, 1); STAGE_B(SB(1, 0), b3); STAGE_B(SB(1, 1), b3h); STAGE_A(SA(1, 0), a3); \
    WAIT_V((KIND) == 1 || (KIND) == 4 ? 9 : (KIND) == 2 || (KIND) == 3 ? 10 : 8); \
    if ((KIND) == 3) asm volatile("" : "+v"(Xa), "+v"(Xb)); \
    if ((KIND) == 2) asm volatile("" : "+v"(fv)); \
    WAIT_L(0); BAR; SCHED; PRIO(1); \
    if ((KIND) == 3) {   \
      _Pragma("unroll") for (int e = 0; e < 8; ++e) { float q0, q1; \
        MF(4 * e); q0 = UNP(Xa, e); SCHED; MF(4 * e + 1); q1 = UNP(Xb, e); SCHED; MF(4 * e + 2); rs[e] = fadd1(q0, q1); SCHED; MF(4 * e + 3); SCHED; } \
      l1 += 8192u; l2 += 8192u; l3 += 8192u; sto += 8192; oo += srdO[2] >> 6; srdO[2] = OUT_BLK;     \
    } else MSEG(1); \
    PRIO(0); BAR; MEMF; SCHED; \
  } while (0)
  auto wait_flags = [&](unsigned f) {
    const unsigned long long t0 = __builtin_amdgcn_s_memrealtime();
    const unsigned pv = lane < 8 ? fb + vl4 : WS_TMO, z = 0;
    for (;;) {
      unsigned v; BLD1(v, srdC, pv, z); WAIT_V(0); asm volatile("" : "+v"(v));
      if (((__builtin_amdgcn_ballot_w64(v >= call) >> (4u * f)) & 15ull) == 15ull) break;
      if ((__builtin_amdgcn_ballot_w64(v == call) >> 63) != 0ull || __builtin_amdgcn_s_memrealtime() - t0 > DEADLINE_TICKS) {
        __hip_atomic_store(tmo, call, __ATOMIC_RELAXED, __HIP_MEMORY_SCOPE_AGENT); bad = 1; break; }
      __builtin_amdgcn_s_sleep(SLP);
    }
    bad = rfl(bad);
  };
  unsigned t = 0;
#pragma unroll 1
  for (; t < 32u; t += 2) TRIP(0, t);
  { const unsigned eb = pb + sp * UNIT_B; E4(true, eb, 0, 1); E4(true, eb, 1, 0); E4(true, eb, 1, 1);
#pragma unroll
    for (int m = 0; m < 4; ++m) { u32x4 w_; PACK(w_, 0, 0, m); *(LAS u32x4*)(lds + sto + m * 8192) = w_; acc[0][0][m][0] = zero4(); acc[0][0][m][1] = zero4(); } }
  cA = nA; cB = nB; tl = (unsigned)NK1 - 2u;
  nrec = 0;
  TRIP(1, 0u);
#pragma unroll 1
  for (t = 2; t < 2u + 2u * n1; t += 2) TRIP(0, t);
  TRIP(2, t); t += 2;
  if ((__builtin_amdgcn_ballot_w64(fv >= call) & 15ull) != 15ull) wait_flags(0u);
  asm volatile("" : "=v"(Ya), "=v"(Yb), "=v"(rs[0]), "=v"(rs[1]), "=v"(rs[2]), "=v"(rs[3]), "=v"(rs[4]), "=v"(rs[5]), "=v"(rs[6]), "=v"(rs[7]));
#pragma unroll 1
  for (int i = 0; i < 4; ++i) { TRIP(3, t); t += 2; }
  TRIP(4, t); t += 2;
#pragma unroll 1
  for (; t < (unsigned)NK1; t += 2) TRIP(0, t);
  { const unsigned eb = pb + ROUND_B + sp * UNIT_B; E4(false, eb, 0, 1); E4(false, eb, 1, 0); E4(false, eb, 1, 1); }
  if (wr == 0) BAR;
  WAIT_V(0);
  { u32x4 sc = srdC; const unsigned pubo = fb + 16u + sp * 4u; sc[2] = pubo + 4u; BST1("sc1", vcall, sc, vl4, pubo); }
  { u32x4 sc = srdC; const unsigned so = WS_CNT + (unsigned)(b * 8 + wid) * 4u; sc[2] = so + 4u; BST1("sc1", vcall, sc, vl4, so); }
  wait_flags(1u);
  u32x4 R[4][3];
  { const unsigned lq = pb + ROUND_B + (sp << 15);
    const unsigned m1 = lq + ((sp + 1u) & 3u) * UNIT_B, m2 = lq + ((sp + 2u) & 3u) * UNIT_B, m3 = lq + ((sp + 3u) & 3u) * UNIT_B;
    BLD4(R[0][0], srdP, vl16, m1, 0); BLD4(R[0][1], srdP, vl16, m2, 0); BLD4(R[0][2], srdP, vl16, m3, 0);
    BLD4(R[1][0], srdP, vl16, m1, 8192); BLD4(R[1][1], srdP, vl16, m2, 8192); BLD4(R[1][2], srdP, vl16, m3, 8192);
    BLD4(R[2][0], srdP, vl16, m1, 16384); BLD4(R[2][1], srdP, vl16, m2, 16384); BLD4(R[2][2], srdP, vl16, m3, 16384);
    BLD4(R[3][0], srdP, vl16, m1, 24576); BLD4(R[3][1], srdP, vl16, m2, 24576); BLD4(R[3][2], srdP, vl16, m3, 24576); }
  {
    const u32x4 so = {u1.olo, u1.ohi, OUT_BLK, 0x00020000u}; const unsigned o = u1.obase + qo;
#pragma unroll
    for (int m = 0; m < 4; ++m) {
      if (m == 0) WAIT_V(9); else if (m == 1) WAIT_V(7); else if (m == 2) WAIT_V(5); else WAIT_V(3);
      asm volatile("" : "+v"(R[m][0]), "+v"(R[m][1]), "+v"(R[m][2]));
      float v[8];
#pragma unroll
      for (int e = 0; e < 4; ++e) { auto r_ = __builtin_amdgcn_permlane16_swap(__float_as_uint(acc[0][0][m][0][e]), __float_as_uint(acc[0][0][m][1][e]), false, false); v[e] = __uint_as_float(r_[0]); v[4 + e] = __uint_as_float(r_[1]); }
      u32x4 w;
#pragma unroll
      for (int e = 0; e < 4; ++e) {
        const float lo_ = (v[2 * e] + blo(R[m][0][e])) + (blo(R[m][1][e]) + blo(R[m][2][e])), hi_ = (v[2 * e + 1] + bhi(R[m][0][e])) + (bhi(R[m][1][e]) + bhi(R[m][2][e]));
        w[e] = bad ? 0x7fc07fc0u : cvtpk(lo_, hi_); }
      if (m == 0) BST4("sc0 sc1", w, so, voffO, o, 0); else if (m == 1) BST4("sc0 sc1", w, so, voffO, o, 16 * NO * 2); else if (m == 2) BST4("sc0 sc1", w, so, voffO, o, 32 * NO * 2); else BST4("sc0 sc1", w, so, voffO, o, 48 * NO * 2);
    }
  }
  WAIT_V(0);
  MEMF; BAR; MEMF;
  if (wid == 0) {
    const unsigned pr = (unsigned)lane;
    const bool mine = pr < (unsigned)NR && pr != (unsigned)rank;
    if (mine) __hip_atomic_store((unsigned*)(peers.ws[pr] + WS_DONE) + rank * 256 + b, call, __ATOMIC_RELAXED, __HIP_MEMORY_SCOPE_SYSTEM);
    bool late = false;
    {
      const unsigned* flag = mine ? (const unsigned*)(myws + WS_DONE) + pr * 256 + b : tmo;
      const unsigned long long t0 = __builtin_amdgcn_s_memrealtime();
      for (;;) {
        const unsigned v = __hip_atomic_load(flag, __ATOMIC_RELAXED, __HIP_MEMORY_SCOPE_SYSTEM);
        if (__builtin_amdgcn_ballot_w64(mine && v < call) == 0ull) break;
        if (__builtin_amdgcn_ballot_w64(!mine && v == call) != 0ull || __builtin_amdgcn_s_memrealtime() - t0 > DEADLINE_TICKS) {
          __hip_atomic_store(tmo, call, __ATOMIC_RELAXED, __HIP_MEMORY_SCOPE_AGENT); late = true; break; }
        __builtin_amdgcn_s_sleep(SLP);
      }
    }
    __builtin_amdgcn_fence(__ATOMIC_ACQUIRE, "");
    if (lane == 0 && (late || __hip_atomic_load(tmo, __ATOMIC_RELAXED, __HIP_MEMORY_SCOPE_AGENT) == call)) peers.out[rank][(size_t)rank * M * NO + (size_t)b] = (u16)0x7fc0;
  }
}

extern "C" void kernel_launch(int rank, int nranks, ncclComm_t comm, hipStream_t stream,
                              void* const* d_in, const int* in_sizes, int n_in,
                              void* d_out, int out_size, const int* in_dtypes, int out_dtype,
                              void* const* d_out_all, void* const* d_ws_all, size_t ws_bytes) {
  if (nranks != NR || (unsigned)rank >= (unsigned)NR || ws_bytes < WS_NEED) return;
  HIP_CHECK(hipFuncSetAttribute((const void*)gemm_a2a, hipFuncAttributeMaxDynamicSharedMemorySize, LDS_BYTES));
  Peers p;
  for (int i = 0; i < NR; ++i) { p.out[i] = (u16*)d_out_all[i]; p.ws[i] = (char*)d_ws_all[i]; }
  gemm_a2a<<<256, 512, LDS_BYTES, stream>>>((const u16*)d_in[0], (const u16*)d_in[1], p, rank);
  HIP_CHECK(hipGetLastError());
}
```

```cpp
#include <hip/hip_runtime.h>
#include <rccl/rccl.h>
#include <cstdio>

#define HIP_CHECK(expr) do { hipError_t _e = (expr); if (_e != hipSuccess) { \
    fprintf(stderr, "HIP error %s at %s:%d\n", hipGetErrorString(_e), __FILE__, __LINE__); \
    return; } } while (0)

#define LAS __attribute__((address_space(3)))
typedef unsigned short u16;
typedef short bf16x8 __attribute__((ext_vector_type(8)));
typedef short s16x4 __attribute__((ext_vector_type(4)));
typedef float f32x4 __attribute__((ext_vector_type(4)));
typedef float f32x2 __attribute__((ext_vector_type(2)));
typedef unsigned u32x4 __attribute__((ext_vector_type(4)));
typedef __bf16 bf16x2_t __attribute__((ext_vector_type(2)));
constexpr int M = 1024, N = 8192, K = 8192, NR = 8, NO = 1024;
constexpr unsigned long long DEADLINE_TICKS = 100000000ull;
constexpr unsigned WS_TMO = 0;
constexpr unsigned WS_CNT = 4096;
constexpr unsigned WS_DONE = 16384;
constexpr unsigned WS_PFLAG = 65536;
constexpr unsigned WS_CTRL_BYTES = 131072;
constexpr unsigned WS_PART = 1u << 18;
constexpr unsigned UNIT_B = 131072, ROUND_B = 256u * UNIT_B, PART_BYTES = 2u * ROUND_B;
constexpr size_t WS_NEED = (size_t)WS_PART + PART_BYTES;
constexpr unsigned OUT_BLK = (unsigned)M * NO * 2;
static_assert(OUT_BLK / 64 == 16u * NO * 2u, "");
constexpr int HTB = 16384, STASH = 8 * HTB, LDS_BYTES = 10 * HTB;
constexpr unsigned kA = 128, kB = 64 * N * 2, hA = 128 * K * 2, hB = 256;
constexpr int NK1 = 32;
#define SLP 2
#define NSTAG ((wid + j) & 7)

__device__ __forceinline__ unsigned cvtpk(float lo, float hi) { f32x2 v = {lo, hi}; bf16x2_t b = __builtin_convertvector(v, bf16x2_t); return __builtin_bit_cast(unsigned, b); }
__device__ __forceinline__ s16x4 trrd(LAS const char* p) { return __builtin_amdgcn_ds_read_tr16_b64_v4i16((LAS s16x4*)p); }
__device__ __forceinline__ unsigned rfl(unsigned x) { return __builtin_amdgcn_readfirstlane(x); }
#define WAIT_V(n) asm volatile("s_waitcnt vmcnt(%0)" :: "n"(n) : "memory")
#define WAIT_L(n) asm volatile("s_waitcnt lgkmcnt(%0)" :: "n"(n) : "memory")
#define BAR __builtin_amdgcn_s_barrier()
#define SCHED __builtin_amdgcn_sched_barrier(0)
#define MEMF asm volatile("" ::: "memory")
#define BLDS(srd, voff, soff, GOFF, ldsbase, LOFF) do { unsigned tmp_; asm volatile("s_add_u32 m0, %1, %2\n\ts_add_u32 %0, %3, %4\n\tbuffer_load_dwordx4 %5, %6, %0 offen lds" \
   : "=&s"(tmp_) : "s"(ldsbase), "i"(LOFF), "s"(soff), "i"(GOFF), "v"(voff), "s"(srd) : "memory", "scc"); } while (0)
#define BLD4(dst, srd, voff, soff, GOFF) do { unsigned tmp_; asm volatile("s_add_u32 %1, %2, %3\n\tbuffer_load_dwordx4 %0, %4, %5, %1 offen sc1 nt" \
   : "=v"(dst), "=&s"(tmp_) : "s"(soff), "i"(GOFF), "v"(voff), "s"(srd) : "memory", "scc"); } while (0)
#define BLD1(dst, srd, voff, soff) asm volatile("buffer_load_dword %0, %1, %2, %3 offen sc1" : "=v"(dst) : "v"(voff), "s"(srd), "s"(soff) : "memory")
#define BST4(MOD, data, srd, voff, soff, GOFF) do { unsigned tmp_; asm volatile("s_add_u32 %0, %1, %2\n\tbuffer_store_dwordx4 %3, %4, %5, %0 offen " MOD "\n\ts_nop 1" \
   : "=&s"(tmp_) : "s"(soff), "i"(GOFF), "v"(data), "v"(voff), "s"(srd) : "memory", "scc"); } while (0)
#define BST1(MOD, data, srd, voff, soff) asm volatile("buffer_store_dword %0, %1, %2, %3 offen " MOD :: "v"(data), "v"(voff), "s"(srd), "s"(soff) : "memory")
__device__ __forceinline__ u32x4 make_srd(const void* p, unsigned bytes) {
  unsigned long long a = (unsigned long long)p;
  u32x4 r; r[0] = rfl((unsigned)a); r[1] = rfl((unsigned)(a >> 32) & 0xffffu); r[2] = bytes; r[3] = 0x00020000u; return r;
}
struct Peers { u16* out[NR]; char* ws[NR]; };
struct Unit { unsigned a, b, obase, olo, ohi; };

__device__ __forceinline__ Unit unit_of(int r, int x, int j, int rank, const Peers& peers) {
  const int h = x >> 2; int d, pnl, pm;
  if (r == 0) {
    if (j < 28) { d = 1 + (j >> 2); pnl = h; pm = j & 3; }
    else { const int e = h * 4 + j - 28; d = e < 7 ? 1 + e : 7; pm = e < 7 ? 0 : 1; pnl = 2; }
  } else if (h == 0) {
    if (j < 28) { d = 1 + (j >> 2); pnl = 3; pm = j & 3; } else { d = 0; pnl = 0; pm = j - 28; }
  } else {
    if (j < 18) { d = 1 + j / 3; pm = 1 + j % 3; pnl = 2; } else if (j < 20) { d = 7; pm = j - 16; pnl = 2; } else { d = 0; pnl = 1 + ((j - 20) >> 2); pm = j & 3; }
  }
  const unsigned dst = (unsigned)((rank + d) & (NR - 1));
  const unsigned long long o = (unsigned long long)peers.out[dst] + (unsigned long long)rank * OUT_BLK;
  Unit u; u.a = (unsigned)(pm * 256 * K * 2); u.b = (unsigned)(((int)dst * 4 + pnl) * 512); u.obase = (unsigned)(pm * 256 * NO * 2 + pnl * 512);
  u.olo = rfl((unsigned)o); u.ohi = rfl((unsigned)(o >> 32) & 0xffffu);
  return u;
}
__device__ __forceinline__ float blo(unsigned w) { return __uint_as_float(w << 16); }
__device__ __forceinline__ float bhi(unsigned w) { return __uint_as_float(w & 0xffff0000u); }
__device__ __forceinline__ f32x4 zero4() { f32x2 a, b; asm volatile("v_mov_b64 %0, 0" : "=v"(a)); asm volatile("v_mov_b64 %0, 0" : "=v"(b)); return __builtin_shufflevector(a, b, 0, 1, 2, 3); }
__device__ __forceinline__ float fadd1(float a, float b) { float r = a + b; asm("" : "+v"(r)); return r; }

__global__ __launch_bounds__(512, 2) void gemm_a2a(const u16* __restrict__ A, const u16* __restrict__ W, Peers peers, int rank) {
  extern __shared__ __attribute__((aligned(16))) char smem_[];
  LAS char* lds = (LAS char*)smem_;
  const int tid = threadIdx.x, wid = __builtin_amdgcn_readfirstlane(tid >> 6), lane = tid & 63, wr = wid >> 2, wc = wid & 3, fr = lane & 15, fq = lane >> 4;
  const int b = blockIdx.x, x = b & 7, j = b >> 3;
  char* myws = peers.ws[rank];
  const u32x4 srdA = make_srd(A, (unsigned)M * K * 2), srdB = make_srd(W, (unsigned)K * N * 2), srdC = make_srd(myws, WS_CTRL_BYTES), srdP = make_srd(myws, (unsigned)WS_NEED);
  const unsigned vl16 = (unsigned)lane * 16u, vl4 = (unsigned)lane * 4u;
  unsigned cntv; { const unsigned so = WS_CNT + (unsigned)(b * 8 + wid) * 4u; BLD1(cntv, srdC, vl4, so); }
  unsigned voffA, voffB;
  { const int row = wid * 8 + (lane >> 3); const int ch = (lane & 7) ^ ((row >> 1) & 7); voffA = (unsigned)(row * K + ch * 8) * 2u; }
  { const int P = wid * 8 + (lane >> 3); const int k = ((P >> 5) << 4) | ((P & 1) << 3) | ((P >> 2) & 7), n64 = (P >> 1) & 1;
    const int sg = ((lane & 7) >> 1) ^ (k & 3), ch = sg * 2 + (lane & 1); voffB = (unsigned)(k * N + n64 * 64 + ch * 8) * 2u; }
  const unsigned ldsw = rfl((unsigned)(size_t)lds + (unsigned)wid * 1024u);
  const int aoff0 = (wr * 64 + fr) * 128 + ((fq ^ (fr >> 1)) << 4), aoff1 = aoff0 ^ 64;
  const int q = fr >> 2, p = fr & 3;
  const int brest = (fq >> 1) * 4096 + q * 512 + (fq & 1) * 128 + 8 * p + (wc >> 1) * 256;
  const int boff0 = brest + ((((wc & 1) * 2 + 0) ^ q) << 5), boff1 = brest + ((((wc & 1) * 2 + 1) ^ q) << 5);
  const unsigned voffO = (unsigned)((wr * 64 + fr) * NO * 2 + (wc * 32 + (fq & 1) * 16 + (fq >> 1) * 8) * 2);
  const unsigned sp = (unsigned)x & 3u, oa = sp >> 1, ob = sp & 1u;
  const unsigned dA = oa ? 0u - hA : hA, dB = ob ? 0u - hB : hB;
  const Unit u0 = unit_of(0, x, j, rank, peers);
#define SA(b_, h) (((b_) * 2 + (h)) * HTB)
#define SB(b_, h) ((4 + (b_) * 2 + (h)) * HTB)
#define STAGE_A(bufoff, so) do { BLDS(srdA2, voffA, so, 0, ldsw, (bufoff)); BLDS(srdA2, voffA, so, 64 * K * 2, ldsw, (bufoff) + 8192); } while (0)
#define STAGE_B(bufoff, so) do { BLDS(srdB2, voffB, so, 0, ldsw, (bufoff)); BLDS(srdB2, voffB, so, 32 * N * 2, ldsw, (bufoff) + 8192); } while (0)
#define LDA2(dst_, b_, h, m0) do { _Pragma("unroll") for (int m = m0; m < m0 + 2; ++m) { dst_[m][0] = *(const LAS bf16x8*)(lds + SA(b_, h) + aoff0 + m * 2048); dst_[m][1] = *(const LAS bf16x8*)(lds + SA(b_, h) + aoff1 + m * 2048); } } while (0)
#define LDA(dst_, b_, h) do { LDA2(dst_, b_, h, 0); LDA2(dst_, b_, h, 2); } while (0)
#define LDB1(dst_, b_, h, boff, s) do { s16x4 lo_ = trrd(lds + SB(b_, h) + boff + (s) * 8192), hi_ = trrd(lds + SB(b_, h) + boff + (s) * 8192 + 2048); dst_ = __builtin_shufflevector(lo_, hi_, 0, 1, 2, 3, 4, 5, 6, 7); } while (0)
#define LDB(dst_, b_, h) do { LDB1(dst_[0][0], b_, h, boff0, 0); LDB1(dst_[0][1], b_, h, boff0, 1); LDB1(dst_[1][0], b_, h, boff1, 0); LDB1(dst_[1][1], b_, h, boff1, 1); } while (0)
#define MFM(i) ((i) >> 3)
#define MFN(i) ((MFM(i) & 1) ? 3 - (((i) >> 1) & 3) : (((i) >> 1) & 3))
#define MFK(i) (((i) ^ ((i) >> 1)) & 1)
#define MFA(ai, i) acc[ai][MFN(i) >> 1][MFM(i)][MFN(i) & 1] = __builtin_amdgcn_mfma_f32_16x16x32_bf16((MFN(i) >> 1) ? B1[MFN(i) & 1][MFK(i)] : B0[MFN(i) & 1][MFK(i)], At[MFM(i)][MFK(i)], acc[ai][MFN(i) >> 1][MFM(i)][MFN(i) & 1], 0, 0, 0)
#define MSEG(ai) do { _Pragma("unroll") for (int i_ = 0; i_ < 32; ++i_) { MFA(ai, i_); SCHED; } } while (0)
#define PRIO(n) __builtin_amdgcn_s_setprio(n)
#define PACK(w, ai, bj, m) do { const f32x4 x_ = acc[ai][bj][m][0], y_ = acc[ai][bj][m][1]; \
    auto r0_ = __builtin_amdgcn_permlane16_swap(cvtpk(x_[0], x_[1]), cvtpk(y_[0], y_[1]), false, false); \
    auto r1_ = __builtin_amdgcn_permlane16_swap(cvtpk(x_[2], x_[3]), cvtpk(y_[2], y_[3]), false, false); \
    w = (u32x4){r0_[0], r1_[0], r0_[1], r1_[1]}; } while (0)
#define E1(ZERO, eq, ai, bj, m) do { u32x4 w_; PACK(w_, ai, bj, m); BST4("sc1 nt", w_, srdP, vl16, eq, (m) * 8192); if (ZERO) { acc[ai][bj][m][0] = zero4(); acc[ai][bj][m][1] = zero4(); } } while (0)
#define E4(ZERO, eb, ai, bj) do { const unsigned eq_ = (eb) + ((sp ^ (unsigned)((ai) * 2 + (bj))) << 15); E1(ZERO, eq_, ai, bj, 0); E1(ZERO, eq_, ai, bj, 1); E1(ZERO, eq_, ai, bj, 2); E1(ZERO, eq_, ai, bj, 3); } while (0)
  bf16x8 At[4][2], B0[2][2], B1[2][2];
  const u32x4 srdA2 = srdA, srdB2 = srdB;
  unsigned cA = u0.a + oa * hA + sp * 32u * kA, cB = u0.b + ob * hB + sp * 32u * kB;
  { const unsigned bh = cB + dB, ah = cA + dA; STAGE_B(SB(0, 0), cB); STAGE_B(SB(0, 1), bh); STAGE_A(SA(0, 0), cA); STAGE_A(SA(0, 1), ah); }
  const Unit u1 = unit_of(1, x, j, rank, peers);
  f32x4 acc[2][2][4][2];
#pragma unroll
  for (int a = 0; a < 2; ++a)
#pragma unroll
    for (int c = 0; c < 2; ++c)
#pragma unroll
      for (int m = 0; m < 4; ++m)
#pragma unroll
        for (int n = 0; n < 2; ++n) acc[a][c][m][n] = zero4();
  if (wr == 1) BAR;
  WAIT_V(2); BAR; MEMF;
  { const unsigned b1 = cB + kB, a1 = cA + kA, b1h = b1 + dB; STAGE_B(SB(1, 0), b1); STAGE_A(SA(1, 0), a1); STAGE_B(SB(1, 1), b1h); }
  asm volatile("" : "+v"(cntv));
  const unsigned call = rfl(cntv) + 1u;
  const unsigned vcall = call;
  WAIT_V(6); BAR; MEMF;
  const unsigned T = ((unsigned)x >> 2) * 32u + (unsigned)j;
  const unsigned pb = WS_PART + T * 4u * UNIT_B + (unsigned)wid * 1024u;
  const unsigned fb = WS_PFLAG + (T * 8u + (unsigned)wid) * 32u;
  const unsigned qo = oa * (128u * NO * 2u) + ob * 256u;
  const unsigned n1 = (unsigned)(NSTAG);
  float rs[8]; u32x4 Xa, Xb, Ya, Yb, ow; unsigned fv;
  unsigned* tmo = (unsigned*)(myws + WS_TMO);
  unsigned bad = 0;
  unsigned nA = u1.a + oa * hA + sp * (unsigned)NK1 * kA, nB = u1.b + ob * hB + sp * (unsigned)NK1 * kB, tl = 30u;
  u32x4 srdO = {u0.olo, u0.ohi, 0u, 0x00020000u}, srdZ = srdC; srdZ[2] = 0;
  unsigned l1 = pb + (sp << 15) + ((sp + 1u) & 3u) * UNIT_B, l2 = pb + (sp << 15) + ((sp + 2u) & 3u) * UNIT_B, l3 = pb + (sp << 15) + ((sp + 3u) & 3u) * UNIT_B, oo = u0.obase + qo, nrec = ~0u;
  int sto = STASH + wid * 1024 + lane * 16;
#define MF(i) MFA(1, i)
#define UNP(L, e) (((e) & 1) ? bhi(L[(e) >> 1]) : blo(L[(e) >> 1]))
#define SP1LOAD(b_, abuf, aso, XA, XB) do { const u32x4 sa_ = (b_) ? srdA2 : srdA; LDB(B0, b_, 0); BLDS(sa_, voffA, aso, 0, ldsw, abuf); LDB(B1, b_, 1); BLDS(sa_, voffA, aso, 64 * K * 2, ldsw, (abuf) + 8192); \
    LDA2(At, b_, 0, 0); XA; LDA2(At, b_, 0, 2); XB; } while (0)
#define TRIP(KIND, t) do { \
    const bool last = (t) == tl; \
    const unsigned a1h = cA + ((t) + 1u) * kA + dA; \
    const unsigned a2 = last ? nA : cA + ((t) + 2u) * kA, b2 = last ? nB : cB + ((t) + 2u) * kB; \
    const unsigned a2h = a2 + dA, b2h = b2 + dB, a3 = a2 + kA, b3 = b2 + kB, b3h = b3 + dB; \
    u32x4 srdA2 = srdA, srdB2 = srdB; srdA2[2] &= last ? nrec : ~0u; srdB2[2] &= last ? nrec : ~0u; \
    if ((KIND) == 3) SP1LOAD(0, SA(1, 1), a1h, BLD4(Xa, srdP, vl16, l1, 0), BLD4(Xb, srdP, vl16, l2, 0)); \
    else if ((KIND) == 2) SP1LOAD(0, SA(1, 1), a1h, BLD1(fv, srdC, vl4, fb), (void)0); \
    else SP1LOAD(0, SA(1, 1), a1h, (void)0, (void)0); \
    WAIT_V((KIND) == 1 ? 20 : (KIND) == 2 ? 9 : (KIND) == 3 ? 12 : (KIND) == 4 ? 10 : 8); \
    WAIT_L(0); BAR; PRIO(1); MSEG(0); PRIO(0); BAR; MEMF; SCHED; \
    LDA(At, 0, 1); STAGE_B(SB(0, 0), b2); STAGE_B(SB(0, 1), b2h); STAGE_A(SA(0, 0), a2); \
    WAIT_V((KIND) == 1 ? 20 : (KIND) == 2 ? 9 : (KIND) == 3 ? 10 : 8); \
    if ((KIND) >= 3) asm volatile("" : "+v"(Ya)); \
    WAIT_L(0); BAR; SCHED; PRIO(1); \
    if ((KIND) >= 3) {   \
      float tp = 0.f; \
      _Pragma("unroll") for (int e = 0; e < 8; ++e) { float q0, q1; \
        MF(4 * e); q0 = UNP(Ya, e); SCHED; MF(4 * e + 1); q1 = UNP(Yb, e); SCHED; MF(4 * e + 2); q0 = fadd1(q0, q1); SCHED; \
        MF(4 * e + 3); q0 = fadd1(rs[e], q0); if (e & 1) ow[e >> 1] = cvtpk(tp, q0); tp = q0; SCHED; } \
    } else MSEG(1); \
    PRIO(0); BAR; MEMF; SCHED; \
    if ((KIND) == 3) { Yb = *(const LAS u32x4*)(lds + sto); SP1LOAD(1, SA(0, 1), a2h, BLD4(Ya, srdP, vl16, l3, 0), BST4("sc0 sc1 nt", ow, srdO, voffO, oo, 0)); } \
    else if ((KIND) == 4) SP1LOAD(1, SA(0, 1), a2h, (void)0, BST4("sc0 sc1 nt", ow, srdO, voffO, oo, 0)); \
    else if ((KIND) == 2) SP1LOAD(1, SA(0, 1), a2h, BST1("", vcall, srdZ, vl4, fb), BST1("", vcall, srdZ, vl4, fb));     \
    else SP1LOAD(1, SA(0, 1), a2h, (void)0, (void)0); \
    WAIT_V((KIND) == 2 ? 11 : (KIND) == 3 ? 12 : (KIND) == 4 ? 9 : 8); \
    if ((KIND) == 1) { const unsigned pubo = fb + sp * 4u; u32x4 sc = srdC; sc[2] = pubo + 4u; BST1("sc1", vcall, sc, vl4, pubo); }     \
    WAIT_L(0); BAR; PRIO(1); MSEG(0); PRIO(0); BAR; MEMF; SCHED; \
    LDA(At, 1, 1); STAGE_B(SB(1, 0), b3); STAGE_B(SB(1, 1), b3h); STAGE_A(SA(1, 0), a3); \
    WAIT_V((KIND) == 1 || (KIND) == 4 ? 9 : (KIND) == 2 || (KIND) == 3 ? 10 : 8); \
    if ((KIND) == 3) asm volatile("" : "+v"(Xa), "+v"(Xb)); \
    if ((KIND) == 2) asm volatile("" : "+v"(fv)); \
    WAIT_L(0); BAR; SCHED; PRIO(1); \
    if ((KIND) == 3) {   \
      _Pragma("unroll") for (int e = 0; e < 8; ++e) { float q0, q1; \
        MF(4 * e); q0 = UNP(Xa, e); SCHED; MF(4 * e + 1); q1 = UNP(Xb, e); SCHED; MF(4 * e + 2); rs[e] = fadd1(q0, q1); SCHED; MF(4 * e + 3); SCHED; } \
      l1 += 8192u; l2 += 8192u; l3 += 8192u; sto += 8192; oo += srdO[2] >> 6; srdO[2] = OUT_BLK;     \
    } else MSEG(1); \
    PRIO(0); BAR; MEMF; SCHED; \
  } while (0)
  auto wait_flags = [&](unsigned f) {
    const unsigned long long t0 = __builtin_amdgcn_s_memrealtime();
    const unsigned pv = lane < 8 ? fb + vl4 : WS_TMO, z = 0;
    for (;;) {
      unsigned v; BLD1(v, srdC, pv, z); WAIT_V(0); asm volatile("" : "+v"(v));
      if (((__builtin_amdgcn_ballot_w64(v >= call) >> (4u * f)) & 15ull) == 15ull) break;
      if ((__builtin_amdgcn_ballot_w64(v == call) >> 63) != 0ull || __builtin_amdgcn_s_memrealtime() - t0 > DEADLINE_TICKS) {
        __hip_atomic_store(tmo, call, __ATOMIC_RELAXED, __HIP_MEMORY_SCOPE_AGENT); bad = 1; break; }
      __builtin_amdgcn_s_sleep(SLP);
    }
    bad = rfl(bad);
  };
  unsigned t = 0;
#pragma unroll 1
  for (; t < 32u; t += 2) TRIP(0, t);
  { const unsigned eb = pb + sp * UNIT_B; E4(true, eb, 0, 1); E4(true, eb, 1, 0); E4(true, eb, 1, 1);
#pragma unroll
    for (int m = 0; m < 4; ++m) { u32x4 w_; PACK(w_, 0, 0, m); *(LAS u32x4*)(lds + sto + m * 8192) = w_; acc[0][0][m][0] = zero4(); acc[0][0][m][1] = zero4(); } }
  cA = nA; cB = nB; tl = (unsigned)NK1 - 2u;
  nrec = 0;
  TRIP(1, 0u);
#pragma unroll 1
  for (t = 2; t < 2u + 2u * n1; t += 2) TRIP(0, t);
  TRIP(2, t); t += 2;
  if ((__builtin_amdgcn_ballot_w64(fv >= call) & 15ull) != 15ull) wait_flags(0u);
  asm volatile("" : "=v"(Ya), "=v"(Yb), "=v"(rs[0]), "=v"(rs[1]), "=v"(rs[2]), "=v"(rs[3]), "=v"(rs[4]), "=v"(rs[5]), "=v"(rs[6]), "=v"(rs[7]));
#pragma unroll 1
  for (int i = 0; i < 4; ++i) { TRIP(3, t); t += 2; }
  TRIP(4, t); t += 2;
#pragma unroll 1
  for (; t < (unsigned)NK1; t += 2) TRIP(0, t);
  { const unsigned eb = pb + ROUND_B + sp * UNIT_B; E4(false, eb, 0, 1); E4(false, eb, 1, 0); E4(false, eb, 1, 1); }
  if (wr == 0) BAR;
  WAIT_V(0);
  { u32x4 sc = srdC; const unsigned pubo = fb + 16u + sp * 4u; sc[2] = pubo + 4u; BST1("sc1", vcall, sc, vl4, pubo); }
  { u32x4 sc = srdC; const unsigned so = WS_CNT + (unsigned)(b * 8 + wid) * 4u; sc[2] = so + 4u; BST1("sc1", vcall, sc, vl4, so); }
  wait_flags(1u);
  u32x4 R[4][3];
  { const unsigned lq = pb + ROUND_B + (sp << 15);
    const unsigned m1 = lq + ((sp + 1u) & 3u) * UNIT_B, m2 = lq + ((sp + 2u) & 3u) * UNIT_B, m3 = lq + ((sp + 3u) & 3u) * UNIT_B;
    BLD4(R[0][0], srdP, vl16, m1, 0); BLD4(R[0][1], srdP, vl16, m2, 0); BLD4(R[0][2], srdP, vl16, m3, 0);
    BLD4(R[1][0], srdP, vl16, m1, 8192); BLD4(R[1][1], srdP, vl16, m2, 8192); BLD4(R[1][2], srdP, vl16, m3, 8192);
    BLD4(R[2][0], srdP, vl16, m1, 16384); BLD4(R[2][1], srdP, vl16, m2, 16384); BLD4(R[2][2], srdP, vl16, m3, 16384);
    BLD4(R[3][0], srdP, vl16, m1, 24576); BLD4(R[3][1], srdP, vl16, m2, 24576); BLD4(R[3][2], srdP, vl16, m3, 24576); }
  {
    const u32x4 so = {u1.olo, u1.ohi, OUT_BLK, 0x00020000u}; const unsigned o = u1.obase + qo;
#pragma unroll
    for (int m = 0; m < 4; ++m) {
      if (m == 0) WAIT_V(9); else if (m == 1) WAIT_V(7); else if (m == 2) WAIT_V(5); else WAIT_V(3);
      asm volatile("" : "+v"(R[m][0]), "+v"(R[m][1]), "+v"(R[m][2]));
      float v[8];
#pragma unroll
      for (int e = 0; e < 4; ++e) { auto r_ = __builtin_amdgcn_permlane16_swap(__float_as_uint(acc[0][0][m][0][e]), __float_as_uint(acc[0][0][m][1][e]), false, false); v[e] = __uint_as_float(r_[0]); v[4 + e] = __uint_as_float(r_[1]); }
      u32x4 w;
#pragma unroll
      for (int e = 0; e < 4; ++e) {
        const float lo_ = (v[2 * e] + blo(R[m][0][e])) + (blo(R[m][1][e]) + blo(R[m][2][e])), hi_ = (v[2 * e + 1] + bhi(R[m][0][e])) + (bhi(R[m][1][e]) + bhi(R[m][2][e]));
        w[e] = bad ? 0x7fc07fc0u : cvtpk(lo_, hi_); }
      if (m == 0) BST4("sc0 sc1 nt", w, so, voffO, o, 0); else if (m == 1) BST4("sc0 sc1 nt", w, so, voffO, o, 16 * NO * 2); else if (m == 2) BST4("sc0 sc1 nt", w, so, voffO, o, 32 * NO * 2); else BST4("sc0 sc1 nt", w, so, voffO, o, 48 * NO * 2);
    }
  }
  WAIT_V(0);
  MEMF; BAR; MEMF;
  if (wid == 0) {
    const unsigned pr = (unsigned)lane;
    const bool mine = pr < (unsigned)NR && pr != (unsigned)rank;
    if (mine) __hip_atomic_store((unsigned*)(peers.ws[pr] + WS_DONE) + rank * 256 + b, call, __ATOMIC_RELAXED, __HIP_MEMORY_SCOPE_SYSTEM);
    bool late = false;
    {
      const unsigned* flag = mine ? (const unsigned*)(myws + WS_DONE) + pr * 256 + b : tmo;
      const unsigned long long t0 = __builtin_amdgcn_s_memrealtime();
      for (;;) {
        const unsigned v = __hip_atomic_load(flag, __ATOMIC_RELAXED, __HIP_MEMORY_SCOPE_SYSTEM);
        if (__builtin_amdgcn_ballot_w64(mine && v < call) == 0ull) break;
        if (__builtin_amdgcn_ballot_w64(!mine && v == call) != 0ull || __builtin_amdgcn_s_memrealtime() - t0 > DEADLINE_TICKS) {
          __hip_atomic_store(tmo, call, __ATOMIC_RELAXED, __HIP_MEMORY_SCOPE_AGENT); late = true; break; }
        __builtin_amdgcn_s_sleep(SLP);
      }
    }
    __builtin_amdgcn_fence(__ATOMIC_ACQUIRE, "");
    if (lane == 0 && (late || __hip_atomic_load(tmo, __ATOMIC_RELAXED, __HIP_MEMORY_SCOPE_AGENT) == call)) peers.out[rank][(size_t)rank * M * NO + (size_t)b] = (u16)0x7fc0;
  }
}

extern "C" void kernel_launch(int rank, int nranks, ncclComm_t comm, hipStream_t stream,
                              void* const* d_in, const int* in_sizes, int n_in,
                              void* d_out, int out_size, const int* in_dtypes, int out_dtype,
                              void* const* d_out_all, void* const* d_ws_all, size_t ws_bytes) {
  if (nranks != NR || (unsigned)rank >= (unsigned)NR || ws_bytes < WS_NEED) return;
  HIP_CHECK(hipFuncSetAttribute((const void*)gemm_a2a, hipFuncAttributeMaxDynamicSharedMemorySize, LDS_BYTES));
  Peers p;
  for (int i = 0; i < NR; ++i) { p.out[i] = (u16*)d_out_all[i]; p.ws[i] = (char*)d_ws_all[i]; }
  gemm_a2a<<<256, 512, LDS_BYTES, stream>>>((const u16*)d_in[0], (const u16*)d_in[1], p, rank);
  HIP_CHECK(hipGetLastError());
}
```
